```python
import numpy as np

kernel_name = "gpu_ag_gemm_m4096_k4096_n4096_bf16_gelu"


ABSMAX_RTOL = 0.008

COMPUTE_DTYPE = "bfloat16"
EPILOGUE = "gelu"
IN_QUANT = None
OUT_QUANT = None
IN_DTYPES = {"x": "bfloat16", "w_mat": "bfloat16"}
WORLD_SIZE = 8

def _to_bf16(x):
    u = np.ascontiguousarray(x, dtype=np.float32).view(np.uint32)
    return ((u + 0x7FFF + ((u >> 16) & 1)) & 0xFFFF0000).view(np.float32)

def setup_inputs():

    x = _to_bf16(np.random.randn(4096, 4096).astype(np.float32))
    w_mat = _to_bf16((np.random.randn(4096, 4096) / np.sqrt(4096)).astype(np.float32))
    return {"x": x, "w_mat": w_mat}

def _epilogue(y):
    c = 0.7978845608028654
    return (0.5 * y * (1.0 + np.tanh(c * (y + 0.044715 * y ** 3)))).astype(np.float32)

def reference(x, w_mat):
    y = (x @ w_mat).astype(np.float32)
    return _epilogue(y)
```

```cpp
#include <hip/hip_runtime.h>
#include <rccl/rccl.h>
#include <cstdio>

#define HIP_CHECK(expr) do { hipError_t _e = (expr); if (_e != hipSuccess) { \
    fprintf(stderr, "HIP error %s at %s:%d\n", hipGetErrorString(_e), __FILE__, __LINE__); \
    return; } } while (0)

namespace {

typedef unsigned u32;
typedef unsigned long long u64;
typedef u32 u32x4 __attribute__((ext_vector_type(4)));
typedef u32 u32x2 __attribute__((ext_vector_type(2)));
typedef float f32x4 __attribute__((ext_vector_type(4)));
typedef __bf16 bf16x8 __attribute__((ext_vector_type(8)));
typedef __bf16 bf16x2 __attribute__((ext_vector_type(2)));
typedef float f32x2 __attribute__((ext_vector_type(2)));

constexpr int NR = 8;
constexpr u32 PANEL = 512u << 10;
constexpr u32 TILEB = 32u << 10;
constexpr size_t OFF_S = 0;
constexpr size_t OFF_P = 32u << 20;
constexpr size_t OFF_AF = 40u << 20;
constexpr size_t OFF_BF = OFF_AF + 16384;
constexpr size_t OFF_CTR = OFF_BF + 32768;
constexpr size_t OFF_TMO = OFF_CTR + 16384;
constexpr size_t OFF_PD = OFF_TMO + 1024;
constexpr size_t OFF_Q = OFF_PD + 1024;
constexpr size_t OFF_DBG = OFF_Q + 2048;
constexpr size_t WS_NEED = OFF_DBG + (256u << 10);
constexpr u64 DEADLINE = 100000000ull;
constexpr int SYS = 17;
constexpr u32 POISON = 0x10000u;
constexpr u32 OOR = 0x80000000u;

#define BWIN 1
#define NOGEMM 0
#define ALIGNED_MAP 1

struct Group { unsigned char dist, slog, qid, nw, wi0, lead, pad0, pad1; u32 slots; };
struct Params {
    const char* x;
    const char* w;
    char* out;
    char* ws[NR];
    Group grp[32];
    int rank;
};

__host__ __device__ __forceinline__ int x_dist(int p) {
    return (0x7520'3210u >> (4 * (p >> 1))) & 7;
}
__host__ __device__ __forceinline__ int x_panel(int p, int b) {
    return ((p >> 3) << 1) | (p & 1);
}
__host__ __device__ __forceinline__ int w_dist(int q) {
    return (int)((0x6530'7610'7430'5410ull >> (4 * q)) & 7);
}
__host__ __device__ __forceinline__ int w_panel(int q, int b) {
    return q >> 2;
}

__device__ __forceinline__ __amdgpu_buffer_rsrc_t rsrc(const void* p, u32 bytes) {
    return __builtin_amdgcn_make_buffer_rsrc(const_cast<void*>(p), 0, bytes, 0x00020000);
}
__device__ __forceinline__ void flag_store(u32* f, u32 v) {
    __hip_atomic_store(f, v, __ATOMIC_RELAXED, __HIP_MEMORY_SCOPE_SYSTEM);
}
__device__ __forceinline__ u32 lds_load(const u32* p) {
    return __builtin_amdgcn_readfirstlane(__hip_atomic_load(p, __ATOMIC_RELAXED, __HIP_MEMORY_SCOPE_WORKGROUP));
}
__device__ __forceinline__ void lds_store(u32* p, u32 v) {
    __hip_atomic_store(p, v, __ATOMIC_RELAXED, __HIP_MEMORY_SCOPE_WORKGROUP);
}
__device__ __forceinline__ u32 pack_bf16(float a, float b) {
    const bf16x2 v = __builtin_convertvector((f32x2){a, b}, bf16x2);
    return __builtin_bit_cast(u32, v);
}
__device__ __forceinline__ float gelu(float y) {
    const float t = y * (-2.302208198f - 0.1029432397f * y * y);
    return y * __builtin_amdgcn_rcpf(1.0f + __builtin_amdgcn_exp2f(t));
}
#define WAIT_VM(n) asm volatile("s_waitcnt vmcnt(" #n ")" ::: "memory")
#define CBAR() asm volatile("" ::: "memory")

#define STAMP(i) do {} while (0)

constexpr u32 STAGE = 49152;
constexpr u32 LDS_POLL = 3 * STAGE;
constexpr u32 LDS_MISC = LDS_POLL + 2048;
constexpr u32 LDS_BYTES = LDS_MISC + 64;
#define LAS __attribute__((address_space(3)))

__global__ void __launch_bounds__(768, 3) ag_gemm_kernel(const Params P) {
    __shared__ __attribute__((aligned(16))) char smem[LDS_BYTES];
    LAS char* const lds = (LAS char*)smem;
    const int wid = __builtin_amdgcn_readfirstlane(threadIdx.x >> 6);
    const int lane = threadIdx.x & 63;
    const int rank = P.rank;
    char* const myws = P.ws[rank];
    if (blockIdx.x < 128 && wid >= 8) return;
    u32* ctr = (u32*)(myws + OFF_CTR) + blockIdx.x * 16 + wid;
    const u32 call = __builtin_amdgcn_readfirstlane(__hip_atomic_load(ctr, __ATOMIC_RELAXED, __HIP_MEMORY_SCOPE_AGENT)) + 1u;
    if (lane == 0) __hip_atomic_store(ctr, call, __ATOMIC_RELAXED, __HIP_MEMORY_SCOPE_AGENT);
    const u32 par = call & 1u;
    u32* const tmo = (u32*)(myws + OFF_TMO);

    if (blockIdx.x < 128) {
        const int g = blockIdx.x;
        if (wid < 2) {
        STAMP(wid * 10);
        const int gi = (g >> 3) * 2 + wid;
        const Group G = P.grp[gi];
        const int dst = (rank ^ G.dist) & 7;
        const u32 slog = G.slog, N = 128u << slog, nw = G.nw, wi = G.wi0 + (g & 7), slots = G.slots, lead = G.lead;
        const u32 dec = 4u * nw - (call - 1u) * N;
        const auto rd = rsrc(P.ws[dst] + OFF_S + (size_t)par * 32 * PANEL, 32 * PANEL);
        const auto rf = rsrc(P.ws[dst] + OFF_AF, 16384);
        const auto rq = rsrc(myws + OFF_Q + G.qid * 128, 4);
        const char* const xs = P.x + (dst & 1) * 4096;
        const char* const wsrc = P.w + (size_t)(dst & 1) * (2u << 20);
        const u32 xbytes = (4u << 20) - (dst & 1) * 4096;
        const u32 l0 = lane ? OOR : 0u, dofs = lane * 16;
        const u32 sox = (lane & 15) * 8192 + (lane >> 4) * 32, sow = (lane >> 4) * 16384 + (lane & 15) * 8;
        auto decode = [&](u32 pos) -> u32 {
            u32 s, i;
            if (slog == 1) { s = pos & 1; i = pos >> 1; }
            else if (pos < 2 * lead) { s = 2 + (pos & 1); i = pos >> 1; }
            else if (pos >= 512 - 2 * lead) { s = pos & 1; i = (pos >> 1) - 128; }
            else { const u32 q = pos - 2 * lead; s = q & 3; i = (q >> 2) + (s >= 2 ? lead : 0u); }
            return pos < N ? ((slots >> (8 * s)) & 31u) * 128 + i : 4096u;
        };
        u32 pa = decode(wi), pb = decode(wi + nw), ca = wi + 2 * nw - dec, cb = wi + 3 * nw - dec, prevf = OOR;
        u32 nst = 0;
        u32x2 a[8], b[8];
#define PLD(r, p) do { const u32 _p = (p) & 4095u, _s = _p >> 7, _u = (_p >> 2) & 31u, _h = _p & 3u; const bool _w = _s >= 16; \
            const auto _rs = rsrc(_w ? wsrc : xs, _w ? (2u << 20) : xbytes); \
            const u32 _o = _w ? (_u * 64 + (_h & 2)) * 1024 + w_panel(_s & 15, dst) * 256 + (_h & 1) * 128 : (x_panel(_s, dst) * 128 + _h * 32) * 8192 + _u * 128; \
            const u32 _v0 = _w ? sow : sox, _v1 = _v0 + (_w ? 1024u : 131072u), _j = _w ? 4096u : 8u; \
            _Pragma("unroll") for (int j = 0; j < 4; ++j) { r[2 * j] = __builtin_amdgcn_raw_buffer_load_b64(_rs, _v0, _o + j * _j, 0); r[2 * j + 1] = __builtin_amdgcn_raw_buffer_load_b64(_rs, _v1, _o + j * _j, 0); } } while (0)
#define PST(r, p) do { const bool _w = (p) >= 2048; const u32 _sa = _w ? 0x05040100u : 0x03020100u, _sb = _w ? 0x07060302u : 0x07060504u; u32x4 f0, f1, f2, f3; \
            _Pragma("unroll") for (int j = 0; j < 4; ++j) { \
                f0[j] = __builtin_amdgcn_perm(r[2 * j + 1].x, r[2 * j].x, _sa); f1[j] = __builtin_amdgcn_perm(r[2 * j + 1].x, r[2 * j].x, _sb); \
                f2[j] = __builtin_amdgcn_perm(r[2 * j + 1].y, r[2 * j].y, _sa); f3[j] = __builtin_amdgcn_perm(r[2 * j + 1].y, r[2 * j].y, _sb); } \
            const u32 _o = (p) < 4096 ? ((p) >> 2) * 16384 + ((p) & 3u) * (_w ? 4096u : 2048u) : OOR, _o2 = _o + (_w ? 2048u : 8192u); \
            __builtin_amdgcn_raw_buffer_store_b128(f0, rd, dofs, _o, SYS); __builtin_amdgcn_raw_buffer_store_b128(f1, rd, dofs + 1024, _o, SYS); \
            __builtin_amdgcn_raw_buffer_store_b128(f2, rd, dofs, _o2, SYS); __builtin_amdgcn_raw_buffer_store_b128(f3, rd, dofs + 1024, _o2, SYS); } while (0)
#define PSTEP(r, p, c) { PST(r, p); const u32 _pn = decode(__builtin_amdgcn_readfirstlane(c) + dec); PLD(r, _pn); \
            c = __builtin_amdgcn_raw_ptr_buffer_atomic_add_i32(1, rq, p < 4096 ? l0 : OOR, 0, 16); WAIT_VM(23); \
            __builtin_amdgcn_raw_buffer_store_b32(call, rf, prevf, 0, SYS); CBAR(); \
            prevf = p < 4096 ? (l0 | (p * 4)) : OOR; nst += p < 4096; p = p < 4096 ? _pn : 4096u; }
        PLD(a, pa); PLD(b, pb); CBAR();
        PSTEP(a, pa, ca) PSTEP(b, pb, cb)
        while (pa < 4096) { PSTEP(a, pa, ca) PSTEP(b, pb, cb) }
        WAIT_VM(0);
        __builtin_amdgcn_raw_buffer_store_b32(call, rf, prevf, 0, SYS);
        CBAR();
        if (lane == 0) __hip_atomic_store((u32*)(myws + OFF_PD) + gi * 8 + (g & 7), call, __ATOMIC_RELAXED, __HIP_MEMORY_SCOPE_AGENT);
        STAMP(wid * 10 + 1);
        }
        const int q = wid >> 1, mp = wid & 1;
        const u32* fb = lane < 2 ? (const u32*)(myws + OFF_BF + g * 128) + lane * 4 + q : tmo;
        bool dead = false;
        const u64 t0 = __builtin_amdgcn_s_memrealtime();
        for (;;) {
            const u32 v = __hip_atomic_load(fb, __ATOMIC_RELAXED, __HIP_MEMORY_SCOPE_SYSTEM);
            if (__ballot(v == call) >> 2) { dead = true; break; }
            if ((__ballot(v >= call) & 3) == 3) break;
            if (__builtin_amdgcn_s_memrealtime() - t0 > DEADLINE) { dead = true; break; }
            __builtin_amdgcn_s_sleep(2);
        }
        CBAR();
        if (dead && lane == 0) __hip_atomic_store(tmo, call, __ATOMIC_RELAXED, __HIP_MEMORY_SCOPE_AGENT);
        if (wid == 2) STAMP(2);
        const auto rp_ = rsrc(myws + OFF_P + (size_t)g * 2 * TILEB + q * 8192, TILEB + 8192);
        const auto ro = rsrc(P.out + (size_t)((g >> 2) * 128 + (q >> 1) * 64 + mp * 32) * 1024 + (g & 3) * 256 + (q & 1) * 128, 31 * 1024 + 128);
        const u32 po = (lane & 1) * 1024 + ((lane >> 1) & 3) * 256 + (lane >> 3) * 16 + mp * 4096;
        const u32 oo = (lane >> 3) * 1024 + (lane & 7) * 16;
        u32x4 p0[4], p1[4];
#pragma unroll
        for (int i = 0; i < 4; ++i) {
            p0[i] = __builtin_amdgcn_raw_buffer_load_b128(rp_, po + (i >> 1) * 2048 + (i & 1) * 128, 0, SYS);
            p1[i] = __builtin_amdgcn_raw_buffer_load_b128(rp_, po + (i >> 1) * 2048 + (i & 1) * 128, TILEB, SYS);
        }
        const float pz = dead ? __builtin_nanf("") : 0.f;
#pragma unroll
        for (int i = 0; i < 4; ++i) {
            u32x4 o;
#pragma unroll
            for (int j = 0; j < 4; ++j) {
                const float lo = __builtin_bit_cast(float, p0[i][j] << 16) + __builtin_bit_cast(float, p1[i][j] << 16) + pz;
                const float hi = __builtin_bit_cast(float, p0[i][j] & 0xffff0000u) + __builtin_bit_cast(float, p1[i][j] & 0xffff0000u) + pz;
                o[j] = pack_bf16(gelu(lo), gelu(hi));
            }
            __builtin_amdgcn_raw_buffer_store_b128(o, ro, oo + i * 8192, 0, 0);
        }
        if (wid == 2) STAMP(5);
        return;
    }

    const int gg = blockIdx.x - 128;
    const int tj = ((gg & 7) >> 1) * 2 + (gg >> 6), tq = (gg & 1) * 8 + ((gg >> 3) & 7);
    bool dead = false;
#define BARRIER() do { asm volatile("s_waitcnt lgkmcnt(0)" ::: "memory"); __builtin_amdgcn_s_barrier(); CBAR(); } while (0)
    LAS u32* const misc = (LAS u32*)(lds + LDS_MISC);
    if (wid >= 8) {
        const int l = wid - 8;
        const u64 t0 = __builtin_amdgcn_s_memrealtime();
        const auto rx = rsrc(myws + OFF_S + (size_t)(par * 32 + 2 * tj) * PANEL, 2 * PANEL);
        const auto rw = rsrc(myws + OFF_S + (size_t)(par * 32 + 16 + tq) * PANEL, PANEL);
        const auto raf = rsrc(myws + OFF_AF, (u32)(OFF_TMO + 4 - OFF_AF));
        const u32 gx = l * 2048 + lane * 16, gw = l * 4096 + lane * 16;
        const u32 fsl = lane >> 4;
        const u32 fo = fsl == 3 ? (u32)(OFF_TMO - OFF_AF) : (fsl == 2 ? 16 + tq : 2 * tj + fsl) * 512 + (lane & 15) * 16 + l * 4;
        const u32 fstep = fsl == 3 ? 0u : 16u;
        LAS u32* const pollbuf = (LAS u32*)(lds + LDS_POLL + l * 256);
        u32 avail = 0, pbase = 0;
#define POLL() do { pbase = avail; __builtin_amdgcn_raw_ptr_buffer_load_lds(raf, pollbuf, 4, fo + min(avail, 31u) * fstep, 0, 0, SYS); } while (0)
#define POLL_READ() do { const u32 _v = *(volatile LAS u32*)(pollbuf + lane); \
        const u64 _m = ~__ballot(_v >= call || pbase + (lane & 15) >= 32); \
        const u32 _a = pbase + __builtin_ctz((((u32)_m | (u32)(_m >> 16) | (u32)(_m >> 32)) & 0xffffu) | 0x10000u); \
        avail = max(avail, _a); if (__ballot(_v == call) >> 48) { dead = true; avail = 64; } } while (0)
#define SPIN(n) do { while (avail < (u32)(n)) { POLL(); WAIT_VM(0); POLL_READ(); \
        if (avail < (u32)(n)) { if (__builtin_amdgcn_s_memrealtime() - t0 > DEADLINE) { dead = true; avail = 64; } else __builtin_amdgcn_s_sleep(1); } } CBAR(); } while (0)
#define DMA1(r, d, v, o) __builtin_amdgcn_raw_ptr_buffer_load_lds(r, d, 16, v, o, 0, SYS)
#define DMA(u, sb) do { LAS char* _x = lds + (sb) + l * 2048; LAS char* _w = lds + (sb) + 32768 + l * 4096; const u32 _o = (u) * 16384; \
        DMA1(rx, _x, gx, _o); DMA1(rx, _x + 1024, gx + 1024, _o); DMA1(rx, _x + 16384, gx, _o + PANEL); DMA1(rx, _x + 17408, gx + 1024, _o + PANEL); \
        DMA1(rw, _w, gw, _o); DMA1(rw, _w + 1024, gw + 1024, _o); DMA1(rw, _w + 2048, gw + 2048, _o); DMA1(rw, _w + 3072, gw + 3072, _o); \
        DMA1(rx, _x + 8192, gx, _o + 8192); DMA1(rx, _x + 9216, gx + 1024, _o + 8192); DMA1(rx, _x + 24576, gx, _o + PANEL + 8192); DMA1(rx, _x + 25600, gx + 1024, _o + PANEL + 8192); CBAR(); } while (0)
        if (lane == 0 && l == 0) misc[0] = 0;
        SPIN(1); DMA(0, 0);
        SPIN(2); POLL(); DMA(1, STAGE);
        u32 sn = 2 * STAGE;
        for (u32 u = 0; u < 30; ++u) {
            WAIT_VM(12); BARRIER();
            POLL_READ();
            SPIN(u + 3); POLL(); DMA(u + 2, sn);
            sn = sn == 2 * STAGE ? 0 : sn + STAGE;
        }
        WAIT_VM(12); BARRIER();
        WAIT_VM(0); BARRIER();
        if (l == 0 && w_dist(tq) != 0) {
            const auto rpd = rsrc(myws + OFF_PD, 1024);
            while (!dead) {
                const u32x4 v = __builtin_amdgcn_raw_buffer_load_b128(rpd, lane * 16, 0, 16);
                CBAR();
                if (__all(v.x >= call && v.y >= call && v.z >= call && v.w >= call)) break;
                if (__builtin_amdgcn_s_memrealtime() - t0 > DEADLINE) { dead = true; break; }
                __builtin_amdgcn_s_sleep(2);
            }
        }
        if (dead && lane == 0) { misc[0] = 1; __hip_atomic_store(tmo, call, __ATOMIC_RELAXED, __HIP_MEMORY_SCOPE_AGENT); }
        BARRIER();
        return;
    }
    const int wr = wid >> 1, wc = wid & 1;
#define COMPUTE(sb) do { const LAS char* _x = lds + (sb) + xo; const LAS char* _w = lds + (sb) + wo; bf16x8 xf[2][4], wf[2][4]; \
        _Pragma("unroll") for (int k = 0; k < 2; ++k) _Pragma("unroll") for (int i = 0; i < 4; ++i) { \
            wf[k][i] = *(const LAS bf16x8*)(_w + k * 8192 + i * 1024); xf[k][i] = *(const LAS bf16x8*)(_x + k * 8192 + i * 1024); } \
        __builtin_amdgcn_sched_barrier(0); \
        _Pragma("unroll") for (int k = 0; k < 2; ++k) _Pragma("unroll") for (int m = 0; m < 4; ++m) _Pragma("unroll") for (int n = 0; n < 4; ++n) \
            acc[m][n] = __builtin_amdgcn_mfma_f32_16x16x32_bf16(wf[k][n], xf[k][m], acc[m][n], 0, 0, 0); \
        __builtin_amdgcn_sched_barrier(0); } while (0)
    const u32 xo = (wr >> 1) * 16384 + (wr & 1) * 4096 + lane * 16, wo = 32768 + wc * 4096 + lane * 16;
    f32x4 acc[4][4];
#pragma unroll
    for (int m = 0; m < 4; ++m)
#pragma unroll
        for (int n = 0; n < 4; ++n) acc[m][n] = (f32x4){0.f, 0.f, 0.f, 0.f};
    u32 sb = 0;
    for (u32 u = 0; u < 32; ++u) {
        BARRIER();
        COMPUTE(sb);
        sb = sb == 2 * STAGE ? 0 : sb + STAGE;
    }
    if (wid == 0) STAMP(3);
    BARRIER();
    dead = *(volatile LAS u32*)misc != 0;
    if (wid == 0) STAMP(6);
    {
        const int tp = 2 * tj + (wr >> 1), quad = wid & 3;
        const int owner = rank ^ w_dist(tq);
        const int tile = ((rank ^ x_dist(tp)) * 4 + x_panel(tp, rank)) * 4 + w_panel(tq, rank);
        const auto rp_ = rsrc(P.ws[owner] + OFF_P + (size_t)(tile * 2 + (rank & 1)) * TILEB + quad * 8192, 8192);
        const float pz = dead ? __builtin_nanf("") : 0.f;
#pragma unroll
        for (int m = 0; m < 4; ++m)
#pragma unroll
            for (int h = 0; h < 2; ++h) {
                u32x4 v;
                v.x = pack_bf16(acc[m][0][2 * h] + pz, acc[m][1][2 * h]);
                v.y = pack_bf16(acc[m][2][2 * h], acc[m][3][2 * h]);
                v.z = pack_bf16(acc[m][0][2 * h + 1], acc[m][1][2 * h + 1]);
                v.w = pack_bf16(acc[m][2][2 * h + 1], acc[m][3][2 * h + 1]);
                __builtin_amdgcn_raw_buffer_store_b128(v, rp_, lane * 16, (m * 2 + h) * 1024, SYS);
                WAIT_VM(0);
            }
        WAIT_VM(0);
        if (lane == 0) flag_store((u32*)(P.ws[owner] + OFF_BF + tile * 128) + (rank & 1) * 4 + quad, call);
        if (wid == 0) STAMP(4);
    }
}

#define LEAD 12
static void make_groups(Group* grp) {
    int g = 0, qid = 0;
    for (int dd = 1; dd <= 8; ++dd) {
        const int d = dd & 7;
        int xs[4], ws[4], nx = 0, nwv = 0;
        for (int s = 0; s < 16; ++s) { if (x_dist(s) == d) xs[nx++] = s; if (w_dist(s) == d) ws[nwv++] = 16 + s; }
        int lists[2][4], ln[2] = {0, 0}, lead[2] = {0, 0}, nl = 0;
        if (nx == 4 && nwv == 4) { for (int h = 0; h < 2; ++h) { lists[h][0] = xs[2 * h]; lists[h][1] = xs[2 * h + 1]; lists[h][2] = ws[2 * h]; lists[h][3] = ws[2 * h + 1]; ln[h] = 4; } nl = 2; }
        else { for (int i = 0; i < nx; ++i) lists[0][ln[0]++] = xs[i]; for (int i = 0; i < nwv; ++i) lists[0][ln[0]++] = ws[i]; nl = 1; if (nx == 2 && nwv == 2) lead[0] = LEAD; }
        for (int h = 0; h < nl; ++h) {
            unsigned slots = 0; for (int i = 0; i < ln[h]; ++i) slots |= (unsigned)lists[h][i] << (8 * i);
            const int ng = ln[h];
            for (int i = 0; i < ng; ++i, ++g) {
                Group& G = grp[g];
                G.dist = d; G.slog = ln[h] == 4 ? 2 : 1; G.qid = qid; G.nw = 8 * ng; G.wi0 = 8 * i; G.lead = lead[h]; G.pad0 = G.pad1 = 0; G.slots = slots;
            }
            ++qid;
        }
    }
    if (g != 32) fprintf(stderr, "group table broken: %d\n", g);
}

}

extern "C" void kernel_launch(int rank, int nranks, ncclComm_t comm,
                              hipStream_t stream,
                              void* const* d_in, const int* in_sizes, int n_in,
                              void* d_out, int out_size,
                              const int* in_dtypes, int out_dtype,
                              void* const* d_out_all,
                              void* const* d_ws_all, size_t ws_bytes) {
    if (nranks != NR || (unsigned)rank >= (unsigned)NR || ws_bytes < WS_NEED || n_in < 2 ||
        in_sizes[0] != 512 * 4096 || in_sizes[1] != 4096 * 512 || out_size != 4096 * 512) {
        fprintf(stderr, "unexpected problem shape\n");
        return;
    }
    Params p;
    p.x = (const char*)d_in[0];
    p.w = (const char*)d_in[1];
    p.out = (char*)d_out;
    for (int i = 0; i < NR; ++i) p.ws[i] = (char*)d_ws_all[i];
    make_groups(p.grp);
    p.rank = rank;
    hipLaunchKernelGGL(ag_gemm_kernel, dim3(256), dim3(768), 0, stream, p);
    HIP_CHECK(hipGetLastError());
}
```

```cpp
#define NOGATE 1
#include <hip/hip_runtime.h>
#include <rccl/rccl.h>
#include <cstdio>

#define HIP_CHECK(expr) do { hipError_t _e = (expr); if (_e != hipSuccess) { \
    fprintf(stderr, "HIP error %s at %s:%d\n", hipGetErrorString(_e), __FILE__, __LINE__); \
    return; } } while (0)

namespace {

typedef unsigned u32;
typedef unsigned long long u64;
typedef u32 u32x4 __attribute__((ext_vector_type(4)));
typedef u32 u32x2 __attribute__((ext_vector_type(2)));
typedef float f32x4 __attribute__((ext_vector_type(4)));
typedef __bf16 bf16x8 __attribute__((ext_vector_type(8)));
typedef short s16x4 __attribute__((ext_vector_type(4)));
typedef __bf16 bf16x2 __attribute__((ext_vector_type(2)));
typedef float f32x2 __attribute__((ext_vector_type(2)));

constexpr int NR = 8;
constexpr u32 PANEL = 512u << 10;
constexpr u32 TILEB = 32u << 10;
constexpr u32 OFF_S = 0;
constexpr u32 OFF_P = 32u << 20;
constexpr u32 OFF_T = 40u << 20;
constexpr u32 OFF_AF = 48u << 20;
constexpr u32 OFF_BF = OFF_AF + 16384;
constexpr u32 OFF_RF = OFF_BF + 16384;
constexpr u32 OFF_CTR = OFF_RF + 4096;
constexpr u32 OFF_TMO = OFF_CTR + 16384;
constexpr u32 OFF_Q = OFF_TMO + 1024;
constexpr u32 OFF_LT = OFF_Q + 4096;
constexpr u32 OFF_DBG = OFF_LT + 1024;
constexpr u32 WS_NEED = OFF_DBG + (256u << 10);
constexpr u64 DEADLINE = 100000000ull;
constexpr int SYS = 17;
constexpr u32 OOR = 0x80000000u;

#define NE 8
#define LEAD 32
#define NPW 4
#define GSLEEP 1
#define RSLEEP 2
#define DSLEEP 1
#define GAUX 17
#define WIN 2
#define HW 128
#define GL0 4
#define RPP 2
#define GL 2
#define GL2 GL
#define GL4 GL
#define GLR 2

struct Group {
    u32 dist, qid, nw, wi0, two, lead, nlead;
    u32 ea, er, eb;
    u32 slots, rsrc_slots, rdst_slots, bslots;
};
struct PGroup { u32 w[8]; };
__host__ __device__ __forceinline__ PGroup pack(const Group& G) {
    return PGroup{{G.dist | G.qid << 8 | G.nw << 16 | G.wi0 << 24, G.two | G.lead << 8 | G.nlead << 16, G.ea | G.er << 16, G.eb | G.bslots << 16, G.slots, G.rsrc_slots, G.rdst_slots, 0u}};
}
__host__ __device__ __forceinline__ Group unpack(const PGroup& p) {
    return Group{p.w[0] & 7u, (p.w[0] >> 8) & 15u, (p.w[0] >> 16) & 255u, p.w[0] >> 24, p.w[1] & 1u, (p.w[1] >> 8) & 255u, p.w[1] >> 16,
                 p.w[2] & 0xffffu, p.w[2] >> 16, p.w[3] & 0xffffu, p.w[4], p.w[5], p.w[6], p.w[3] >> 16};
}
struct Params {
    const char* x;
    const char* w;
    char* out;
    char* ws[NR];
    int rank, pad[9];
    PGroup grp[16 * NPW];
};

__host__ __device__ __forceinline__ int x_dist(int p) { return (0x7520'3210u >> (4 * (p >> 1))) & 7; }
__host__ __device__ __forceinline__ int x_panel(int p) { return ((p >> 3) << 1) | (p & 1); }
__host__ __device__ __forceinline__ int w_dist(int q) { return (int)((0x6530'7610'7430'5410ull >> (4 * q)) & 7); }
__host__ __device__ __forceinline__ int w_panel(int q) { return q >> 2; }

__device__ __forceinline__ __amdgpu_buffer_rsrc_t rsrc(const void* p, u32 bytes) {
    return __builtin_amdgcn_make_buffer_rsrc(const_cast<void*>(p), 0, bytes, 0x00020000);
}
__device__ __forceinline__ u32 pack_bf16(float a, float b) {
    const bf16x2 v = __builtin_convertvector((f32x2){a, b}, bf16x2);
    return __builtin_bit_cast(u32, v);
}
__device__ __forceinline__ float gelu(float y) {
    const float t = y * (-2.302208198f - 0.1029432397f * y * y);
    return y * __builtin_amdgcn_rcpf(1.0f + __builtin_amdgcn_exp2f(t));
}
#define WAIT_VM(n) asm volatile("s_waitcnt vmcnt(" #n ")" ::: "memory")
#define CBAR() asm volatile("" ::: "memory")
#define RFL(x) ((u32)__builtin_amdgcn_readfirstlane(x))

#define STAMP(i) do {} while (0)

constexpr u32 STAGE = 49152;
constexpr u32 LDS_POLL = 3 * STAGE;
constexpr u32 LDS_MISC = LDS_POLL + 1024;
constexpr u32 LDS_BYTES = LDS_MISC + 64;
#define LAS __attribute__((address_space(3)))

__global__ void __launch_bounds__(768, 3) ag_gemm_kernel(const Params P) {
    __shared__ __attribute__((aligned(16))) char smem[LDS_BYTES];
    LAS char* const lds = (LAS char*)smem;
    const int wid = RFL(threadIdx.x >> 6);
    const int lane = threadIdx.x & 63;
    const int rank = P.rank;
    char* const myws = P.ws[rank];
    u32* const ctr = (u32*)(myws + OFF_CTR) + blockIdx.x * 16 + wid;
    const u32 ctrv = __hip_atomic_load(ctr, __ATOMIC_RELAXED, __HIP_MEMORY_SCOPE_AGENT);
    u32* const tmo = (u32*)(myws + OFF_TMO);
    const u32 l0 = lane ? OOR : 0u, l16 = lane * 16;

#define BARRIER() do { asm volatile("s_waitcnt lgkmcnt(0)" ::: "memory"); __builtin_amdgcn_s_barrier(); CBAR(); } while (0)
#define LDS_LD(p) RFL(*(volatile LAS u32*)(p))
#define LDS_ST(p, v) do { *(volatile LAS u32*)(p) = (v); } while (0)
    if (blockIdx.x < 128) {
        const int g = blockIdx.x, pair = wid & 3;
        bool dead = false;
        const Group G = unpack(P.grp[pair * 16 + (g >> 3)]);
        const int dst = (rank ^ G.dist) & 7;
        LAS char* const ring = lds + pair * 16384;
        LAS u32* const rdesc = (LAS u32*)(lds + 65536 + pair * 128);
        const auto rq = rsrc(myws + OFF_Q + G.qid * 256, 256);
        enum { K_COPY = 0, K_W = 1, K_SIG = 2, K_END = 3 };
        if (wid >= 8) {
            if (lane < 2) rdesc[16 + lane] = 0;
            BARRIER();
            const u32 call = RFL(ctrv) + 1u;
            if (lane == 0) __hip_atomic_store(ctr, call, __ATOMIC_RELAXED, __HIP_MEMORY_SCOPE_AGENT);
            if (!G.nw) return;
            const u64 t0 = __builtin_amdgcn_s_memrealtime();
            const u32 nw = G.nw, lead = G.lead, nlead = G.nlead, nr = G.er;
            const u32 sbase = OFF_S + (call & 1u) * 32 * PANEL;
            const u32 dec = 2u * nw - (call - 1u) * G.eb;
            const u32 urev = (dst & 4) ? 31u : 0u;
            const char* const xs = P.x + (dst & 1) * 4096;
            const char* const wsrc = P.w + (size_t)(dst & 1) * (2u << 20);
            const u32 xbytes = (4u << 20) - (dst & 1) * 4096;
            const auto rm = rsrc(myws, WS_NEED);
            const u32 vx = (lane & 15) * 8192 + (lane >> 4) * 16, vw = (lane >> 3) * 1024 + (lane & 7) * 16;
            const u32 dvr = OFF_AF + ((G.rsrc_slots >> (8 * (lane >> 4))) & 31u) * 512 + (lane & 15) * 16, dvb = OFF_RF + G.dist * 512 + (lane & 31) * 16;
            u32x4 dv = {0u, 0u, 0u, 0u};
            u32 cur = G.wi0 + (g & 7), c = cur + nw - dec, j = 0, pub = 0, used = 0, nst = 0;
            bool sig = G.eb == G.ea, gate = false;
            auto timed_out = [&]() { return __builtin_amdgcn_s_memrealtime() - t0 > DEADLINE; };
            auto publish = [&](u32 n) {
                if (n == pub) return;
                while (gate && !dead) {
                    const u32 v = __builtin_amdgcn_raw_buffer_load_b32(rq, 0, 128, 16);
                    CBAR();
                    if (RFL(v) >= call * nw) break;
                    if (timed_out()) dead = true;
                    __builtin_amdgcn_s_sleep(1);
                }
                if (gate) STAMP(pair * 10 + 7);
                gate = false;
                pub = n; LDS_ST(rdesc + 16, n);
            };
            auto slot_wait = [&]() {
                if (used + 4 > j) return;
                used = LDS_LD(rdesc + 17);
                if (used + 4 > j) return;
                WAIT_VM(0); publish(j);
                while (used + 4 <= j) { __builtin_amdgcn_s_sleep(1); used = LDS_LD(rdesc + 17); }
            };
            auto word = [&](u32 w) -> u32 {
                const u32 l = w >> 2;
                const u32 v0 = __builtin_amdgcn_readlane(dv.x, l), v1 = __builtin_amdgcn_readlane(dv.y, l), v2 = __builtin_amdgcn_readlane(dv.z, l), v3 = __builtin_amdgcn_readlane(dv.w, l);
                return (w & 2) ? ((w & 1) ? v3 : v2) : ((w & 1) ? v1 : v0);
            };
            auto dep_wait = [&](u32 w, u32 sh, u32 dvo) {
                if ((word(w) >> sh) >= call) return;
                WAIT_VM(0); publish(j);
                while (!dead) {
                    dv = __builtin_amdgcn_raw_buffer_load_b128(rm, dvo, 0, SYS);
                    u32 t = RFL(__hip_atomic_load(tmo, __ATOMIC_RELAXED, __HIP_MEMORY_SCOPE_AGENT));
                    asm volatile("" : "+s"(t));
                    if ((word(w) >> sh) >= call) break;
                    if (t == call || timed_out()) dead = true;
                    __builtin_amdgcn_s_sleep(DSLEEP);
                }
                CBAR();
            };
            auto post = [&](u32 kind) { slot_wait(); LDS_ST(rdesc + (j & 3) * 4 + 3, kind); WAIT_VM(0); ++j; publish(j); };
            for (;;) {
                if (cur >= G.ea && !sig) { sig = true; dv = (u32x4){0u, 0u, 0u, 0u}; STAMP(pair * 10 + 1); }
                if (cur >= G.eb) break;
                u32 kind = K_COPY, so, s1 = 1024, s2 = 2048, dof, d2 = 2048, fo, vo = l16, src = 2;
                if (cur >= G.ea) {
                    const u32 r = cur - G.ea, hf = r & 1;
                    dep_wait((r >> 1) & 127u, 10, dvb);
                    const u32 id = word((r >> 1) & 127u) & 1023u, tj = id >> 7, tq = (id >> 3) & 15u, wv = id & 7u;
                    const u32 tp = 2 * tj + (wv >> 2), tile = ((rank ^ x_dist(tp)) * 4 + x_panel(tp)) * 4 + w_panel(tq);
                    so = OFF_T + id * 8192 + hf * 4096;
                    dof = OFF_P + (tile * 2 + (rank & 1)) * TILEB + (wv & 3) * 8192 + hf * 4096;
                    fo = OFF_BF + tile * 128 + ((rank & 1) * 8 + (wv & 3) * 2 + hf) * 4;
                } else if (G.two && cur - HW < nr) {
                    const u32 r = cur - HW, k = r & 3, i = r >> 2, sd = (G.rdst_slots >> (8 * k)) & 31u;
                    dep_wait(k * 64 + i, 0, dvr);
                    so = sbase + ((G.rsrc_slots >> (8 * k)) & 31u) * PANEL + i * 4096;
                    dof = sbase + sd * PANEL + (i ^ 124u) * 4096;
                    fo = OFF_AF + (sd * 128 + (i ^ 124u)) * 4;
                } else {
                    u32 s, i;
                    if (G.two) { const u32 p = cur < HW ? cur : cur - nr; s = p & 1; i = p >> 1; }
                    else if (cur < 2 * lead) { s = 2 + (cur & 1); i = cur >> 1; }
                    else if (cur >= 4 * nlead - 2 * lead) { s = cur & 1; i = (cur >> 1) - nlead; }
                    else { const u32 q = cur - 2 * lead; s = q & 3; i = (q >> 2) + (s >= 2 ? lead : 0u); }
                    const u32 slot = (G.slots >> (8 * s)) & 31u, u = (i >> 2) ^ urev, h = i & 3;
                    fo = OFF_AF + (slot * 128 + i) * 4;
                    dof = sbase + slot * PANEL + (i >> 2) * 16384;
                    if (slot >= 16) { kind = K_W; src = 1; vo = vw; s1 = 8192; s2 = 16384; so = (u * 64 + (h >> 1) * 32) * 1024 + w_panel(slot & 15) * 256 + (h & 1) * 128; dof += h * 4096; }
                    else { src = 0; vo = vx; s1 = 131072; s2 = 64; d2 = 8192; so = (x_panel(slot) * 128 + h * 32) * 8192 + u * 128; dof += h * 2048; }
                }
                slot_wait();
                const auto rs = rsrc(src == 2 ? (const char*)myws : src ? wsrc : xs, src == 2 ? WS_NEED : src ? (2u << 20) : xbytes);
                LAS char* const sl = ring + (j & 3) * 4096;
                __builtin_amdgcn_raw_ptr_buffer_load_lds(rs, sl, 16, vo, so, 0, SYS);
                __builtin_amdgcn_raw_ptr_buffer_load_lds(rs, sl + 1024, 16, vo, so + s1, 0, SYS);
                __builtin_amdgcn_raw_ptr_buffer_load_lds(rs, sl + 2048, 16, vo, so + s2, 0, SYS);
                __builtin_amdgcn_raw_ptr_buffer_load_lds(rs, sl + 3072, 16, vo, so + s1 + s2, 0, SYS);
                *(volatile LAS u32x4*)(rdesc + (j & 3) * 4) = (u32x4){dof, d2, fo, kind};
                WAIT_VM(4);
                publish(j);
                cur = RFL(c) + dec;
                c = __builtin_amdgcn_raw_ptr_buffer_atomic_add_i32(1, rq, l0, 0, 16);
                CBAR();
                ++j; ++nst;
            }
            post(K_END);
            if (dead && lane == 0) __hip_atomic_store(tmo, call, __ATOMIC_RELAXED, __HIP_MEMORY_SCOPE_AGENT);
            return;
        }
        BARRIER();
        const u32 call = RFL(ctrv) + 1u;
        if (lane == 0) __hip_atomic_store(ctr, call, __ATOMIC_RELAXED, __HIP_MEMORY_SCOPE_AGENT);
        if (wid < 4 && G.nw) {
            STAMP(pair * 10);
            const auto rd = rsrc(P.ws[dst], WS_NEED);
            const u32 tro = ((lane >> 4) * 8 + ((lane >> 2) & 3)) * 128 + (lane & 3) * 32;
            u32 k = 0, full = 0, pend = OOR;
#define FLAG() do { __builtin_amdgcn_raw_buffer_store_b32(call, rd, pend, 0, SYS); CBAR(); pend = OOR; } while (0)
            for (;;) {
                if (full <= k) {
                    full = LDS_LD(rdesc + 16);
                    if (full <= k) {
                        WAIT_VM(0); FLAG();
                        while (full <= k) { __builtin_amdgcn_s_sleep(1); full = LDS_LD(rdesc + 16); }
                    }
                    CBAR();
                }
                const u32x4 dd = *(volatile LAS u32x4*)(rdesc + (k & 3) * 4);
                const u32 dof = RFL(dd.x), d2 = RFL(dd.y), fo = RFL(dd.z), kind = RFL(dd.w);
                if (kind >= K_SIG) {
                    WAIT_VM(0); FLAG();
                    if (kind == K_END) break;
                    (void)__builtin_amdgcn_raw_ptr_buffer_atomic_add_i32(1, rq, l0, 128, 16);
                    STAMP(pair * 10 + 1);
                    ++k; LDS_ST(rdesc + 17, k);
                    continue;
                }
                const LAS char* const sl = ring + (k & 3) * 4096;
                u32x4 f[4];
                if (kind == K_W) {
#pragma unroll
                    for (int n = 0; n < 4; ++n) {
                        const u32x2 lo = __builtin_bit_cast(u32x2, __builtin_amdgcn_ds_read_tr16_b64_v4i16((LAS s16x4*)(sl + tro + n * 8)));
                        const u32x2 hi = __builtin_bit_cast(u32x2, __builtin_amdgcn_ds_read_tr16_b64_v4i16((LAS s16x4*)(sl + tro + 512 + n * 8)));
                        f[n] = (u32x4){lo.x, lo.y, hi.x, hi.y};
                    }
                } else {
#pragma unroll
                    for (int n = 0; n < 4; ++n) f[n] = *(const LAS u32x4*)(sl + l16 + n * 1024);
                }
                asm volatile("s_waitcnt lgkmcnt(0)" : "+v"(f[0]), "+v"(f[1]), "+v"(f[2]), "+v"(f[3]) :: "memory");
                ++k; LDS_ST(rdesc + 17, k);
                __builtin_amdgcn_raw_buffer_store_b128(f[0], rd, l16, dof, SYS);
                __builtin_amdgcn_raw_buffer_store_b128(f[1], rd, l16 + 1024, dof, SYS);
                __builtin_amdgcn_raw_buffer_store_b128(f[2], rd, l16, dof + d2, SYS);
                __builtin_amdgcn_raw_buffer_store_b128(f[3], rd, l16 + 1024, dof + d2, SYS);
                WAIT_VM(4); FLAG();
                pend = l0 | fo;
            }
            STAMP(pair * 10 + 4);
        }
        const int q = wid >> 1, mp = wid & 1;
        const u32* fb = lane < 2 ? (const u32*)(myws + OFF_BF + g * 128) + lane * 8 + wid : tmo;
        const u64 t0 = __builtin_amdgcn_s_memrealtime();
        while (!dead) {
            const u32 v = __hip_atomic_load(fb, __ATOMIC_RELAXED, __HIP_MEMORY_SCOPE_SYSTEM);
            if (__ballot(v == call) >> 2) { dead = true; break; }
            if ((__ballot(v >= call) & 3) == 3) break;
            if (__builtin_amdgcn_s_memrealtime() - t0 > DEADLINE) { dead = true; break; }
            __builtin_amdgcn_s_sleep(RSLEEP);
        }
        CBAR();
        if (dead && lane == 0) __hip_atomic_store(tmo, call, __ATOMIC_RELAXED, __HIP_MEMORY_SCOPE_AGENT);
        if (wid == 7) STAMP(40);
        const auto rp_ = rsrc(myws + OFF_P + (size_t)g * 2 * TILEB + q * 8192, TILEB + 8192);
        const auto ro = rsrc(P.out + (size_t)((g >> 2) * 128 + (q >> 1) * 64 + mp * 32) * 1024 + (g & 3) * 256 + (q & 1) * 128, 31 * 1024 + 128);
        const u32 po = (lane & 1) * 1024 + ((lane >> 1) & 3) * 256 + (lane >> 3) * 16 + mp * 4096;
        const u32 oo = (lane >> 3) * 1024 + (lane & 7) * 16;
        u32x4 p0[4], p1[4];
#pragma unroll
        for (int i = 0; i < 4; ++i) {
            p0[i] = __builtin_amdgcn_raw_buffer_load_b128(rp_, po + (i >> 1) * 2048 + (i & 1) * 128, 0, SYS);
            p1[i] = __builtin_amdgcn_raw_buffer_load_b128(rp_, po + (i >> 1) * 2048 + (i & 1) * 128, TILEB, SYS);
        }
        const float pz = dead ? __builtin_nanf("") : 0.f;
#pragma unroll
        for (int i = 0; i < 4; ++i) {
            u32x4 o;
#pragma unroll
            for (int j = 0; j < 4; ++j) {
                const float lo = __builtin_bit_cast(float, p0[i][j] << 16) + __builtin_bit_cast(float, p1[i][j] << 16) + pz;
                const float hi = __builtin_bit_cast(float, p0[i][j] & 0xffff0000u) + __builtin_bit_cast(float, p1[i][j] & 0xffff0000u) + pz;
                o[j] = pack_bf16(gelu(lo), gelu(hi));
            }
            __builtin_amdgcn_raw_buffer_store_b128(o, ro, oo + i * 8192, 0, 0);
        }
        STAMP(41 + wid);
        return;
    }

    const u32 call = RFL(ctrv) + 1u;
    if (lane == 0) __hip_atomic_store(ctr, call, __ATOMIC_RELAXED, __HIP_MEMORY_SCOPE_AGENT);
    const u32 par = call & 1u;
    const int gg = blockIdx.x - 128;
    const int tj = ((gg & 7) >> 1) * 2 + (gg >> 6), tq = (gg & 1) * 8 + ((gg >> 3) & 7);
    LAS u32* const misc = (LAS u32*)(lds + LDS_MISC);
    if (wid >= 8) {
        const int l = wid - 8;
        bool dead = false;
        const u64 t0 = __builtin_amdgcn_s_memrealtime();
        const auto rx = rsrc(myws + OFF_S + (size_t)(par * 32 + 2 * tj) * PANEL, 2 * PANEL);
        const auto rw = rsrc(myws + OFF_S + (size_t)(par * 32 + 16 + tq) * PANEL, PANEL);
        const auto raf = rsrc(myws + OFF_AF, OFF_TMO + 4 - OFF_AF);
        const u32 gx = l * 2048 + l16, gw = l * 4096 + l16;
        const u32 fsl = lane >> 4;
        const u32 fo = fsl == 3 ? OFF_TMO - OFF_AF : (fsl == 2 ? 16 + tq : 2 * tj + fsl) * 512 + (lane & 15) * 16 + l * 4;
        const u32 fstep = fsl == 3 ? 0u : 16u;
        LAS u32* const pollbuf = (LAS u32*)(lds + LDS_POLL + l * 256);
        u32 avail = 0, pbase = 0;
#define POLL() do { pbase = avail; __builtin_amdgcn_raw_ptr_buffer_load_lds(raf, pollbuf, 4, fo + min(avail, 31u) * fstep, 0, 0, SYS); } while (0)
#define POLL_READ() do { const u32 _v = *(volatile LAS u32*)(pollbuf + lane); \
        const u64 _m = ~__ballot(_v >= call || pbase + (lane & 15) >= 32); \
        const u32 _a = pbase + __builtin_ctz((((u32)_m | (u32)(_m >> 16) | (u32)(_m >> 32)) & 0xffffu) | 0x10000u); \
        avail = max(avail, _a); if (__ballot(_v == call) >> 48) { dead = true; avail = 64; } } while (0)
#define SPIN(n) do { while (avail < (u32)(n)) { POLL(); WAIT_VM(0); POLL_READ(); \
        if (avail < (u32)(n)) { if (__builtin_amdgcn_s_memrealtime() - t0 > DEADLINE) { dead = true; avail = 64; } else __builtin_amdgcn_s_sleep(GSLEEP); } } CBAR(); } while (0)
#define DMA1(r, d, v, o) __builtin_amdgcn_raw_ptr_buffer_load_lds(r, d, 16, v, o, 0, GAUX)
#define DMA(u, sb) do { LAS char* _x = lds + (sb) + l * 2048; LAS char* _w = lds + (sb) + 32768 + l * 4096; const u32 _o = (u) * 16384; \
        DMA1(rx, _x, gx, _o); DMA1(rx, _x + 1024, gx + 1024, _o); DMA1(rx, _x + 16384, gx, _o + PANEL); DMA1(rx, _x + 17408, gx + 1024, _o + PANEL); \
        DMA1(rw, _w, gw, _o); DMA1(rw, _w + 1024, gw + 1024, _o); DMA1(rw, _w + 2048, gw + 2048, _o); DMA1(rw, _w + 3072, gw + 3072, _o); \
        DMA1(rx, _x + 8192, gx, _o + 8192); DMA1(rx, _x + 9216, gx + 1024, _o + 8192); DMA1(rx, _x + 24576, gx, _o + PANEL + 8192); DMA1(rx, _x + 25600, gx + 1024, _o + PANEL + 8192); CBAR(); } while (0)
        if (lane == 0) { misc[l] = 0; misc[8 + l] = 0; }
        SPIN(1); DMA(0, 0);
        SPIN(2); POLL(); DMA(1, STAGE);
        u32 sn = 2 * STAGE;
        for (u32 u = 0; u < 30; ++u) {
            WAIT_VM(12); BARRIER();
            POLL_READ();
            SPIN(u + 3); POLL(); DMA(u + 2, sn);
            sn = sn == 2 * STAGE ? 0 : sn + STAGE;
        }
        if (l == 0) STAMP(2);
        if (dead && lane == 0) { misc[l] = 1; __hip_atomic_store(tmo, call, __ATOMIC_RELAXED, __HIP_MEMORY_SCOPE_AGENT); }
        WAIT_VM(12); BARRIER();
        if (l == 0 && lane == 0)
            misc[4] = __hip_atomic_fetch_add((u32*)(myws + OFF_LT + w_dist(tq) * 128), 1u, __ATOMIC_RELAXED, __HIP_MEMORY_SCOPE_AGENT);
        WAIT_VM(0); BARRIER();
        return;
    }
    const int wr = wid >> 1, wc = wid & 1;
#define COMPUTE(sb) do { const LAS char* _x = lds + (sb) + xo; const LAS char* _w = lds + (sb) + wo; bf16x8 xf[2][4], wf[2][4]; \
        _Pragma("unroll") for (int k = 0; k < 2; ++k) _Pragma("unroll") for (int i = 0; i < 4; ++i) { \
            wf[k][i] = *(const LAS bf16x8*)(_w + k * 8192 + i * 1024); xf[k][i] = *(const LAS bf16x8*)(_x + k * 8192 + i * 1024); } \
        __builtin_amdgcn_sched_barrier(0); \
        _Pragma("unroll") for (int k = 0; k < 2; ++k) _Pragma("unroll") for (int m = 0; m < 4; ++m) _Pragma("unroll") for (int n = 0; n < 4; ++n) \
            acc[m][n] = __builtin_amdgcn_mfma_f32_16x16x32_bf16(wf[k][n], xf[k][m], acc[m][n], 0, 0, 0); \
        __builtin_amdgcn_sched_barrier(0); } while (0)
    const u32 xo = (wr >> 1) * 16384 + (wr & 1) * 4096 + l16, wo = 32768 + wc * 4096 + l16;
    f32x4 acc[4][4];
#pragma unroll
    for (int m = 0; m < 4; ++m)
#pragma unroll
        for (int n = 0; n < 4; ++n) acc[m][n] = (f32x4){0.f, 0.f, 0.f, 0.f};
    u32 sb = 0;
    for (u32 u = 0; u < 32; ++u) {
        BARRIER();
        COMPUTE(sb);
        sb = sb == 2 * STAGE ? 0 : sb + STAGE;
    }
    if (wid == 0) STAMP(3);
    {
        const u32x4 dv = *(volatile LAS u32x4*)misc;
        const float pz = (dv.x | dv.y | dv.z | dv.w) ? __builtin_nanf("") : 0.f;
        const bool own = w_dist(tq) == 0;
        const int tp = 2 * tj + (wr >> 1);
        const u32 tile = ((rank ^ x_dist(tp)) * 4 + x_panel(tp)) * 4 + w_panel(tq);
        const auto rm = rsrc(myws, WS_NEED);
        const u32 to = own ? OFF_P + (tile * 2 + (rank & 1)) * TILEB + (wid & 3) * 8192 : OFF_T + ((tj * 16 + tq) * 8 + wid) * 8192;
        const auto rt = rsrc(myws + to, 8192);
        const u32 fo = own ? OFF_BF + tile * 128 + ((rank & 1) * 8 + (wid & 3) * 2) * 4 : OFF_RF + w_dist(tq) * 512 + ((RFL(*(volatile LAS u32*)(misc + 4)) & 15u) * 8 + wid) * 4;
        const u32 fv = own ? call : call << 10 | ((tj * 16 + tq) * 8 + wid);
#pragma unroll
        for (int m = 0; m < 4; ++m)
#pragma unroll
            for (int h = 0; h < 2; ++h) {
                u32x4 v;
                v.x = pack_bf16(acc[m][2 * h][0] + pz, acc[m][2 * h][1]);
                v.y = pack_bf16(acc[m][2 * h][2], acc[m][2 * h][3]);
                v.z = pack_bf16(acc[m][2 * h + 1][0], acc[m][2 * h + 1][1]);
                v.w = pack_bf16(acc[m][2 * h + 1][2], acc[m][2 * h + 1][3]);
                __builtin_amdgcn_raw_buffer_store_b128(v, rt, l16 + (m * 2 + h) * 1024, 0, SYS);
            }
        WAIT_VM(0);
        __builtin_amdgcn_raw_buffer_store_b32(fv, rm, lane < (own ? 2 : 1) ? lane * 4 : OOR, fo, SYS);
        if (wid == 0) STAMP(4);
    }
}

static void make_groups(PGroup* grp) {
    int g = 0, qid = 0;
    for (int i = 0; i < 16 * NPW; ++i) grp[i] = PGroup{};
    int nl = 0;
    auto add = [&](int ng, Group G) { G.qid = qid++; G.nw = 8 * ng; for (int i = 0; i < ng; ++i) { G.wi0 = 8 * i;
        if (G.dist) { grp[(g / 14) * 16 + g % 14] = pack(G); ++g; } else { grp[(nl >> 1) * 16 + 14 + (nl & 1)] = pack(G); ++nl; } } };
    for (int dd = 1; dd <= 8; ++dd) {
        const int d = dd & 7;
        u32 xs[4], ws[4]; int nx = 0, nwv = 0;
        for (int s = 0; s < 16; ++s) { if (x_dist(s) == d) xs[nx++] = s; if (w_dist(s) == d) ws[nwv++] = 16 + s; }
        Group G = {};
        G.dist = d; G.nlead = 128;
        if (d == 0) {
            G.ea = G.eb = 512;
            for (int h = 0; h < 2; ++h) { G.slots = xs[2 * h] | xs[2 * h + 1] << 8 | ws[2 * h] << 16 | ws[2 * h + 1] << 24; add(GL0, G); }
            continue;
        }
        if (d == 2) { G.slots = xs[0] | xs[1] << 8 | xs[2] << 16 | xs[3] << 24; G.ea = G.eb = 512; add(GL2, G); }
        else if (d & 1) { G.slots = xs[0] | xs[1] << 8 | ws[0] << 16 | ws[1] << 24; G.nlead = 128 - 4 * NE; G.lead = LEAD; G.ea = 256 + 2 * G.nlead; G.eb = G.ea + 256; add(GL, G); }
        else {
            G.two = 1; G.slots = ws[0] | ws[1] << 8; G.er = 16 * NE; G.ea = 256 + G.er; G.eb = G.ea + 256;
            G.rsrc_slots = d == 4 ? (17u | 19u << 8 | 21u << 16 | 23u << 24) : (25u | 27u << 8 | 29u << 16 | 30u << 24);
            G.rdst_slots = d == 4 ? (19u | 17u << 8 | 23u << 16 | 21u << 24) : (27u | 25u << 8 | 30u << 16 | 29u << 24);
            add(GL4, G);
        }
    }
    if (g > 56 || nl > 8) fprintf(stderr, "group table broken: %d %d\n", g, nl);
}

}

extern "C" void kernel_launch(int rank, int nranks, ncclComm_t comm,
                              hipStream_t stream,
                              void* const* d_in, const int* in_sizes, int n_in,
                              void* d_out, int out_size,
                              const int* in_dtypes, int out_dtype,
                              void* const* d_out_all,
                              void* const* d_ws_all, size_t ws_bytes) {
    if (nranks != NR || (unsigned)rank >= (unsigned)NR || ws_bytes < WS_NEED || n_in < 2 ||
        in_sizes[0] != 512 * 4096 || in_sizes[1] != 4096 * 512 || out_size != 4096 * 512) {
        fprintf(stderr, "unexpected problem shape\n");
        return;
    }
    Params p;
    p.x = (const char*)d_in[0];
    p.w = (const char*)d_in[1];
    p.out = (char*)d_out;
    for (int i = 0; i < NR; ++i) p.ws[i] = (char*)d_ws_all[i];
    make_groups(p.grp);
    p.rank = rank;
    hipLaunchKernelGGL(ag_gemm_kernel, dim3(256), dim3(768), 0, stream, p);
    HIP_CHECK(hipGetLastError());
}
```

```cpp
#include <hip/hip_runtime.h>
#include <rccl/rccl.h>
#include <cstdio>
#include <cstdlib>

#define HIP_CHECK(expr) do { hipError_t _e = (expr); if (_e != hipSuccess) { \
    fprintf(stderr, "HIP error %s at %s:%d\n", hipGetErrorString(_e), __FILE__, __LINE__); \
    return; } } while (0)

namespace {

typedef unsigned u32;
typedef unsigned long long u64;
typedef u32 u32x4 __attribute__((ext_vector_type(4)));
typedef u32 u32x2 __attribute__((ext_vector_type(2)));
typedef float f32x4 __attribute__((ext_vector_type(4)));
typedef __bf16 bf16x8 __attribute__((ext_vector_type(8)));
typedef short s16x4 __attribute__((ext_vector_type(4)));
typedef __bf16 bf16x2 __attribute__((ext_vector_type(2)));
typedef float f32x2 __attribute__((ext_vector_type(2)));

constexpr int NR = 8;
constexpr u32 PANEL = 512u << 10;
constexpr u32 TILEB = 32u << 10;
constexpr u32 OFF_S = 0;
constexpr u32 OFF_P = 32u << 20;
constexpr u32 OFF_T = 40u << 20;
constexpr u32 OFF_AF = 48u << 20;
constexpr u32 OFF_BF = OFF_AF + 16384;
constexpr u32 OFF_RF = OFF_BF + 16384;
constexpr u32 OFF_CTR = OFF_RF + 4096;
constexpr u32 OFF_TMO = OFF_CTR + 16384;
constexpr u32 OFF_Q = OFF_TMO + 1024;
constexpr u32 OFF_LT = OFF_Q + 4096;
constexpr u32 OFF_DBG = OFF_LT + 1024;
constexpr u32 WS_NEED = OFF_DBG + (1024u << 10);
constexpr u64 DEADLINE = 100000000ull;
constexpr int SYS = 17;
constexpr u32 OOR = 0x80000000u;

#define NE 7
#define LEAD 64
#define NPW 4
#define UP 22
#define D2F 64
#define RING 2
#define RNUM 4
#define WL 32
#define GSLEEP 1
#define RSLEEP 2
#define DSLEEP 1
#define GAUX 17
#define WS 3
#define HW 128
#define GL0 4
#define RPP 2
#define GL 2
#define GL2 GL
#define GL4 GL
#define GLR 2

struct Group { u32 w[4]; };
struct Params {
    const char* x;
    const char* w;
    char* out;
    char* ws[NR];
    int rank; u32 pn;
    Group grp[16 * NPW];
    unsigned short tab[2][512];
};

__host__ __device__ __forceinline__ int x_dist(int p) { return (0x7520'3210u >> (4 * (p >> 1))) & 7; }
__host__ __device__ __forceinline__ int x_panel(int p) { return ((p >> 3) << 1) | (p & 1); }
__host__ __device__ __forceinline__ int w_dist(int q) { return (int)((0x6530'7610'7430'5410ull >> (4 * q)) & 7); }
__host__ __device__ __forceinline__ int w_panel(int q) { return q >> 2; }

__device__ __forceinline__ __amdgpu_buffer_rsrc_t rsrc(const void* p, u32 bytes) {
    return __builtin_amdgcn_make_buffer_rsrc(const_cast<void*>(p), 0, bytes, 0x00020000);
}
__device__ __forceinline__ u32 pack_bf16(float a, float b) {
    const bf16x2 v = __builtin_convertvector((f32x2){a, b}, bf16x2);
    return __builtin_bit_cast(u32, v);
}
__device__ __forceinline__ float gelu(float y) {
    const float t = y * (-2.302208198f - 0.1029432397f * y * y);
    return y * __builtin_amdgcn_rcpf(1.0f + __builtin_amdgcn_exp2f(t));
}
#define WAIT_VM(n) asm volatile("s_waitcnt vmcnt(" #n ")" ::: "memory")
#define CBAR() asm volatile("" ::: "memory")
#define RFL(x) ((u32)__builtin_amdgcn_readfirstlane(x))

#define STAMP(i) do {} while (0)

constexpr u32 STAGE = 49152;
constexpr u32 LDS_POLL = 3 * STAGE;
constexpr u32 LDS_MISC = LDS_POLL + 1024;
constexpr u32 LDS_BYTES = LDS_MISC + 64;
#define LAS __attribute__((address_space(3)))

__global__ void __launch_bounds__(768, 3) ag_gemm_kernel(const Params P) {
    __shared__ __attribute__((aligned(16))) char smem[LDS_BYTES];
    LAS char* const lds = (LAS char*)smem;
    const int wid = RFL(threadIdx.x >> 6);
    const int lane = threadIdx.x & 63;
    const int rank = P.rank;
    char* const myws = P.ws[rank];
    u32* const ctr = (u32*)(myws + OFF_CTR) + blockIdx.x * 16 + wid;
    const u32 ctrv = __hip_atomic_load(ctr, __ATOMIC_RELAXED, __HIP_MEMORY_SCOPE_AGENT);
    u32* const tmo = (u32*)(myws + OFF_TMO);
    const u32 l0 = lane ? OOR : 0u, l16 = lane * 16;

#define BARRIER() do { asm volatile("s_waitcnt lgkmcnt(0)" ::: "memory"); __builtin_amdgcn_s_barrier(); CBAR(); } while (0)
#define LDS_LD(p) RFL(*(volatile LAS u32*)(p))
#define LDS_ST(p, v) do { *(volatile LAS u32*)(p) = (v); } while (0)
    if (blockIdx.x < 128) {
        const int g = blockIdx.x, pair = wid & 3;
        bool dead = false;
        const Group GG = P.grp[pair * 16 + (g >> 3)];
        struct { u32 dist, qid, nw, wi0, ea, eb, slots, tab; } G = {GG.w[0] & 7u, (GG.w[0] >> 8) & 15u, (GG.w[0] >> 16) & 255u, GG.w[0] >> 24, GG.w[1] & 0xffffu, GG.w[1] >> 16, GG.w[2], GG.w[3]};
        const int dst = (rank ^ G.dist) & 7;
        LAS char* const ring = lds + pair * 16384;
        LAS u32* const rdesc = (LAS u32*)(lds + 65536 + pair * 128);
        const auto rq = rsrc(myws + OFF_Q + G.qid * 256, 256);
        enum { K_COPY = 0, K_W = 1, K_SIG = 2, K_END = 3 };
        if (wid >= 8) {
            if (lane < 2) rdesc[16 + lane] = 0;
            BARRIER();
            const u32 call = RFL(ctrv) + 1u;
            if (lane == 0) __hip_atomic_store(ctr, call, __ATOMIC_RELAXED, __HIP_MEMORY_SCOPE_AGENT);
            if (!G.nw) return;
            const u64 t0 = __builtin_amdgcn_s_memrealtime();
            const u32 nw = G.nw, up = P.pn & 255u, une = P.pn >> 8;
            const u32 rsl = G.dist == 4 ? (17u | 19u << 8 | 21u << 16 | 23u << 24) : (25u | 27u << 8 | 29u << 16 | 30u << 24);
            const u32 rdl = G.dist == 4 ? (19u | 17u << 8 | 23u << 16 | 21u << 24) : (27u | 25u << 8 | 30u << 16 | 29u << 24);
            const u32* const tab = (const u32*)P.tab[G.tab & 1];
            const u32 sbase = OFF_S + (call & 1u) * 32 * PANEL;
            const u32 dec = 2u * nw - (call - 1u) * G.eb;
            const u32 urev = (dst & 4) ? 31u : 0u;
            const char* const xs = P.x + (dst & 1) * 4096;
            const char* const wsrc = P.w + (size_t)(dst & 1) * (2u << 20);
            const u32 xbytes = (4u << 20) - (dst & 1) * 4096;
            const auto rm = rsrc(myws, WS_NEED);
            const u32 vx = (lane & 15) * 8192 + (lane >> 4) * 16, vw = (lane >> 3) * 1024 + (lane & 7) * 16;
            const u32 dvr = OFF_AF + ((rsl >> (8 * (lane >> 4))) & 31u) * 512 + (lane & 15) * 16, dvb = OFF_RF + G.dist * 512 + (lane & 31) * 16;
            u32x4 dv = {0u, 0u, 0u, 0u};
            u32 cur = G.wi0 + (g & 7), c = cur + nw - dec, j = 0, pub = 0, used = 0, nst = 0;
            bool sig = G.eb == G.ea, gate = false;
            auto timed_out = [&]() { return __builtin_amdgcn_s_memrealtime() - t0 > DEADLINE; };
            auto publish = [&](u32 n) {
                if (n == pub) return;
                while (gate && !dead) {
                    const u32 v = __builtin_amdgcn_raw_buffer_load_b32(rq, 0, 128, 16);
                    CBAR();
                    if (RFL(v) >= call * nw) break;
                    if (timed_out()) dead = true;
                    __builtin_amdgcn_s_sleep(1);
                }
                if (gate) STAMP(pair * 10 + 7);
                gate = false;
                pub = n; LDS_ST(rdesc + 16, n);
            };
            auto slot_wait = [&]() {
                if (used + RING > j) return;
                used = LDS_LD(rdesc + 17);
                if (used + RING > j) return;
                WAIT_VM(0); publish(j);
                while (used + RING <= j) { __builtin_amdgcn_s_sleep(1); used = LDS_LD(rdesc + 17); }
            };
            auto word = [&](u32 w) -> u32 {
                const u32 l = w >> 2;
                const u32 v0 = __builtin_amdgcn_readlane(dv.x, l), v1 = __builtin_amdgcn_readlane(dv.y, l), v2 = __builtin_amdgcn_readlane(dv.z, l), v3 = __builtin_amdgcn_readlane(dv.w, l);
                return (w & 2) ? ((w & 1) ? v3 : v2) : ((w & 1) ? v1 : v0);
            };
            auto dep_wait = [&](u32 w, u32 sh, u32 dvo) {
                if ((word(w) >> sh) >= call) return;
                WAIT_VM(0); publish(j);
                while (!dead) {
                    dv = __builtin_amdgcn_raw_buffer_load_b128(rm, dvo, 0, SYS);
                    u32 t = RFL(__hip_atomic_load(tmo, __ATOMIC_RELAXED, __HIP_MEMORY_SCOPE_AGENT));
                    asm volatile("" : "+s"(t));
                    if ((word(w) >> sh) >= call) break;
                    if (t == call || timed_out()) dead = true;
                    __builtin_amdgcn_s_sleep(DSLEEP);
                }
                CBAR();
            };
            auto post = [&](u32 kind) { slot_wait(); LDS_ST(rdesc + (j & 3) * 4 + 3, kind); WAIT_VM(0); ++j; publish(j); };
            for (;;) {
                if (cur >= G.ea && !sig) { sig = true; dv = (u32x4){0u, 0u, 0u, 0u}; STAMP(pair * 10 + 1); }
                if (cur >= G.eb) break;
                u32 kind = K_COPY, so, s1 = 1024, s2 = 2048, dof, d2 = 2048, fo, vo = l16, src = 2;
                if (cur >= G.ea) {
                    const u32 r = cur - G.ea, hf = r & 1;
                    dep_wait((r >> 1) & 127u, 10, dvb);
                    const u32 id = word((r >> 1) & 127u) & 1023u, tj = id >> 7, tq = (id >> 3) & 15u, wv = id & 7u;
                    const u32 tp = 2 * tj + (wv >> 2), tile = ((rank ^ x_dist(tp)) * 4 + x_panel(tp)) * 4 + w_panel(tq);
                    so = OFF_T + id * 8192 + hf * 4096;
                    dof = OFF_P + (tile * 2 + (rank & 1)) * TILEB + (wv & 3) * 8192 + hf * 4096;
                    fo = OFF_BF + tile * 128 + ((rank & 1) * 8 + (wv & 3) * 2 + hf) * 4;
                } else {
                    const u32 df = G.tab >> 8;
                    const u32 e = (G.tab & 3) ? (tab[cur >> 1] >> ((cur & 1) * 16)) & 1023u
                                : cur < 2 * df ? (cur >> 1) | (cur & 1) << 7 : cur >= 512 - 2 * df ? ((cur >> 1) - 128) | (2 + (cur & 1)) << 7
                                : (((cur - 2 * df) >> 2) + ((cur - 2 * df) & 2 ? 0u : df)) | ((cur - 2 * df) & 3) << 7;
                    const u32 i = e & 127u, k = (e >> 7) & 3u;
                    if (e >> 9) {
                    const u32 sd = (rdl >> (8 * k)) & 31u;
                    dep_wait(k * 64 + i, 0, dvr);
                    so = sbase + ((rsl >> (8 * k)) & 31u) * PANEL + i * 4096;
                    dof = sbase + sd * PANEL + (i + 4 * up) * 4096;
                    fo = OFF_AF + (sd * 128 + i + 4 * up) * 4;
                    } else {
                    const u32 v = i >> 2, slot = (G.slots >> (8 * k)) & 31u, u = (v < up ? v : v < up + une ? 31 + up - v : v - une) ^ urev, h = i & 3;
                    fo = OFF_AF + (slot * 128 + i) * 4;
                    dof = sbase + slot * PANEL + (i >> 2) * 16384;
                    if (slot >= 16) { kind = K_W; src = 1; vo = vw; s1 = 8192; s2 = 16384; so = (u * 64 + (h >> 1) * 32) * 1024 + w_panel(slot & 15) * 256 + (h & 1) * 128; dof += h * 4096; }
                    else { src = 0; vo = vx; s1 = 131072; s2 = 64; d2 = 8192; so = (x_panel(slot) * 128 + h * 32) * 8192 + u * 128; dof += h * 2048; }
                    }
                }
                slot_wait();
                const auto rs = rsrc(src == 2 ? (const char*)myws : src ? wsrc : xs, src == 2 ? WS_NEED : src ? (2u << 20) : xbytes);
                LAS char* const sl = ring + (j & 3) * 4096;
                __builtin_amdgcn_raw_ptr_buffer_load_lds(rs, sl, 16, vo, so, 0, SYS);
                __builtin_amdgcn_raw_ptr_buffer_load_lds(rs, sl + 1024, 16, vo, so + s1, 0, SYS);
                __builtin_amdgcn_raw_ptr_buffer_load_lds(rs, sl + 2048, 16, vo, so + s2, 0, SYS);
                __builtin_amdgcn_raw_ptr_buffer_load_lds(rs, sl + 3072, 16, vo, so + s1 + s2, 0, SYS);
                *(volatile LAS u32x4*)(rdesc + (j & 3) * 4) = (u32x4){dof, d2, fo, kind};
                WAIT_VM(4);
                publish(j);
                cur = RFL(c) + dec;
                c = __builtin_amdgcn_raw_ptr_buffer_atomic_add_i32(1, rq, l0, 0, 16);
                CBAR();
                ++j; ++nst;
            }
            post(K_END);
            if (dead && lane == 0) __hip_atomic_store(tmo, call, __ATOMIC_RELAXED, __HIP_MEMORY_SCOPE_AGENT);
            return;
        }
        BARRIER();
        const u32 call = RFL(ctrv) + 1u;
        if (lane == 0) __hip_atomic_store(ctr, call, __ATOMIC_RELAXED, __HIP_MEMORY_SCOPE_AGENT);
        if (wid < 4 && G.nw) {
            STAMP(pair * 10);
            const auto rd = rsrc(P.ws[dst], WS_NEED);
            const u32 tro = ((lane >> 4) * 8 + ((lane >> 2) & 3)) * 128 + (lane & 3) * 32;
            u32 k = 0, full = 0, pend = OOR;
#define FLAG() do { __builtin_amdgcn_raw_buffer_store_b32(call, rd, pend, 0, SYS); CBAR(); pend = OOR; } while (0)
            for (;;) {
                if (full <= k) {
                    full = LDS_LD(rdesc + 16);
                    if (full <= k) {
                        WAIT_VM(0); FLAG();
                        while (full <= k) { __builtin_amdgcn_s_sleep(1); full = LDS_LD(rdesc + 16); }
                    }
                    CBAR();
                }
                const u32x4 dd = *(volatile LAS u32x4*)(rdesc + (k & 3) * 4);
                const u32 dof = RFL(dd.x), d2 = RFL(dd.y), fo = RFL(dd.z), kind = RFL(dd.w);
                if (kind >= K_SIG) {
                    WAIT_VM(0); FLAG();
                    if (kind == K_END) break;
                    (void)__builtin_amdgcn_raw_ptr_buffer_atomic_add_i32(1, rq, l0, 128, 16);
                    STAMP(pair * 10 + 1);
                    ++k; LDS_ST(rdesc + 17, k);
                    continue;
                }
                const LAS char* const sl = ring + (k & 3) * 4096;
                u32x4 f[4];
                if (kind == K_W) {
#pragma unroll
                    for (int n = 0; n < 4; ++n) {
                        const u32x2 lo = __builtin_bit_cast(u32x2, __builtin_amdgcn_ds_read_tr16_b64_v4i16((LAS s16x4*)(sl + tro + n * 8)));
                        const u32x2 hi = __builtin_bit_cast(u32x2, __builtin_amdgcn_ds_read_tr16_b64_v4i16((LAS s16x4*)(sl + tro + 512 + n * 8)));
                        f[n] = (u32x4){lo.x, lo.y, hi.x, hi.y};
                    }
                } else {
#pragma unroll
                    for (int n = 0; n < 4; ++n) f[n] = *(const LAS u32x4*)(sl + l16 + n * 1024);
                }
                asm volatile("s_waitcnt lgkmcnt(0)" : "+v"(f[0]), "+v"(f[1]), "+v"(f[2]), "+v"(f[3]) :: "memory");
                ++k; LDS_ST(rdesc + 17, k);
#define WSTEP(i) do { asm volatile("s_waitcnt vmcnt(%0)" :: "i"(WS - 1) : "memory"); if (i == WS - 1) FLAG(); } while (0)
                const u32 vo0 = l16 + dof, vo1 = vo0 + d2;
                WSTEP(0); __builtin_amdgcn_raw_buffer_store_b128(f[0], rd, vo0, 0, SYS);
                WSTEP(1); __builtin_amdgcn_raw_buffer_store_b128(f[1], rd, vo0 + 1024, 0, SYS);
                WSTEP(2); __builtin_amdgcn_raw_buffer_store_b128(f[2], rd, vo1, 0, SYS);
                WSTEP(3); __builtin_amdgcn_raw_buffer_store_b128(f[3], rd, vo1 + 1024, 0, SYS);
                if (WS > 4) { WAIT_VM(4); FLAG(); }
                pend = l0 | fo;
            }
            STAMP(pair * 10 + 4);
        }
        const int q = wid >> 1, mp = wid & 1;
        const u32* fb = lane < 2 ? (const u32*)(myws + OFF_BF + g * 128) + lane * 8 + wid : tmo;
        const u64 t0 = __builtin_amdgcn_s_memrealtime();
        while (!dead) {
            const u32 v = __hip_atomic_load(fb, __ATOMIC_RELAXED, __HIP_MEMORY_SCOPE_SYSTEM);
            if (__ballot(v == call) >> 2) { dead = true; break; }
            if ((__ballot(v >= call) & 3) == 3) break;
            if (__builtin_amdgcn_s_memrealtime() - t0 > DEADLINE) { dead = true; break; }
            __builtin_amdgcn_s_sleep(RSLEEP);
        }
        CBAR();
        if (dead && lane == 0) __hip_atomic_store(tmo, call, __ATOMIC_RELAXED, __HIP_MEMORY_SCOPE_AGENT);
        if (wid == 7) STAMP(40);
        const auto rp_ = rsrc(myws + OFF_P + (size_t)g * 2 * TILEB + q * 8192, TILEB + 8192);
        const auto ro = rsrc(P.out + (size_t)((g >> 2) * 128 + (q >> 1) * 64 + mp * 32) * 1024 + (g & 3) * 256 + (q & 1) * 128, 31 * 1024 + 128);
        const u32 po = (lane & 1) * 1024 + ((lane >> 1) & 3) * 256 + (lane >> 3) * 16 + mp * 4096;
        const u32 oo = (lane >> 3) * 1024 + (lane & 7) * 16;
        u32x4 p0[4], p1[4];
#pragma unroll
        for (int i = 0; i < 4; ++i) {
            p0[i] = __builtin_amdgcn_raw_buffer_load_b128(rp_, po + (i >> 1) * 2048 + (i & 1) * 128, 0, SYS);
            p1[i] = __builtin_amdgcn_raw_buffer_load_b128(rp_, po + (i >> 1) * 2048 + (i & 1) * 128, TILEB, SYS);
        }
        const float pz = dead ? __builtin_nanf("") : 0.f;
#pragma unroll
        for (int i = 0; i < 4; ++i) {
            u32x4 o;
#pragma unroll
            for (int j = 0; j < 4; ++j) {
                const float lo = __builtin_bit_cast(float, p0[i][j] << 16) + __builtin_bit_cast(float, p1[i][j] << 16) + pz;
                const float hi = __builtin_bit_cast(float, p0[i][j] & 0xffff0000u) + __builtin_bit_cast(float, p1[i][j] & 0xffff0000u) + pz;
                o[j] = pack_bf16(gelu(lo), gelu(hi));
            }
            __builtin_amdgcn_raw_buffer_store_b128(o, ro, oo + i * 8192, 0, 16);
        }
        STAMP(41 + wid);
        return;
    }

    const u32 call = RFL(ctrv) + 1u;
    if (lane == 0) __hip_atomic_store(ctr, call, __ATOMIC_RELAXED, __HIP_MEMORY_SCOPE_AGENT);
    const u32 par = call & 1u;
    const int gg = blockIdx.x - 128;
    const int tj = ((gg & 7) >> 1) * 2 + (gg >> 6), tq = (gg & 1) * 8 + ((gg >> 3) & 7);
    LAS u32* const misc = (LAS u32*)(lds + LDS_MISC);
    if (wid >= 8) {
        const int l = wid - 8;
        bool dead = false;
        const u64 t0 = __builtin_amdgcn_s_memrealtime();
        const auto rx = rsrc(myws + OFF_S + (size_t)(par * 32 + 2 * tj) * PANEL, 2 * PANEL);
        const auto rw = rsrc(myws + OFF_S + (size_t)(par * 32 + 16 + tq) * PANEL, PANEL);
        const auto raf = rsrc(myws + OFF_AF, OFF_TMO + 4 - OFF_AF);
        const u32 gx = l * 2048 + l16, gw = l * 4096 + l16;
        const u32 fsl = lane >> 4;
        const u32 fo = fsl == 3 ? OFF_TMO - OFF_AF : (fsl == 2 ? 16 + tq : 2 * tj + fsl) * 512 + (lane & 15) * 16 + l * 4;
        const u32 fstep = fsl == 3 ? 0u : 16u;
        LAS u32* const pollbuf = (LAS u32*)(lds + LDS_POLL + l * 256);
        u32 avail = 0, pbase = 0;
#define POLL() do { pbase = avail; __builtin_amdgcn_raw_ptr_buffer_load_lds(raf, pollbuf, 4, fo + min(avail, 31u) * fstep, 0, 0, SYS); } while (0)
#define POLL_READ() do { const u32 _v = *(volatile LAS u32*)(pollbuf + lane); \
        const u64 _m = ~__ballot(_v >= call || pbase + (lane & 15) >= 32); \
        const u32 _a = pbase + __builtin_ctz((((u32)_m | (u32)(_m >> 16) | (u32)(_m >> 32)) & 0xffffu) | 0x10000u); \
        avail = max(avail, _a); if (__ballot(_v == call) >> 48) { dead = true; avail = 64; } } while (0)
#define SPIN(n) do { while (avail < (u32)(n)) { POLL(); WAIT_VM(0); POLL_READ(); \
        if (avail < (u32)(n)) { if (__builtin_amdgcn_s_memrealtime() - t0 > DEADLINE) { dead = true; avail = 64; } else __builtin_amdgcn_s_sleep(GSLEEP); } } CBAR(); } while (0)
#define DMA1(r, d, v, o) __builtin_amdgcn_raw_ptr_buffer_load_lds(r, d, 16, v, o, 0, GAUX)
#define DMA(u, sb) do { LAS char* _x = lds + (sb) + l * 2048; LAS char* _w = lds + (sb) + 32768 + l * 4096; const u32 _o = (u) * 16384; \
        DMA1(rx, _x, gx, _o); DMA1(rx, _x + 1024, gx + 1024, _o); DMA1(rx, _x + 16384, gx, _o + PANEL); DMA1(rx, _x + 17408, gx + 1024, _o + PANEL); \
        DMA1(rw, _w, gw, _o); DMA1(rw, _w + 1024, gw + 1024, _o); DMA1(rw, _w + 2048, gw + 2048, _o); DMA1(rw, _w + 3072, gw + 3072, _o); \
        DMA1(rx, _x + 8192, gx, _o + 8192); DMA1(rx, _x + 9216, gx + 1024, _o + 8192); DMA1(rx, _x + 24576, gx, _o + PANEL + 8192); DMA1(rx, _x + 25600, gx + 1024, _o + PANEL + 8192); CBAR(); } while (0)
        if (lane == 0) misc[l] = 0;
        SPIN(1); DMA(0, 0);
        SPIN(2); POLL(); DMA(1, STAGE);
        u32 sn = 2 * STAGE;
        for (u32 u = 0; u < 30; ++u) {
            WAIT_VM(12); BARRIER();
            POLL_READ();
            SPIN(u + 3); POLL(); DMA(u + 2, sn);
            sn = sn == 2 * STAGE ? 0 : sn + STAGE;
        }
        if (l == 0) STAMP(2);
        if (dead && lane == 0) { misc[l] = 1; __hip_atomic_store(tmo, call, __ATOMIC_RELAXED, __HIP_MEMORY_SCOPE_AGENT); }
        WAIT_VM(12); BARRIER();
        if (l == 0 && lane == 0)
            misc[4] = __hip_atomic_fetch_add((u32*)(myws + OFF_LT + w_dist(tq) * 128), 1u, __ATOMIC_RELAXED, __HIP_MEMORY_SCOPE_AGENT);
        WAIT_VM(0); BARRIER();
        return;
    }
    const int wr = wid >> 1, wc = wid & 1;
#define COMPUTE(sb) do { const LAS char* _x = lds + (sb) + xo; const LAS char* _w = lds + (sb) + wo; bf16x8 xf[2][4], wf[2][4]; \
        _Pragma("unroll") for (int k = 0; k < 2; ++k) _Pragma("unroll") for (int i = 0; i < 4; ++i) { \
            wf[k][i] = *(const LAS bf16x8*)(_w + k * 8192 + i * 1024); xf[k][i] = *(const LAS bf16x8*)(_x + k * 8192 + i * 1024); } \
        __builtin_amdgcn_sched_barrier(0); \
        _Pragma("unroll") for (int k = 0; k < 2; ++k) _Pragma("unroll") for (int m = 0; m < 4; ++m) _Pragma("unroll") for (int n = 0; n < 4; ++n) \
            acc[m][n] = __builtin_amdgcn_mfma_f32_16x16x32_bf16(wf[k][n], xf[k][m], acc[m][n], 0, 0, 0); \
        __builtin_amdgcn_sched_barrier(0); } while (0)
    const u32 xo = (wr >> 1) * 16384 + (wr & 1) * 4096 + l16, wo = 32768 + wc * 4096 + l16;
    f32x4 acc[4][4];
#pragma unroll
    for (int m = 0; m < 4; ++m)
#pragma unroll
        for (int n = 0; n < 4; ++n) acc[m][n] = (f32x4){0.f, 0.f, 0.f, 0.f};
    u32 sb = 0;
    for (u32 u = 0; u < 32; ++u) {
        BARRIER();
        COMPUTE(sb);
        sb = sb == 2 * STAGE ? 0 : sb + STAGE;
    }
    if (wid == 0) STAMP(3);
    {
        const u32x4 dv = *(volatile LAS u32x4*)misc;
        const float pz = (dv.x | dv.y | dv.z | dv.w) ? __builtin_nanf("") : 0.f;
        const bool own = w_dist(tq) == 0;
        const int tp = 2 * tj + (wr >> 1);
        const u32 tile = ((rank ^ x_dist(tp)) * 4 + x_panel(tp)) * 4 + w_panel(tq);
        const auto rm = rsrc(myws, WS_NEED);
        const u32 to = own ? OFF_P + (tile * 2 + (rank & 1)) * TILEB + (wid & 3) * 8192 : OFF_T + ((tj * 16 + tq) * 8 + wid) * 8192;
        const auto rt = rsrc(myws + to, 8192);
        const u32 fo = own ? OFF_BF + tile * 128 + ((rank & 1) * 8 + (wid & 3) * 2) * 4 : OFF_RF + w_dist(tq) * 512 + ((RFL(*(volatile LAS u32*)(misc + 4)) & 15u) * 8 + wid) * 4;
        const u32 fv = own ? call : call << 10 | ((tj * 16 + tq) * 8 + wid);
#pragma unroll
        for (int m = 0; m < 4; ++m)
#pragma unroll
            for (int h = 0; h < 2; ++h) {
                u32x4 v;
                v.x = pack_bf16(acc[m][2 * h][0] + pz, acc[m][2 * h][1]);
                v.y = pack_bf16(acc[m][2 * h][2], acc[m][2 * h][3]);
                v.z = pack_bf16(acc[m][2 * h + 1][0], acc[m][2 * h + 1][1]);
                v.w = pack_bf16(acc[m][2 * h + 1][2], acc[m][2 * h + 1][3]);
                __builtin_amdgcn_raw_buffer_store_b128(v, rt, l16 + (m * 2 + h) * 1024, 0, SYS);
            }
        WAIT_VM(0);
        __builtin_amdgcn_raw_buffer_store_b32(fv, rm, lane < (own ? 2 : 1) ? lane * 4 : OOR, fo, SYS);
        if (wid == 0) STAMP(4);
    }
}

static int knob(const char*, int d) { return d; }
static void make_schedule(Params& p) {
    const int ne = knob("NE", NE), up = knob("UP", UP), lead = knob("LEAD", LEAD), hw = knob("HW", HW), rnum = knob("RNUM", RNUM), wl = knob("WL", WL);
    const int gl = knob("GL", GL), gl2 = knob("GL2", GL2), gl4 = knob("GL4", GL4);
    p.pn = up | ne << 8;
    int n1 = 0;
    for (int t = -lead; t < 128; ++t) {
        const int wi = t + lead;
        if (t >= 0) { p.tab[1][n1++] = t; p.tab[1][n1++] = t | 1 << 7; }
        if (wi < 128 && (wi < 4 * up || wi >= 4 * (up + ne))) { p.tab[1][n1++] = wi | 2 << 7; p.tab[1][n1++] = wi | 3 << 7; }
    }
    int n0 = 0, nd = 0, r = 0;
    auto direct = [&](int e) { p.tab[0][n0++] = e; if (++nd >= hw) for (int k = 0; k < rnum && r < 16 * ne; ++k, ++r) p.tab[0][n0++] = (r >> 2) | (r & 3) << 7 | 1 << 9; };
    for (int t = -wl; t < 128; ++t) { if (t + wl < 128) direct(t + wl); if (t >= 0) direct(t | 1 << 7); }
    for (; r < 16 * ne; ++r) p.tab[0][n0++] = (r >> 2) | (r & 3) << 7 | 1 << 9;
    for (int i = n0; i < 512; ++i) p.tab[0][i] = 0;
    for (int i = n1; i < 512; ++i) p.tab[1][i] = 0;
    for (int i = 0; i < 16 * NPW; ++i) p.grp[i] = Group{};
    int g = 0, nl = 0, qid = 0;
    auto add = [&](int ng, u32 d, u32 ea, u32 eb, u32 slots, u32 tab) {
        for (int i = 0; i < ng; ++i) {
            const Group G = {{d | qid << 8 | (8 * ng) << 16 | (8 * i) << 24, ea | eb << 16, slots, tab}};
            if (d) { p.grp[(g / 14) * 16 + g % 14] = G; ++g; } else { p.grp[(nl >> 1) * 16 + 14 + (nl & 1)] = G; ++nl; }
        }
        ++qid;
    };
    for (int dd = 1; dd <= 8; ++dd) {
        const int d = dd & 7;
        u32 xs[4], ws[4]; int nx = 0, nwv = 0;
        for (int s = 0; s < 16; ++s) { if (x_dist(s) == d) xs[nx++] = s; if (w_dist(s) == d) ws[nwv++] = 16 + s; }
        if (d == 0) for (int h = 0; h < 2; ++h) add(4, 0, 512, 512, xs[2 * h] | xs[2 * h + 1] << 8 | ws[2 * h] << 16 | ws[2 * h + 1] << 24, 0);
        else if (d == 2) add(gl2, d, 512, 512, xs[0] | xs[1] << 8 | xs[2] << 16 | xs[3] << 24, knob("D2F", D2F) << 8);
        else if (d & 1) add(gl, d, n1, n1 + 256, xs[0] | xs[1] << 8 | ws[0] << 16 | ws[1] << 24, 1);
        else add(gl4, d, n0, n0 + 256, ws[0] | ws[1] << 8, 2);
    }
}

}

extern "C" void kernel_launch(int rank, int nranks, ncclComm_t comm,
                              hipStream_t stream,
                              void* const* d_in, const int* in_sizes, int n_in,
                              void* d_out, int out_size,
                              const int* in_dtypes, int out_dtype,
                              void* const* d_out_all,
                              void* const* d_ws_all, size_t ws_bytes) {
    if (nranks != NR || (unsigned)rank >= (unsigned)NR || ws_bytes < WS_NEED || n_in < 2 ||
        in_sizes[0] != 512 * 4096 || in_sizes[1] != 4096 * 512 || out_size != 4096 * 512) {
        fprintf(stderr, "unexpected problem shape\n");
        return;
    }
    Params p;
    p.x = (const char*)d_in[0];
    p.w = (const char*)d_in[1];
    p.out = (char*)d_out;
    for (int i = 0; i < NR; ++i) p.ws[i] = (char*)d_ws_all[i];
    make_schedule(p);
    p.rank = rank;
    hipLaunchKernelGGL(ag_gemm_kernel, dim3(256), dim3(768), 0, stream, p);
    HIP_CHECK(hipGetLastError());
}
```

```cpp
#include <hip/hip_runtime.h>
#include <rccl/rccl.h>
#include <cstdio>
#include <cstdlib>

#define HIP_CHECK(expr) do { hipError_t _e = (expr); if (_e != hipSuccess) { \
    fprintf(stderr, "HIP error %s at %s:%d\n", hipGetErrorString(_e), __FILE__, __LINE__); \
    return; } } while (0)

namespace {

typedef unsigned u32;
typedef unsigned long long u64;
typedef u32 u32x4 __attribute__((ext_vector_type(4)));
typedef u32 u32x2 __attribute__((ext_vector_type(2)));
typedef float f32x4 __attribute__((ext_vector_type(4)));
typedef float f32x2 __attribute__((ext_vector_type(2)));
typedef short s16x4 __attribute__((ext_vector_type(4)));
typedef __bf16 bf16x8 __attribute__((ext_vector_type(8)));
typedef __bf16 bf16x2 __attribute__((ext_vector_type(2)));

constexpr int NR = 8;
constexpr u32 PANEL = 512u << 10;
constexpr u32 OFF_S = 0;
constexpr u32 OFF_T = 32u << 20;
constexpr u32 OFF_P = 40u << 20;
constexpr u32 OFF_AF = 44u << 20;
constexpr u32 OFF_RF = OFF_AF + 16384;
constexpr u32 OFF_ST = OFF_RF + 4096;
constexpr u32 OFF_CTR = OFF_ST + 4096;
constexpr u32 OFF_TMO = OFF_CTR + 16384;
constexpr u32 OFF_Q = OFF_TMO + 1024;
constexpr u32 OFF_LT = OFF_Q + 4096;
constexpr u32 OFF_DBG = OFF_LT + 1024;
constexpr u32 WS_NEED = OFF_DBG + (1024u << 10);
constexpr u64 DEADLINE = 100000000ull;
constexpr int SYS = 17;
constexpr int AGT = 16;
constexpr u32 OOR = 0x80000000u;

#define NE 8
#define UP 22
#define LEAD 64
#define HW 160
#define RNUM 4
#define WL 24
#define D2F 64
#define WS 3
#define WP 3
#define RING 2
#define GX 13
#define NPW 4

struct Group { u32 w[4]; };
struct Params {
    const char* x;
    const char* w;
    char* out;
    char* ws[NR];
    int rank; u32 pn;
    Group grp[16 * NPW];
    unsigned short tab[2][512];
};

__host__ __device__ __forceinline__ u32 x_dist(u32 p) { return (0x7520'3210u >> (4 * (p >> 1))) & 7u; }
__host__ __device__ __forceinline__ u32 x_panel(u32 p) { return ((p >> 3) << 1) | (p & 1u); }
__host__ __device__ __forceinline__ u32 w_dist(u32 q) { return (u32)(0x6530'7610'7430'5410ull >> (4 * q)) & 7u; }
__host__ __device__ __forceinline__ u32 w_panel(u32 q) { return q >> 2; }

__device__ __forceinline__ __amdgpu_buffer_rsrc_t rsrc(const void* p, u32 bytes) {
    return __builtin_amdgcn_make_buffer_rsrc(const_cast<void*>(p), 0, bytes, 0x00020000);
}
__device__ __forceinline__ u32 pack_bf16(float a, float b) {
    return __builtin_bit_cast(u32, __builtin_convertvector((f32x2){a, b}, bf16x2));
}
__device__ __forceinline__ float gelu(float y) {
    const float t = y * (-2.302208198f - 0.1029432397f * y * y);
    return y * __builtin_amdgcn_rcpf(1.0f + __builtin_amdgcn_exp2f(t));
}
#define WAIT_VM(n) asm volatile("s_waitcnt vmcnt(%0)" :: "i"(n) : "memory")
#define CBAR() asm volatile("" ::: "memory")
#define RFL(x) ((u32)__builtin_amdgcn_readfirstlane(x))
#define LAS __attribute__((address_space(3)))
#define BARRIER() do { asm volatile("s_waitcnt lgkmcnt(0)" ::: "memory"); __builtin_amdgcn_s_barrier(); CBAR(); } while (0)
#define LDS_LD(p) RFL(*(volatile LAS u32*)(p))
#define LDS_ST(p, v) do { *(volatile LAS u32*)(p) = (v); } while (0)
#define DMA16(r, d, v, o) __builtin_amdgcn_raw_ptr_buffer_load_lds(r, d, 16, v, o, 0, SYS)

#define STAMP(i) do {} while (0)

constexpr u32 STAGE = 49152;
constexpr u32 LDS_POLL = 3 * STAGE;
constexpr u32 LDS_MISC = LDS_POLL + 1024;
constexpr u32 LDS_BYTES = LDS_MISC + 64;
enum { K_COPY, K_W, K_RED, K_SIG, K_END };

__global__ void __launch_bounds__(768, 3) ag_gemm_kernel(const Params P) {
    __shared__ __attribute__((aligned(16))) char smem[LDS_BYTES];
    LAS char* const lds = (LAS char*)smem;
    const u32 wid = RFL(threadIdx.x >> 6);
    const u32 lane = threadIdx.x & 63;
    const u32 rank = P.rank & 7;
    char* const myws = P.ws[rank];
    if (blockIdx.x < 128 && (wid & 4)) return;
    u32* const ctr = (u32*)(myws + OFF_CTR) + blockIdx.x * 16 + wid;
    const u32 ctrv = __hip_atomic_load(ctr, __ATOMIC_RELAXED, __HIP_MEMORY_SCOPE_AGENT);
    u32* const tmo = (u32*)(myws + OFF_TMO);
    const u32 l0 = lane ? OOR : 0u, l16 = lane * 16;
    const auto rm = rsrc(myws, WS_NEED);

    if (blockIdx.x < 128) {
        const u32 g = blockIdx.x, pair = wid & 3;
        const Group G = P.grp[pair * 16 + (g >> 3)];
        const u32 dist = G.w[0] & 7u, nw = (G.w[0] >> 16) & 255u, ea = G.w[1] & 0xffffu, eb = G.w[1] >> 16;
        const u32 peer = rank ^ dist, qo = OFF_Q + ((G.w[0] >> 8) & 15u) * 256;
        const auto rd = rsrc(P.ws[peer], WS_NEED);
        LAS char* const ring = lds + pair * 32768;
        LAS u32* const rdesc = (LAS u32*)(lds + 131072 + pair * 128);
        bool dead = false;
        if (wid >= 8) {
            if (lane < 2) rdesc[16 + lane] = 0;
            const u32 up = P.pn & 255u, une = (P.pn >> 8) & 255u, df = G.w[3] >> 8;
            const u32 rsl = dist == 4 ? (17u | 19u << 8 | 21u << 16 | 23u << 24) : (25u | 27u << 8 | 29u << 16 | 30u << 24);
            const u32 rdl = dist == 4 ? (19u | 17u << 8 | 23u << 16 | 21u << 24) : (27u | 25u << 8 | 30u << 16 | 29u << 24);
            const u32* const tab = (const u32*)P.tab[G.w[3] & 1];
            const u32 urev = (peer & 4) ? 31u : 0u;
            const char* const xs = P.x + (peer & 1) * 4096;
            const char* const wsrc = P.w + (size_t)(peer & 1) * (2u << 20);
            const u32 xbytes = (4u << 20) - (peer & 1) * 4096;
            const u32 vx = (lane & 15) * 8192 + (lane >> 4) * 16, vw = (lane >> 3) * 1024 + (lane & 7) * 16;
            u32x4 dv = {0u, 0u, 0u, 0u};
            u32 cur = (G.w[0] >> 24) + (g & 7), c = 0, j = 0, pub = 0, used = 0, call = 0, dec = 0, sbase = 0;
            u64 t0 = 0;
            auto timed_out = [&]() { return __builtin_amdgcn_s_memrealtime() - t0 > DEADLINE; };
            auto publish = [&](u32 n) { if (n != pub) { pub = n; LDS_ST(rdesc + 16, n); } };
            auto slot_wait = [&]() {
                if (used + RING > j) return;
                used = LDS_LD(rdesc + 17);
                if (used + RING > j) return;
                WAIT_VM(0); publish(j);
                while (used + RING <= j) { __builtin_amdgcn_s_sleep(1); used = LDS_LD(rdesc + 17); }
            };
            auto word = [&](u32 w) -> u32 {
                const u32 l = w >> 2;
                const u32 v0 = __builtin_amdgcn_readlane(dv.x, l), v1 = __builtin_amdgcn_readlane(dv.y, l), v2 = __builtin_amdgcn_readlane(dv.z, l), v3 = __builtin_amdgcn_readlane(dv.w, l);
                return (w & 2) ? ((w & 1) ? v3 : v2) : ((w & 1) ? v1 : v0);
            };
            auto dep_wait = [&](u32 w, u32 sh, u32 want, u32 dvo) {
                if ((word(w) >> sh) >= want) return;
                WAIT_VM(0); publish(j);
                while (!dead) {
                    dv = __builtin_amdgcn_raw_buffer_load_b128(rm, dvo, 0, SYS);
                    u32 t = RFL(__hip_atomic_load(tmo, __ATOMIC_RELAXED, __HIP_MEMORY_SCOPE_AGENT));
                    asm volatile("" : "+s"(t));
                    if ((word(w) >> sh) >= want) break;
                    if (t == call || timed_out()) dead = true;
                    __builtin_amdgcn_s_sleep(1);
                }
                CBAR();
            };
            auto post = [&](u32 kind) { slot_wait(); LDS_ST(rdesc + (j & 3) * 4 + 3, kind); WAIT_VM(0); ++j; publish(j); };
            const u32 dvr = OFF_AF + ((rsl >> (8 * (lane >> 4))) & 31u) * 512 + (lane & 15) * 16;
            bool first = true;
            while (first || cur < ea) {
                const u32 e = (G.w[3] & 3) ? (tab[cur >> 1] >> ((cur & 1) * 16)) & 1023u
                            : cur < 2 * df ? (cur >> 1) | (cur & 1) << 7 : cur >= 512 - 2 * df ? ((cur >> 1) - 128) | (2 + (cur & 1)) << 7
                            : (((cur - 2 * df) >> 2) + ((cur - 2 * df) & 2 ? 0u : df)) | ((cur - 2 * df) & 3) << 7;
                const u32 i = e & 127u, k = (e >> 7) & 3u;
                u32 kind = K_COPY, so, s1 = 1024, s2 = 2048, pc, d2 = 2048, vo = l16, src = 2;
                if (e >> 9) {
                    dep_wait(k * 64 + i, 0, call, dvr);
                    so = sbase + ((rsl >> (8 * k)) & 31u) * PANEL + i * 4096;
                    pc = ((rdl >> (8 * k)) & 31u) * 128 + i + 4 * up;
                } else {
                    const u32 v = i >> 2, slot = (G.w[2] >> (8 * k)) & 31u, u = (v < up ? v : v < up + une ? 31 + up - v : v - une) ^ urev, h = i & 3;
                    pc = slot * 128 + i;
                    if (slot >= 16) { kind = K_W; src = 1; vo = vw; s1 = 8192; s2 = 16384; so = (u * 64 + (h >> 1) * 32) * 1024 + w_panel(slot & 15) * 256 + (h & 1) * 128; }
                    else { src = 0; vo = vx; s1 = 131072; s2 = 64; d2 = 8192; so = (x_panel(slot) * 128 + h * 32) * 8192 + u * 128; }
                }
                slot_wait();
                const auto rs = rsrc(src == 2 ? (const char*)myws : src ? wsrc : xs, src == 2 ? WS_NEED : src ? (2u << 20) : xbytes);
                LAS char* const sl = ring + (j & 3) * 8192;
                if (!nw) so = OOR;
                DMA16(rs, sl, vo, so); DMA16(rs, sl + 1024, vo, so + s1); DMA16(rs, sl + 2048, vo, so + s2); DMA16(rs, sl + 3072, vo, so + s1 + s2);
                *(volatile LAS u32x4*)(rdesc + (j & 3) * 4) = (u32x4){pc, d2, 0u, kind};
                if (first) {
                    BARRIER();
                    asm volatile("s_waitcnt vmcnt(4)\n\tv_readfirstlane_b32 %0, %1" : "=s"(call) : "v"(ctrv) : "memory");
                    ++call;
                    if (lane == 0) __hip_atomic_store(ctr, call, __ATOMIC_RELAXED, __HIP_MEMORY_SCOPE_AGENT);
                    if (!nw) return;
                    t0 = __builtin_amdgcn_s_memrealtime();
                    sbase = OFF_S + (call & 1u) * 32 * PANEL;
                    dec = 2u * nw - (call - 1u) * eb;
                    c = cur + nw - dec;
                    first = false;
                } else {
                    WAIT_VM(4);
                    publish(j);
                }
                cur = RFL(c) + dec;
                c = __builtin_amdgcn_raw_ptr_buffer_atomic_add_i32(1, rm, l0, qo, AGT);
                CBAR();
                ++j;
            }
            if (eb > ea) {
                post(K_SIG);
                dv = (u32x4){0u, 0u, 0u, 0u};
                const u32 dvb = OFF_RF + ((G.w[3] >> 4) & 7u) * 512 + (lane & 31) * 16;
                const u32 po = (lane & 1) * 1024 + ((lane >> 1) & 3) * 256 + (lane >> 3) * 16;
                u32 so = 0, unit = 0, tk = 0;
                auto resolve = [&]() {
                    if (cur < eb) {
                        const u32 r = cur - ea;
                        dep_wait((r >> 1) & 127u, 10, call, dvb);
                        const u32 id = word((r >> 1) & 127u) & 1023u, tq = (id >> 3) & 15u, wv = id & 7u, tp = 2 * (id >> 7) + (wv >> 2);
                        so = OFF_T + id * 8192 + (r & 1) * 4096;
                        unit = (((peer ^ x_dist(tp)) * 4 + x_panel(tp)) * 4 + w_panel(tq)) * 8 + (wv & 3) * 2 + (r & 1);
                    }
                    tk = __builtin_amdgcn_raw_ptr_buffer_atomic_add_i32(1, rm, cur < eb && !dead ? l0 : OOR, OFF_ST + unit * 4, AGT);
                    CBAR();
                };
                resolve();
                while (dist && !dead) {
                    const u32 v = RFL(__builtin_amdgcn_raw_buffer_load_b32(rm, 0, qo + 128, SYS));
                    CBAR();
                    if (v + (P.pn >> 16) >= call * nw) break;
                    if (timed_out()) dead = true;
                    __builtin_amdgcn_s_sleep(1);
                }
                STAMP(pair * 10 + 7);
                while (cur < eb) {
                    slot_wait();
                    LAS char* const sl = ring + (j & 3) * 8192;
                    WAIT_VM(WP + 5); DMA16(rd, sl, po, so);
                    WAIT_VM(WP + 5); DMA16(rd, sl + 1024, po + 128, so);
                    WAIT_VM(WP + 5); DMA16(rd, sl + 2048, po + 2048, so);
                    WAIT_VM(WP + 5); DMA16(rd, sl + 3072, po + 2176, so);
                    CBAR();
                    const u32 st = dead ? 2u : RFL(tk) & 3u;
                    WAIT_VM(4);
                    publish(j);
                    const u32 oo = st == 3 ? OFF_P + unit * 4096 : OOR;
                    DMA16(rm, sl + 4096, l16, oo); DMA16(rm, sl + 5120, l16 + 1024, oo); DMA16(rm, sl + 6144, l16 + 2048, oo); DMA16(rm, sl + 7168, l16 + 3072, oo);
                    *(volatile LAS u32x4*)(rdesc + (j & 3) * 4) = (u32x4){unit, st, 0u, K_RED};
                    cur = RFL(c) + dec;
                    c = __builtin_amdgcn_raw_ptr_buffer_atomic_add_i32(1, rm, l0, qo, AGT);
                    CBAR();
                    ++j;
                    resolve();
                }
            }
            post(K_END);
            if (dead && lane == 0) {
                __hip_atomic_store(tmo, call, __ATOMIC_RELAXED, __HIP_MEMORY_SCOPE_AGENT);
                *(volatile u32*)(P.out + (size_t)g * 32768 + pair * 4) = 0x7fc07fc0u;
            }
            return;
        }
        BARRIER();
        const u32 call = RFL(ctrv) + 1u;
        if (lane == 0) __hip_atomic_store(ctr, call, __ATOMIC_RELAXED, __HIP_MEMORY_SCOPE_AGENT);
        if (!nw) return;
        STAMP(pair * 10);
        const u32 sbase = OFF_S + (call & 1u) * 32 * PANEL + l16;
        const u32 tro = ((lane >> 4) * 8 + ((lane >> 2) & 3)) * 128 + (lane & 3) * 32;
        const auto ro = rsrc(P.out, 4u << 20);
        u32 k = 0, full = 0, pend = OOR;
#define FLAG() do { __builtin_amdgcn_raw_buffer_store_b32(call, rd, pend, 0, SYS); CBAR(); pend = OOR; } while (0)
        for (;;) {
            if (full <= k) {
                full = LDS_LD(rdesc + 16);
                if (full <= k) {
                    if (RFL(pend) != OOR) { WAIT_VM(0); FLAG(); }
                    while (full <= k) { __builtin_amdgcn_s_sleep(1); full = LDS_LD(rdesc + 16); }
                }
                CBAR();
            }
            const u32x4 dd = *(volatile LAS u32x4*)(rdesc + (k & 3) * 4);
            const u32 d0 = RFL(dd.x), d1 = RFL(dd.y), kind = RFL(dd.w);
            if (kind >= K_SIG) {
                if (kind == K_SIG) (void)__builtin_amdgcn_raw_ptr_buffer_atomic_add_i32(1, rd, l0, qo + 128, AGT);
                WAIT_VM(0); FLAG();
                if (kind == K_END) break;
                STAMP(pair * 10 + 1);
                ++k; LDS_ST(rdesc + 17, k);
                continue;
            }
            const LAS char* const sl = ring + (k & 3) * 8192;
            u32x4 f[4];
            if (kind == K_W) {
#pragma unroll
                for (int n = 0; n < 4; ++n) {
                    const u32x2 lo = __builtin_bit_cast(u32x2, __builtin_amdgcn_ds_read_tr16_b64_v4i16((LAS s16x4*)(sl + tro + n * 8)));
                    const u32x2 hi = __builtin_bit_cast(u32x2, __builtin_amdgcn_ds_read_tr16_b64_v4i16((LAS s16x4*)(sl + tro + 512 + n * 8)));
                    f[n] = (u32x4){lo.x, lo.y, hi.x, hi.y};
                }
            } else {
#pragma unroll
                for (int n = 0; n < 4; ++n) f[n] = *(const LAS u32x4*)(sl + l16 + n * 1024);
            }
            if (kind != K_RED) {
                asm volatile("s_waitcnt lgkmcnt(0)" : "+v"(f[0]), "+v"(f[1]), "+v"(f[2]), "+v"(f[3]) :: "memory");
                ++k; LDS_ST(rdesc + 17, k);
                const u32 vo0 = sbase + (d0 >> 2) * 16384 + (d0 & 3u) * (d1 == 8192 ? 2048u : 4096u), vo1 = vo0 + d1;
#define WSTEP(i) do { WAIT_VM(WS - 1); if (i == WS - 1) FLAG(); } while (0)
                WSTEP(0); __builtin_amdgcn_raw_buffer_store_b128(f[0], rd, vo0, 0, SYS);
                WSTEP(1); __builtin_amdgcn_raw_buffer_store_b128(f[1], rd, vo0 + 1024, 0, SYS);
                WSTEP(2); __builtin_amdgcn_raw_buffer_store_b128(f[2], rd, vo1, 0, SYS);
                WSTEP(3); __builtin_amdgcn_raw_buffer_store_b128(f[3], rd, vo1 + 1024, 0, SYS);
                if (WS > 4) { WAIT_VM(4); FLAG(); }
                pend = l0 | (OFF_AF + d0 * 4);
                continue;
            }
            u32x4 o[4];
#pragma unroll
            for (int n = 0; n < 4; ++n) o[n] = *(const LAS u32x4*)(sl + 4096 + l16 + n * 1024);
            asm volatile("s_waitcnt lgkmcnt(0)" : "+v"(f[0]), "+v"(f[1]), "+v"(f[2]), "+v"(f[3]), "+v"(o[0]), "+v"(o[1]), "+v"(o[2]), "+v"(o[3]) :: "memory");
            ++k; LDS_ST(rdesc + 17, k);
            const u32 pp = OFF_P + d0 * 4096 + l16;
            if (d1 == 0) {
#pragma unroll
                for (int n = 0; n < 4; ++n) __builtin_amdgcn_raw_buffer_store_b128(f[n], rm, pp + n * 1024, 0, SYS);
                WAIT_VM(0);
                (void)__builtin_amdgcn_raw_ptr_buffer_atomic_add_i32(2, rm, l0, OFF_ST + d0 * 4, AGT);
                CBAR();
                continue;
            }
            dead = d1 == 2;
            if (d1 == 1) {
                const u64 t0 = __builtin_amdgcn_s_memrealtime();
                while ((RFL(__builtin_amdgcn_raw_buffer_load_b32(rm, 0, OFF_ST + d0 * 4, AGT)) & 3u) != 0) {
                    CBAR();
                    if (__builtin_amdgcn_s_memrealtime() - t0 > DEADLINE) { dead = true; break; }
                    __builtin_amdgcn_s_sleep(1);
                }
                CBAR();
#pragma unroll
                for (int n = 0; n < 4; ++n) o[n] = __builtin_amdgcn_raw_buffer_load_b128(rm, pp + n * 1024, 0, SYS);
                asm volatile("" : "+v"(o[0]), "+v"(o[1]), "+v"(o[2]), "+v"(o[3]));
            }
            const float pz = dead ? __builtin_nanf("") : 0.f;
            const u32 oo = ((d0 >> 5) * 128 + ((d0 >> 2) & 1) * 64 + (d0 & 1) * 32 + (lane >> 3)) * 1024 + ((d0 >> 3) & 3) * 256 + ((d0 >> 1) & 1) * 128 + (lane & 7) * 16;
#pragma unroll
            for (int n = 0; n < 4; ++n) {
                u32x4 r;
#pragma unroll
                for (int q = 0; q < 4; ++q) {
                    const float lo = __builtin_bit_cast(float, f[n][q] << 16) + __builtin_bit_cast(float, o[n][q] << 16) + pz;
                    const float hi = __builtin_bit_cast(float, f[n][q] & 0xffff0000u) + __builtin_bit_cast(float, o[n][q] & 0xffff0000u) + pz;
                    r[q] = pack_bf16(gelu(lo), gelu(hi));
                }
                __builtin_amdgcn_raw_buffer_store_b128(r, ro, oo + n * 8192, 0, AGT);
            }
        }
        STAMP(pair * 10 + 4);
        return;
    }

    const u32 call = RFL(ctrv) + 1u;
    if (lane == 0) __hip_atomic_store(ctr, call, __ATOMIC_RELAXED, __HIP_MEMORY_SCOPE_AGENT);
    const u32 gg = blockIdx.x - 128;
    const u32 tj = ((gg & 7) >> 1) * 2 + (gg >> 6), tq = (gg & 1) * 8 + ((gg >> 3) & 7);
    const u32 list = w_dist(tq) ? w_dist(tq) : (tq >> 3) * 2;
    LAS u32* const misc = (LAS u32*)(lds + LDS_MISC);
    if (wid >= 8) {
        const u32 l = wid - 8;
        bool dead = false;
        const u64 t0 = __builtin_amdgcn_s_memrealtime();
        const u32 xb = OFF_S + ((call & 1u) * 32 + 2 * tj) * PANEL, wb = OFF_S + ((call & 1u) * 32 + 16 + tq) * PANEL;
        const u32 gx = l * 2048 + l16, gw = l * 4096 + l16;
        const u32 fsl = lane >> 4;
        const u32 fo = fsl == 3 ? OFF_TMO : OFF_AF + (fsl == 2 ? 16 + tq : 2 * tj + fsl) * 512 + (lane & 15) * 16 + l * 4;
        const u32 fstep = fsl == 3 ? 0u : 16u;
        LAS u32* const pollbuf = (LAS u32*)(lds + LDS_POLL + l * 256);
        u32 avail = 0, pbase = 0;
#define POLL() do { pbase = avail; __builtin_amdgcn_raw_ptr_buffer_load_lds(rm, pollbuf, 4, fo + min(avail, 31u) * fstep, 0, 0, SYS); } while (0)
#define POLL_READ() do { const u32 _v = *(volatile LAS u32*)(pollbuf + lane); \
        const u64 _m = ~__ballot(_v >= call || pbase + (lane & 15) >= 32); \
        const u32 _a = pbase + __builtin_ctz((((u32)_m | (u32)(_m >> 16) | (u32)(_m >> 32)) & 0xffffu) | 0x10000u); \
        avail = max(avail, _a); if (__ballot(_v == call) >> 48) { dead = true; avail = 64; } } while (0)
#define SPIN(n) do { while (avail < (u32)(n)) { POLL(); WAIT_VM(0); POLL_READ(); \
        if (avail < (u32)(n)) { if (__builtin_amdgcn_s_memrealtime() - t0 > DEADLINE) { dead = true; avail = 64; } else __builtin_amdgcn_s_sleep(1); } } CBAR(); } while (0)
#define STAGE_IN(u, sb) do { LAS char* _x = lds + (sb) + l * 2048; LAS char* _w = lds + (sb) + 32768 + l * 4096; const u32 _x0 = xb + (u) * 16384, _w0 = wb + (u) * 16384; \
        DMA16(rm, _x, gx, _x0); DMA16(rm, _x + 1024, gx + 1024, _x0); DMA16(rm, _x + 16384, gx, _x0 + PANEL); DMA16(rm, _x + 17408, gx + 1024, _x0 + PANEL); \
        DMA16(rm, _w, gw, _w0); DMA16(rm, _w + 1024, gw + 1024, _w0); DMA16(rm, _w + 2048, gw + 2048, _w0); DMA16(rm, _w + 3072, gw + 3072, _w0); \
        DMA16(rm, _x + 8192, gx, _x0 + 8192); DMA16(rm, _x + 9216, gx + 1024, _x0 + 8192); DMA16(rm, _x + 24576, gx, _x0 + PANEL + 8192); DMA16(rm, _x + 25600, gx + 1024, _x0 + PANEL + 8192); CBAR(); } while (0)
        if (lane == 0) misc[l] = 0;
        SPIN(1); STAGE_IN(0, 0);
        SPIN(2); POLL(); STAGE_IN(1, STAGE);
        u32 sn = 2 * STAGE;
        for (u32 u = 0; u < 30; ++u) {
            WAIT_VM(12); BARRIER();
            POLL_READ();
            SPIN(u + 3); POLL(); STAGE_IN(u + 2, sn);
            sn = sn == 2 * STAGE ? 0 : sn + STAGE;
        }
        if (l == 0) STAMP(2);
        if (dead && lane == 0) { misc[l] = 1; __hip_atomic_store(tmo, call, __ATOMIC_RELAXED, __HIP_MEMORY_SCOPE_AGENT); }
        WAIT_VM(12); BARRIER();
        if (l == 0 && lane == 0)
            misc[4] = __hip_atomic_fetch_add((u32*)(myws + OFF_LT + list * 128), 1u, __ATOMIC_RELAXED, __HIP_MEMORY_SCOPE_AGENT);
        WAIT_VM(0); BARRIER();
        return;
    }
    const u32 xo = (wid >> 2) * 16384 + ((wid >> 1) & 1) * 4096 + l16, wo = 32768 + (wid & 1) * 4096 + l16;
    f32x4 acc[4][4];
#pragma unroll
    for (int m = 0; m < 4; ++m)
#pragma unroll
        for (int n = 0; n < 4; ++n) acc[m][n] = (f32x4){0.f, 0.f, 0.f, 0.f};
    u32 sb = 0;
    for (u32 u = 0; u < 32; ++u) {
        BARRIER();
        const LAS char* const xp = lds + sb + xo; const LAS char* const wp = lds + sb + wo;
        bf16x8 xf[2][4], wf[2][4];
#pragma unroll
        for (int k = 0; k < 2; ++k)
#pragma unroll
            for (int i = 0; i < 4; ++i) { wf[k][i] = *(const LAS bf16x8*)(wp + k * 8192 + i * 1024); xf[k][i] = *(const LAS bf16x8*)(xp + k * 8192 + i * 1024); }
        __builtin_amdgcn_sched_barrier(0);
#pragma unroll
        for (int k = 0; k < 2; ++k)
#pragma unroll
            for (int m = 0; m < 4; ++m)
#pragma unroll
                for (int n = 0; n < 4; ++n) acc[m][n] = __builtin_amdgcn_mfma_f32_16x16x32_bf16(wf[k][n], xf[k][m], acc[m][n], 0, 0, 0);
        __builtin_amdgcn_sched_barrier(0);
        sb = sb == 2 * STAGE ? 0 : sb + STAGE;
    }
    if (wid == 0) STAMP(3);
    const u32x4 dv = *(volatile LAS u32x4*)misc;
    const float pz = (dv.x | dv.y | dv.z | dv.w) ? __builtin_nanf("") : 0.f;
    const u32 id = (tj * 16 + tq) * 8 + wid;
    const auto rt = rsrc(myws + OFF_T + id * 8192, 8192);
#pragma unroll
    for (int m = 0; m < 4; ++m)
#pragma unroll
        for (int h = 0; h < 2; ++h) {
            u32x4 v;
            v.x = pack_bf16(acc[m][2 * h][0] + pz, acc[m][2 * h][1]);
            v.y = pack_bf16(acc[m][2 * h][2], acc[m][2 * h][3]);
            v.z = pack_bf16(acc[m][2 * h + 1][0], acc[m][2 * h + 1][1]);
            v.w = pack_bf16(acc[m][2 * h + 1][2], acc[m][2 * h + 1][3]);
            __builtin_amdgcn_raw_buffer_store_b128(v, rt, l16 + (m * 2 + h) * 1024, 0, SYS);
        }
    const u32 fo = OFF_RF + list * 512 + ((LDS_LD(misc + 4) & 15u) * 8 + wid) * 4;
    WAIT_VM(0);
    __builtin_amdgcn_raw_buffer_store_b32(call << 10 | id, rsrc(P.ws[rank ^ w_dist(tq)], WS_NEED), l0, fo, SYS);
    if (wid == 0) STAMP(4);
}

static int knob(const char*, int d) { return d; }
static void make_schedule(Params& p) {
    const int ne = knob("NE", NE), up = knob("UP", UP), lead = knob("LEAD", LEAD), hw = knob("HW", HW), rnum = knob("RNUM", RNUM), wl = knob("WL", WL);
    p.pn = up | ne << 8 | knob("GX", GX) << 16;
    int n1 = 0;
    for (int t = -lead; t < 128; ++t) {
        const int wi = t + lead;
        if (t >= 0) { p.tab[1][n1++] = t; p.tab[1][n1++] = t | 1 << 7; }
        if (wi < 128 && (wi < 4 * up || wi >= 4 * (up + ne))) { p.tab[1][n1++] = wi | 2 << 7; p.tab[1][n1++] = wi | 3 << 7; }
    }
    int n0 = 0, nd = 0, r = 0;
    auto direct = [&](int e) { p.tab[0][n0++] = e; if (++nd >= hw) for (int k = 0; k < rnum && r < 16 * ne; ++k, ++r) p.tab[0][n0++] = (r >> 2) | (r & 3) << 7 | 1 << 9; };
    for (int t = -wl; t < 128; ++t) { if (t + wl < 128) direct(t + wl); if (t >= 0) direct(t | 1 << 7); }
    for (; r < 16 * ne; ++r) p.tab[0][n0++] = (r >> 2) | (r & 3) << 7 | 1 << 9;
    for (int i = n0; i < 512; ++i) p.tab[0][i] = 0;
    for (int i = n1; i < 512; ++i) p.tab[1][i] = 0;
    for (int i = 0; i < 16 * NPW; ++i) p.grp[i] = Group{};
    int g = 0, nl = 0, qid = 0;
    auto add = [&](int ng, u32 d, u32 ea, u32 eb, u32 slots, u32 tab) {
        for (int i = 0; i < ng; ++i) {
            const Group G = {{d | qid << 8 | (8 * ng) << 16 | (8 * i) << 24, ea | eb << 16, slots, tab}};
            if (d) p.grp[g++] = G; else { p.grp[(nl >> 1) * 16 + 14 + (nl & 1)] = G; ++nl; }
        }
        ++qid;
    };
    for (u32 dd = 1; dd <= 8; ++dd) {
        const u32 d = dd & 7;
        u32 xs[4], ws[4]; int nx = 0, nwv = 0;
        for (u32 s = 0; s < 16; ++s) { if (x_dist(s) == d) xs[nx++] = s; if (w_dist(s) == d) ws[nwv++] = 16 + s; }
        if (d == 0) for (u32 h = 0; h < 2; ++h) add(4, 0, 512, 768, xs[2 * h] | xs[2 * h + 1] << 8 | ws[2 * h] << 16 | ws[2 * h + 1] << 24, (2 * h) << 4);
        else if (d == 2) add(2, d, 512, 512, xs[0] | xs[1] << 8 | xs[2] << 16 | xs[3] << 24, knob("D2F", D2F) << 8);
        else if (d & 1) add(2, d, n1, n1 + 256, xs[0] | xs[1] << 8 | ws[0] << 16 | ws[1] << 24, 1 | d << 4);
        else add(2, d, n0, n0 + 256, ws[0] | ws[1] << 8, 2 | d << 4);
    }
}

}

extern "C" void kernel_launch(int rank, int nranks, ncclComm_t comm,
                              hipStream_t stream,
                              void* const* d_in, const int* in_sizes, int n_in,
                              void* d_out, int out_size,
                              const int* in_dtypes, int out_dtype,
                              void* const* d_out_all,
                              void* const* d_ws_all, size_t ws_bytes) {
    if (nranks != NR || (unsigned)rank >= (unsigned)NR || ws_bytes < WS_NEED || n_in < 2 ||
        in_sizes[0] != 512 * 4096 || in_sizes[1] != 4096 * 512 || out_size != 4096 * 512) {
        fprintf(stderr, "unexpected problem shape\n");
        return;
    }
    Params p;
    p.x = (const char*)d_in[0];
    p.w = (const char*)d_in[1];
    p.out = (char*)d_out;
    for (int i = 0; i < NR; ++i) p.ws[i] = (char*)d_ws_all[i];
    p.rank = rank;
    make_schedule(p);
    hipLaunchKernelGGL(ag_gemm_kernel, dim3(256), dim3(768), 0, stream, p);
    HIP_CHECK(hipGetLastError());
}
```

```cpp
#include <hip/hip_runtime.h>
#include <rccl/rccl.h>
#include <cstdio>
#include <cstdlib>

#define HIP_CHECK(expr) do { hipError_t _e = (expr); if (_e != hipSuccess) { \
    fprintf(stderr, "HIP error %s at %s:%d\n", hipGetErrorString(_e), __FILE__, __LINE__); \
    return; } } while (0)

namespace {

typedef unsigned u32;
typedef unsigned long long u64;
typedef u32 u32x4 __attribute__((ext_vector_type(4)));
typedef u32 u32x2 __attribute__((ext_vector_type(2)));
typedef float f32x4 __attribute__((ext_vector_type(4)));
typedef float f32x2 __attribute__((ext_vector_type(2)));
typedef short s16x4 __attribute__((ext_vector_type(4)));
typedef __bf16 bf16x8 __attribute__((ext_vector_type(8)));
typedef __bf16 bf16x2 __attribute__((ext_vector_type(2)));

constexpr int NR = 8;
constexpr u32 PANEL = 512u << 10;
constexpr u32 OFF_S = 0;
constexpr u32 OFF_T = 32u << 20;
constexpr u32 OFF_P = 40u << 20;
constexpr u32 OFF_AF = 44u << 20;
constexpr u32 OFF_RF = OFF_AF + 16384;
constexpr u32 OFF_ST = OFF_RF + 4096;
constexpr u32 OFF_CTR = OFF_ST + 4096;
constexpr u32 OFF_TMO = OFF_CTR + 16384;
constexpr u32 OFF_Q = OFF_TMO + 1024;
constexpr u32 OFF_LT = OFF_Q + 4096;
constexpr u32 OFF_DBG = OFF_LT + 1024;
constexpr u32 WS_NEED = OFF_DBG + (1024u << 10);
constexpr u64 DEADLINE = 100000000ull;
constexpr int SYS = 17;
constexpr int AGT = 16;
constexpr u32 OOR = 0x80000000u;

#define NE 8
#define UP 24
#define LEAD 40
#define HW 192
#define RNUM 4
#define WL 24
#define D2F 96
#define WS 3
#define WP 3
#define RING 2
#define RINGB 2
#define GX 14
#define NPW 4

struct Group { u32 w[4]; };
struct Params {
    const char* x;
    const char* w;
    char* out;
    char* ws[NR];
    int rank; u32 pn;
    Group grp[16 * NPW];
    unsigned short tab[2][512];
};

__host__ __device__ __forceinline__ u32 x_dist(u32 p) { return (0x7520'3210u >> (4 * (p >> 1))) & 7u; }
__host__ __device__ __forceinline__ u32 x_panel(u32 p) { return ((p >> 3) << 1) | (p & 1u); }
__host__ __device__ __forceinline__ u32 w_dist(u32 q) { return (u32)(0x6530'7610'7430'5410ull >> (4 * q)) & 7u; }
__host__ __device__ __forceinline__ u32 w_panel(u32 q) { return q >> 2; }

__device__ __forceinline__ __amdgpu_buffer_rsrc_t rsrc(const void* p, u32 bytes) {
    return __builtin_amdgcn_make_buffer_rsrc(const_cast<void*>(p), 0, bytes, 0x00020000);
}
__device__ __forceinline__ u32 pack_bf16(float a, float b) {
    return __builtin_bit_cast(u32, __builtin_convertvector((f32x2){a, b}, bf16x2));
}
__device__ __forceinline__ float gelu(float y) {
    const float t = y * (-2.302208198f - 0.1029432397f * y * y);
    return y * __builtin_amdgcn_rcpf(1.0f + __builtin_amdgcn_exp2f(t));
}
#define WAIT_VM(n) asm volatile("s_waitcnt vmcnt(%0)" :: "i"(n) : "memory")
#define CBAR() asm volatile("" ::: "memory")
#define RFL(x) ((u32)__builtin_amdgcn_readfirstlane(x))
#define INL __attribute__((always_inline))
#define LAS __attribute__((address_space(3)))
#define BARRIER() do { asm volatile("s_waitcnt lgkmcnt(0)" ::: "memory"); __builtin_amdgcn_s_barrier(); CBAR(); } while (0)
#define LDS_LD(p) RFL(*(volatile LAS u32*)(p))
#define LDS_ST(p, v) do { *(volatile LAS u32*)(p) = (v); } while (0)
#define DMA16(r, d, v, o) __builtin_amdgcn_raw_ptr_buffer_load_lds(r, d, 16, v, o, 0, SYS)

#define STAMP(i) do {} while (0)

constexpr u32 STAGE = 49152;
constexpr u32 LDS_POLL = 3 * STAGE;
constexpr u32 LDS_MISC = LDS_POLL + 1024;
constexpr u32 LDS_BYTES = LDS_MISC + 64;
enum { K_COPY, K_W, K_RED, K_SIG, K_END };

__global__ void __launch_bounds__(768, 3) ag_gemm_kernel(const Params P) {
    __shared__ __attribute__((aligned(16))) char smem[LDS_BYTES];
    LAS char* const lds = (LAS char*)smem;
    const u32 wid = RFL(threadIdx.x >> 6);
    const u32 lane = threadIdx.x & 63;
    const u32 rank = P.rank & 7;
    char* const myws = P.ws[rank];
    if (blockIdx.x < 128 && (wid & 4)) return;
    u32* const ctr = (u32*)(myws + OFF_CTR) + blockIdx.x * 16 + wid;
    const u32 ctrv = __hip_atomic_load(ctr, __ATOMIC_RELAXED, __HIP_MEMORY_SCOPE_AGENT);
    u32* const tmo = (u32*)(myws + OFF_TMO);
    const u32 l0 = lane ? OOR : 0u, l16 = lane * 16;
    const auto rm = rsrc(myws, WS_NEED);
    const u32 vx = (lane & 15) * 8192 + (lane >> 4) * 16, vw = (lane >> 3) * 1024 + (lane & 7) * 16;
    const u32 tro = ((lane >> 4) * 8 + ((lane >> 2) & 3)) * 128 + (lane & 3) * 32;

    if (blockIdx.x < 128) {
        const u32 g = blockIdx.x, pair = wid & 3;
        const Group G = P.grp[pair * 16 + (g >> 3)];
        const u32 dist = G.w[0] & 7u, nw = (G.w[0] >> 16) & 255u, ea = G.w[1] & 0xffffu, eb = G.w[1] >> 16;
        const u32 peer = rank ^ dist, qo = OFF_Q + ((G.w[0] >> 8) & 15u) * 256;
        const auto rd = rsrc(P.ws[peer], WS_NEED);
        LAS char* const ring = lds + pair * 32768;
        LAS u32* const rdesc = (LAS u32*)(lds + 131072 + pair * 128);
        bool dead = false;
        if (wid >= 8) {
            if (lane < 2) rdesc[16 + lane] = 0;
            const u32 up = P.pn & 255u, une = (P.pn >> 8) & 255u, df = G.w[3] >> 8;
            const u32 rsl = dist == 4 ? (17u | 19u << 8 | 21u << 16 | 23u << 24) : (25u | 27u << 8 | 29u << 16 | 30u << 24);
            const u32 rdl = dist == 4 ? (19u | 17u << 8 | 23u << 16 | 21u << 24) : (27u | 25u << 8 | 30u << 16 | 29u << 24);
            const u32* const tab = (const u32*)P.tab[G.w[3] & 1];
            const u32 urev = (peer & 4) ? 31u : 0u;
            const char* const xs = P.x + (peer & 1) * 4096;
            const char* const wsrc = P.w + (size_t)(peer & 1) * (2u << 20);
            const u32 xbytes = (4u << 20) - (peer & 1) * 4096;
            u32x4 dv = {0u, 0u, 0u, 0u};
            u32 cur = (G.w[0] >> 24) + (g & 7), c = 0, j = 0, pub = 0, used = 0, call = 0, dec = 0, sbase = 0, depth = RING;
            u64 t0 = 0;
            auto timed_out = [&]() INL { return __builtin_amdgcn_s_memrealtime() - t0 > DEADLINE; };
            auto publish = [&](u32 n) INL { if (n != pub) { pub = n; LDS_ST(rdesc + 16, n); } };
            auto slot_wait = [&]() INL {
                if (used + depth > j) return;
                used = LDS_LD(rdesc + 17);
                if (used + depth > j) return;
                WAIT_VM(0); publish(j);
                while (used + depth <= j) { __builtin_amdgcn_s_sleep(1); used = LDS_LD(rdesc + 17); }
            };
            auto word = [&](u32 w) INL -> u32 {
                const u32 l = w >> 2;
                const u32 v0 = __builtin_amdgcn_readlane(dv.x, l), v1 = __builtin_amdgcn_readlane(dv.y, l), v2 = __builtin_amdgcn_readlane(dv.z, l), v3 = __builtin_amdgcn_readlane(dv.w, l);
                return (w & 2) ? ((w & 1) ? v3 : v2) : ((w & 1) ? v1 : v0);
            };
            auto dep_wait = [&](u32 w, u32 sh, u32 want, u32 dvo) INL {
                if ((word(w) >> sh) >= want) return;
                WAIT_VM(0); publish(j);
                while (!dead) {
                    dv = __builtin_amdgcn_raw_buffer_load_b128(rm, dvo, 0, SYS);
                    u32 t = RFL(__hip_atomic_load(tmo, __ATOMIC_RELAXED, __HIP_MEMORY_SCOPE_AGENT));
                    asm volatile("" : "+s"(t));
                    if ((word(w) >> sh) >= want) break;
                    if (t == call || timed_out()) dead = true;
                    __builtin_amdgcn_s_sleep(1);
                }
                CBAR();
            };
            auto post = [&](u32 kind) INL { slot_wait(); LDS_ST(rdesc + (j & 3) * 4 + 3, kind); WAIT_VM(0); ++j; publish(j); };
            const u32 dvr = OFF_AF + ((rsl >> (8 * (lane >> 4))) & 31u) * 512 + (lane & 15) * 16;
#define PUSH_STEP(first) do { \
                const u32 e = (G.w[3] & 3) ? (tab[cur >> 1] >> ((cur & 1) * 16)) & 1023u \
                            : cur < 2 * df ? (cur >> 1) | (cur & 1) << 7 : cur >= 512 - 2 * df ? ((cur >> 1) - 128) | (2 + (cur & 1)) << 7 \
                            : (((cur - 2 * df) >> 2) + ((cur - 2 * df) & 2 ? 0u : df)) | ((cur - 2 * df) & 3) << 7; \
                const u32 i = e & 127u, k = (e >> 7) & 3u; \
                u32 kind = K_COPY, so, s1 = 1024, s2 = 2048, pc, d2 = 2048, vo = l16, src = 2; \
                if (e >> 9) { \
                    dep_wait(k * 64 + i, 0, call, dvr); \
                    so = sbase + ((rsl >> (8 * k)) & 31u) * PANEL + i * 4096; \
                    pc = ((rdl >> (8 * k)) & 31u) * 128 + i + 4 * up; \
                } else { \
                    const u32 v = i >> 2, slot = (G.w[2] >> (8 * k)) & 31u, u = (v < up ? v : v < up + une ? 31 + up - v : v - une) ^ urev, h = i & 3; \
                    pc = slot * 128 + i; \
                    if (slot >= 16) { kind = K_W; src = 1; vo = vw; s1 = 8192; s2 = 16384; so = (u * 64 + (h >> 1) * 32) * 1024 + w_panel(slot & 15) * 256 + (h & 1) * 128; } \
                    else { src = 0; vo = vx; s1 = 131072; s2 = 64; d2 = 8192; so = (x_panel(slot) * 128 + h * 32) * 8192 + u * 128; } \
                } \
                slot_wait(); \
                const auto rs = rsrc(src == 2 ? myws + 0 : src ? wsrc + 0 : xs + 0, src == 2 ? WS_NEED : src ? (2u << 20) : xbytes); \
                LAS char* const sl = ring + (j & 3) * 8192; \
                if (first && (!nw || !ea)) so = OOR; \
                DMA16(rs, sl, vo, so); DMA16(rs, sl + 1024, vo, so + s1); DMA16(rs, sl + 2048, vo, so + s2); DMA16(rs, sl + 3072, vo, so + s1 + s2); \
                *(volatile LAS u32x4*)(rdesc + (j & 3) * 4) = (u32x4){pc, d2, 0u, kind}; \
                if (first) { \
                    BARRIER(); \
                    asm volatile("s_waitcnt vmcnt(4)\n\tv_readfirstlane_b32 %0, %1" : "=s"(call) : "v"(ctrv) : "memory"); \
                    ++call; \
                    if (lane == 0) __hip_atomic_store(ctr, call, __ATOMIC_RELAXED, __HIP_MEMORY_SCOPE_AGENT); \
                    t0 = __builtin_amdgcn_s_memrealtime(); \
                    sbase = OFF_S + (call & 1u) * 32 * PANEL; \
                    dec = 2u * nw - (call - 1u) * eb; \
                    c = cur + nw - dec; \
                    if (!nw || !ea) break; \
                } else { \
                    WAIT_VM(4); \
                    publish(j); \
                } \
                cur = RFL(c) + dec; \
                c = __builtin_amdgcn_raw_ptr_buffer_atomic_add_i32(1, rm, l0, qo, AGT); \
                CBAR(); \
                ++j; \
             \
            } while (0)
            PUSH_STEP(true);
            if (!nw) return;
            while (cur < ea) PUSH_STEP(false);
            if (eb > ea) {
                if (dist) post(K_SIG);
                depth = RINGB;
                dv = (u32x4){0u, 0u, 0u, 0u};
                const u32 dvb = OFF_RF + ((G.w[3] >> 4) & 7u) * 512 + (lane & 31) * 16;
                const u32 po = (lane & 1) * 1024 + ((lane >> 1) & 3) * 256 + (lane >> 3) * 16;
                u32 so = 0, unit = 0, tk = 0;
                auto resolve = [&]() INL {
                    if (cur < eb) {
                        const u32 r = cur - ea;
                        dep_wait((r >> 1) & 127u, 12, call, dvb);
                        const u32 id = word((r >> 1) & 127u) & 1023u, tq = (id >> 3) & 15u, wv = id & 7u, tp = 2 * (id >> 7) + (wv >> 2);
                        so = (word((r >> 1) & 127u) >> (10 + (r & 1))) & 1 ? OFF_T + id * 8192 + (r & 1) * 4096 : OOR;
                        unit = (((peer ^ x_dist(tp)) * 4 + x_panel(tp)) * 4 + w_panel(tq)) * 8 + (wv & 3) * 2 + (r & 1);
                    }
                    tk = __builtin_amdgcn_raw_ptr_buffer_atomic_add_i32(dist ? 1 : 0, rm, cur < eb && !dead ? l0 : OOR, OFF_ST + unit * 4, AGT);
                    CBAR();
                };
                resolve();
                while (dist && !dead) {
                    const u32 v = RFL(__builtin_amdgcn_raw_buffer_load_b32(rm, 0, qo + 128, SYS));
                    CBAR();
                    if (v + (P.pn >> 16) >= call * nw) break;
                    if (timed_out()) dead = true;
                    __builtin_amdgcn_s_sleep(1);
                }
                STAMP(pair * 10 + 7);
                while (cur < eb) {
                    slot_wait();
                    LAS char* const sl = ring + (j & 3) * 8192;
                    WAIT_VM(WP + 5); DMA16(rd, sl, po, so);
                    WAIT_VM(WP + 5); DMA16(rd, sl + 1024, po + 128, so);
                    WAIT_VM(WP + 5); DMA16(rd, sl + 2048, po + 2048, so);
                    WAIT_VM(WP + 5); DMA16(rd, sl + 3072, po + 2176, so);
                    CBAR();
                    const u32 tv = RFL(tk) & 3u, st = dead ? 2u : so == OOR ? 4u : dist ? tv : tv ? 1u : 3u;
                    WAIT_VM(4);
                    publish(j);
                    const u32 t = (unit >> 6) & 1, cp = (unit >> 3) & 3, ob = peer ^ (t << 2 | ((cp ^ ((cp >> 1) & t)) & 1) << 1 | 1);
                    if (st == 1 || (st == 3 && ob == rank)) {
                        u32 tj2 = t * 4, tq2 = cp * 4;
                        while (x_dist(2 * tj2) != (ob ^ (unit >> 7))) ++tj2;
                        while (w_dist(tq2) != (ob ^ rank)) ++tq2;
                        const auto r2 = rsrc(P.ws[ob], WS_NEED);
                        const u32 o2 = OFF_T + (tj2 * 16 + tq2) * 65536 + (so & 0xf000u);
                        DMA16(r2, sl + 4096, po, o2); DMA16(r2, sl + 5120, po + 128, o2); DMA16(r2, sl + 6144, po + 2048, o2); DMA16(r2, sl + 7168, po + 2176, o2);
                    } else {
                        const u32 oo = st == 3 ? OFF_P + unit * 4096 : OOR;
                        DMA16(rm, sl + 4096, l16, oo); DMA16(rm, sl + 5120, l16 + 1024, oo); DMA16(rm, sl + 6144, l16 + 2048, oo); DMA16(rm, sl + 7168, l16 + 3072, oo);
                    }
                    *(volatile LAS u32x4*)(rdesc + (j & 3) * 4) = (u32x4){unit, st, 0u, K_RED};
                    cur = RFL(c) + dec;
                    c = __builtin_amdgcn_raw_ptr_buffer_atomic_add_i32(1, rm, l0, qo, AGT);
                    CBAR();
                    ++j;
                    resolve();
                }
            }
            post(K_END);
            if (dead && lane == 0) {
                __hip_atomic_store(tmo, call, __ATOMIC_RELAXED, __HIP_MEMORY_SCOPE_AGENT);
                *(volatile u32*)(P.out + (size_t)g * 32768 + pair * 4) = 0x7fc07fc0u;
            }
            return;
        }
        BARRIER();
        const u32 call = RFL(ctrv) + 1u;
        if (lane == 0) __hip_atomic_store(ctr, call, __ATOMIC_RELAXED, __HIP_MEMORY_SCOPE_AGENT);
        if (!nw) return;
        STAMP(pair * 10);
        const u32 sbase = OFF_S + (call & 1u) * 32 * PANEL + l16;
        const auto ro = rsrc(P.out, 4u << 20);
        u32 k = 0, full = 0, pend = OOR, park = OOR;
#define PARKED() do { (void)__builtin_amdgcn_raw_ptr_buffer_atomic_add_i32(2, rm, l0, park, AGT); CBAR(); park = OOR; } while (0)
#define FLAG() do { __builtin_amdgcn_raw_buffer_store_b32(call, rd, pend, 0, SYS); CBAR(); pend = OOR; } while (0)
        for (;;) {
            if (full <= k) {
                full = LDS_LD(rdesc + 16);
                if (full <= k) {
                    if (RFL(pend) != OOR) { WAIT_VM(0); FLAG(); }
                    while (full <= k) { __builtin_amdgcn_s_sleep(1); full = LDS_LD(rdesc + 16); }
                }
                CBAR();
            }
            const u32x4 dd = *(volatile LAS u32x4*)(rdesc + (k & 3) * 4);
            const u32 d0 = RFL(dd.x), d1 = RFL(dd.y), kind = RFL(dd.w);
            if (kind >= K_SIG) {
                if (kind == K_SIG) (void)__builtin_amdgcn_raw_ptr_buffer_atomic_add_i32(1, rd, l0, qo + 128, AGT);
                WAIT_VM(0); FLAG();
                if (kind == K_END) break;
                STAMP(pair * 10 + 1);
                ++k; LDS_ST(rdesc + 17, k);
                continue;
            }
            const LAS char* const sl = ring + (k & 3) * 8192;
            u32x4 f[4];
            if (kind == K_W) {
#pragma unroll
                for (int n = 0; n < 4; ++n) {
                    const u32x2 lo = __builtin_bit_cast(u32x2, __builtin_amdgcn_ds_read_tr16_b64_v4i16((LAS s16x4*)(sl + tro + n * 8)));
                    const u32x2 hi = __builtin_bit_cast(u32x2, __builtin_amdgcn_ds_read_tr16_b64_v4i16((LAS s16x4*)(sl + tro + 512 + n * 8)));
                    f[n] = (u32x4){lo.x, lo.y, hi.x, hi.y};
                }
            } else {
#pragma unroll
                for (int n = 0; n < 4; ++n) f[n] = *(const LAS u32x4*)(sl + l16 + n * 1024);
            }
            if (kind != K_RED) {
                asm volatile("s_waitcnt lgkmcnt(0)" : "+v"(f[0]), "+v"(f[1]), "+v"(f[2]), "+v"(f[3]) :: "memory");
                ++k; LDS_ST(rdesc + 17, k);
                const u32 vo0 = sbase + (d0 >> 2) * 16384 + (d0 & 3u) * (d1 == 8192 ? 2048u : 4096u), vo1 = vo0 + d1;
#define WSTEP(i) do { WAIT_VM(WS - 1); if (i == WS - 1) FLAG(); } while (0)
                WSTEP(0); __builtin_amdgcn_raw_buffer_store_b128(f[0], rd, vo0, 0, SYS);
                WSTEP(1); __builtin_amdgcn_raw_buffer_store_b128(f[1], rd, vo0 + 1024, 0, SYS);
                WSTEP(2); __builtin_amdgcn_raw_buffer_store_b128(f[2], rd, vo1, 0, SYS);
                WSTEP(3); __builtin_amdgcn_raw_buffer_store_b128(f[3], rd, vo1 + 1024, 0, SYS);
                if (WS > 4) { WAIT_VM(4); FLAG(); }
                pend = l0 | (OFF_AF + d0 * 4);
                continue;
            }
            u32x4 o[4];
#pragma unroll
            for (int n = 0; n < 4; ++n) o[n] = *(const LAS u32x4*)(sl + 4096 + l16 + n * 1024);
            asm volatile("s_waitcnt lgkmcnt(0)" : "+v"(f[0]), "+v"(f[1]), "+v"(f[2]), "+v"(f[3]), "+v"(o[0]), "+v"(o[1]), "+v"(o[2]), "+v"(o[3]) :: "memory");
            ++k; LDS_ST(rdesc + 17, k);
            if (d1 == 4) continue;
            const u32 pp = OFF_P + d0 * 4096 + l16;
            if (d1 == 0) {
#pragma unroll
                for (int n = 0; n < 4; ++n) __builtin_amdgcn_raw_buffer_store_b128(f[n], rm, pp + n * 1024, 0, SYS);
                park = OFF_ST + d0 * 4;
                WAIT_VM(0); PARKED();
                continue;
            }
            dead = d1 == 2;
            const float pz = dead ? __builtin_nanf("") : 0.f;
            const u32 oo = ((d0 >> 5) * 128 + ((d0 >> 2) & 1) * 64 + (d0 & 1) * 32 + (lane >> 3)) * 1024 + ((d0 >> 3) & 3) * 256 + ((d0 >> 1) & 1) * 128 + (lane & 7) * 16;
#pragma unroll
            for (int n = 0; n < 4; ++n) {
                u32x4 r;
#pragma unroll
                for (int q = 0; q < 4; ++q) {
                    const float lo = __builtin_bit_cast(float, f[n][q] << 16) + __builtin_bit_cast(float, o[n][q] << 16) + pz;
                    const float hi = __builtin_bit_cast(float, f[n][q] & 0xffff0000u) + __builtin_bit_cast(float, o[n][q] & 0xffff0000u) + pz;
                    r[q] = pack_bf16(gelu(lo), gelu(hi));
                }
                __builtin_amdgcn_raw_buffer_store_b128(r, ro, oo + n * 8192, 0, AGT);
            }

        }
        STAMP(pair * 10 + 4);
        return;
    }

    const u32 gg = blockIdx.x - 128;
    const u32 tj = ((gg & 7) >> 1) * 2 + (gg >> 6), tq = (gg & 1) * 8 + ((gg >> 3) & 7);
    const u32 list = w_dist(tq) ? w_dist(tq) : (tq >> 3) * 2;
    LAS u32* const misc = (LAS u32*)(lds + LDS_MISC);
    if (wid >= 8) {
        const u32 l = wid - 8;
        const u32 call = RFL(ctrv) + 1u;
        if (lane == 0) __hip_atomic_store(ctr, call, __ATOMIC_RELAXED, __HIP_MEMORY_SCOPE_AGENT);
        bool dead = false;
        const u64 t0 = __builtin_amdgcn_s_memrealtime();
        const u32 xb = OFF_S + ((call & 1u) * 32 + 2 * tj) * PANEL, wb = OFF_S + ((call & 1u) * 32 + 16 + tq) * PANEL;
        const u32 gx = l * 2048 + l16, gw = l * 4096 + l16;
        const u32 fsl = lane >> 4;
        const u32 fo = fsl == 3 ? OFF_TMO : OFF_AF + (fsl == 2 ? 16 + tq : 2 * tj + fsl) * 512 + (lane & 15) * 16 + l * 4;
        const u32 fstep = fsl == 3 ? 0u : 16u;
        LAS u32* const pollbuf = (LAS u32*)(lds + LDS_POLL + l * 256);
        u32 avail = 0, pbase = 0;
#define POLL() do { pbase = avail; __builtin_amdgcn_raw_ptr_buffer_load_lds(rm, pollbuf, 4, fo + min(avail, 31u) * fstep, 0, 0, SYS); } while (0)
#define POLL_READ() do { const u32 _v = *(volatile LAS u32*)(pollbuf + lane); \
        const u64 _m = ~__ballot(_v >= call || pbase + (lane & 15) >= 32); \
        const u32 _a = pbase + __builtin_ctz((((u32)_m | (u32)(_m >> 16) | (u32)(_m >> 32)) & 0xffffu) | 0x10000u); \
        avail = max(avail, _a); if (__ballot(_v == call) >> 48) { dead = true; avail = 64; } } while (0)
#define SPIN(n) do { while (avail < (u32)(n)) { POLL(); WAIT_VM(0); POLL_READ(); \
        if (avail < (u32)(n)) { if (__builtin_amdgcn_s_memrealtime() - t0 > DEADLINE) { dead = true; avail = 64; } else __builtin_amdgcn_s_sleep(1); } } CBAR(); } while (0)
#define STAGE_IN(u, sb) do { LAS char* _x = lds + (sb) + l * 2048; LAS char* _w = lds + (sb) + 32768 + l * 4096; const u32 _x0 = xb + (u) * 16384, _w0 = wb + (u) * 16384; \
        DMA16(rm, _x, gx, _x0); DMA16(rm, _x + 1024, gx + 1024, _x0); DMA16(rm, _x + 16384, gx, _x0 + PANEL); DMA16(rm, _x + 17408, gx + 1024, _x0 + PANEL); \
        DMA16(rm, _w, gw, _w0); DMA16(rm, _w + 1024, gw + 1024, _w0); DMA16(rm, _w + 2048, gw + 2048, _w0); DMA16(rm, _w + 3072, gw + 3072, _w0); \
        DMA16(rm, _x + 8192, gx, _x0 + 8192); DMA16(rm, _x + 9216, gx + 1024, _x0 + 8192); DMA16(rm, _x + 24576, gx, _x0 + PANEL + 8192); DMA16(rm, _x + 25600, gx + 1024, _x0 + PANEL + 8192); CBAR(); } while (0)
        if (lane == 0) misc[l] = 0;
        SPIN(1); STAGE_IN(0, 0);
        SPIN(2); POLL(); STAGE_IN(1, STAGE);
        u32 sn = 2 * STAGE;
        for (u32 u = 0; u < 30; ++u) {
            WAIT_VM(12); BARRIER();
            POLL_READ();
            SPIN(u + 3); POLL(); STAGE_IN(u + 2, sn);
            sn = sn == 2 * STAGE ? 0 : sn + STAGE;
        }
        if (l == 0) STAMP(2);
        if (dead && lane == 0) { misc[l] = 1; __hip_atomic_store(tmo, call, __ATOMIC_RELAXED, __HIP_MEMORY_SCOPE_AGENT); }
        WAIT_VM(12); BARRIER();
        if (l == 0 && lane == 0)
            misc[4] = __hip_atomic_fetch_add((u32*)(myws + OFF_LT + list * 128), 1u, __ATOMIC_RELAXED, __HIP_MEMORY_SCOPE_AGENT);
        WAIT_VM(0); BARRIER();
        return;
    }
    u32 call;
    {
        const u32 h = wid & 3, isw = wid >> 2, v = gg & 31u, pn = gg >> 5, slot = isw ? 16 + pn * 4 : (pn >> 1) * 8 + (pn & 1);
        const u32 up = P.pn & 255u, une = (P.pn >> 8) & 255u, u = (v < up ? v : v < up + une ? 31 + up - v : v - une) ^ ((rank & 4) ? 31u : 0u);
        LAS char* const sc = lds + 2 * STAGE + wid * 4096;
        const auto rs = rsrc(isw ? P.w : P.x, 4u << 20);
        const u32 so = isw ? ((rank & 1) * 2048 + u * 64 + (h >> 1) * 32) * 1024 + pn * 256 + (h & 1) * 128 : (pn * 128 + h * 32) * 8192 + (rank & 1) * 4096 + u * 128;
        const u32 s1 = isw ? 8192u : 131072u, s2 = isw ? 16384u : 64u, vo = isw ? vw : vx;
        DMA16(rs, sc, vo, so); DMA16(rs, sc + 1024, vo, so + s1); DMA16(rs, sc + 2048, vo, so + s2); DMA16(rs, sc + 3072, vo, so + s1 + s2);
        CBAR();
        call = RFL(ctrv) + 1u;
        if (lane == 0) __hip_atomic_store(ctr, call, __ATOMIC_RELAXED, __HIP_MEMORY_SCOPE_AGENT);
        WAIT_VM(1);
        u32x4 f[4];
        if (isw) {
#pragma unroll
            for (int n = 0; n < 4; ++n) {
                const u32x2 lo = __builtin_bit_cast(u32x2, __builtin_amdgcn_ds_read_tr16_b64_v4i16((LAS s16x4*)(sc + tro + n * 8)));
                const u32x2 hi = __builtin_bit_cast(u32x2, __builtin_amdgcn_ds_read_tr16_b64_v4i16((LAS s16x4*)(sc + tro + 512 + n * 8)));
                f[n] = (u32x4){lo.x, lo.y, hi.x, hi.y};
            }
        } else {
#pragma unroll
            for (int n = 0; n < 4; ++n) f[n] = *(const LAS u32x4*)(sc + l16 + n * 1024);
        }
        const u32 vo0 = OFF_S + ((call & 1u) * 32 + slot) * PANEL + v * 16384 + h * (isw ? 4096u : 2048u) + l16, vo1 = vo0 + (isw ? 2048u : 8192u);
        __builtin_amdgcn_raw_buffer_store_b128(f[0], rm, vo0, 0, SYS); __builtin_amdgcn_raw_buffer_store_b128(f[1], rm, vo0 + 1024, 0, SYS);
        __builtin_amdgcn_raw_buffer_store_b128(f[2], rm, vo1, 0, SYS); __builtin_amdgcn_raw_buffer_store_b128(f[3], rm, vo1 + 1024, 0, SYS);
        WAIT_VM(0);
        __builtin_amdgcn_raw_buffer_store_b32(call, rm, l0, OFF_AF + (slot * 128 + v * 4 + h) * 4, SYS);
        if (wid == 0) STAMP(6);
    }
    const u32 xo = (wid >> 2) * 16384 + ((wid >> 1) & 1) * 4096 + l16, wo = 32768 + (wid & 1) * 4096 + l16;
    f32x4 acc[4][4];
#pragma unroll
    for (int m = 0; m < 4; ++m)
#pragma unroll
        for (int n = 0; n < 4; ++n) acc[m][n] = (f32x4){0.f, 0.f, 0.f, 0.f};
    u32 sb = 0;
    for (u32 u = 0; u < 32; ++u) {
        BARRIER();
        const LAS char* const xp = lds + sb + xo; const LAS char* const wp = lds + sb + wo;
        bf16x8 xf[2][4], wf[2][4];
#pragma unroll
        for (int k = 0; k < 2; ++k)
#pragma unroll
            for (int i = 0; i < 4; ++i) { wf[k][i] = *(const LAS bf16x8*)(wp + k * 8192 + i * 1024); xf[k][i] = *(const LAS bf16x8*)(xp + k * 8192 + i * 1024); }
        __builtin_amdgcn_sched_barrier(0);
#pragma unroll
        for (int k = 0; k < 2; ++k)
#pragma unroll
            for (int m = 0; m < 4; ++m)
#pragma unroll
                for (int n = 0; n < 4; ++n) acc[m][n] = __builtin_amdgcn_mfma_f32_16x16x32_bf16(wf[k][n], xf[k][m], acc[m][n], 0, 0, 0);
        __builtin_amdgcn_sched_barrier(0);
        sb = sb == 2 * STAGE ? 0 : sb + STAGE;
    }
    if (wid == 0) STAMP(3);
    const u32x4 dv = *(volatile LAS u32x4*)misc;
    const float pz = (dv.x | dv.y | dv.z | dv.w) ? __builtin_nanf("") : 0.f;
    const u32 id = (tj * 16 + tq) * 8 + wid;
    const auto rt = rsrc(myws + OFF_T + id * 8192, 8192);
#pragma unroll
    for (int m = 0; m < 4; ++m)
#pragma unroll
        for (int h = 0; h < 2; ++h) {
            u32x4 v;
            v.x = pack_bf16(acc[m][2 * h][0] + pz, acc[m][2 * h][1]);
            v.y = pack_bf16(acc[m][2 * h][2], acc[m][2 * h][3]);
            v.z = pack_bf16(acc[m][2 * h + 1][0], acc[m][2 * h + 1][1]);
            v.w = pack_bf16(acc[m][2 * h + 1][2], acc[m][2 * h + 1][3]);
            __builtin_amdgcn_raw_buffer_store_b128(v, rt, l16 + (m * 2 + h) * 1024, 0, SYS);
        }
    const u32 fo = OFF_RF + list * 512 + ((LDS_LD(misc + 4) & 15u) * 8 + wid) * 4;
    WAIT_VM(0);
    u32 todo = 3;
    if (w_dist(tq) == 0) {
        const u32 tp = 2 * tj + (wid >> 2), so = OFF_ST + ((((rank ^ x_dist(tp)) * 4 + x_panel(tp)) * 4 + w_panel(tq)) * 8 + (wid & 3) * 2) * 4;
        const u32 old = __builtin_amdgcn_raw_ptr_buffer_atomic_add_i32(1, rm, lane < 2 ? lane * 4 : OOR, so, AGT);
        (void)__builtin_amdgcn_raw_ptr_buffer_atomic_add_i32(2, rm, lane < 2 && !(old & 3u) ? lane * 4 : OOR, so, AGT);
        todo = (u32)__ballot(old & 3u) & 3u;
    }
    __builtin_amdgcn_raw_buffer_store_b32(call << 12 | todo << 10 | id, rsrc(P.ws[rank ^ w_dist(tq)], WS_NEED), l0, fo, SYS);
    if (wid == 0) STAMP(4);
}

static int knob(const char*, int d) { return d; }
static void make_schedule(Params& p) {
    const int ne = knob("NE", NE), up = knob("UP", UP), lead = knob("LEAD", LEAD), hw = knob("HW", HW), rnum = knob("RNUM", RNUM), wl = knob("WL", WL);
    p.pn = up | ne << 8 | knob("GX", GX) << 16;
    int n1 = 0;
    for (int t = -lead; t < 128; ++t) {
        const int wi = t + lead;
        if (t >= 0) { p.tab[1][n1++] = t; p.tab[1][n1++] = t | 1 << 7; }
        if (wi < 128 && (wi < 4 * up || wi >= 4 * (up + ne))) { p.tab[1][n1++] = wi | 2 << 7; p.tab[1][n1++] = wi | 3 << 7; }
    }
    int n0 = 0, nd = 0, r = 0;
    auto direct = [&](int e) { p.tab[0][n0++] = e; if (++nd >= hw) for (int k = 0; k < rnum && r < 16 * ne; ++k, ++r) p.tab[0][n0++] = (r >> 2) | (r & 3) << 7 | 1 << 9; };
    for (int t = -wl; t < 128; ++t) { if (t + wl < 128) direct(t + wl); if (t >= 0) direct(t | 1 << 7); }
    for (; r < 16 * ne; ++r) p.tab[0][n0++] = (r >> 2) | (r & 3) << 7 | 1 << 9;
    for (int i = n0; i < 512; ++i) p.tab[0][i] = 0;
    for (int i = n1; i < 512; ++i) p.tab[1][i] = 0;
    for (int i = 0; i < 16 * NPW; ++i) p.grp[i] = Group{};
    int g = 0, nl = 0, qid = 0;
    auto add = [&](int ng, u32 d, u32 ea, u32 eb, u32 slots, u32 tab) {
        for (int i = 0; i < ng; ++i) {
            const Group G = {{d | qid << 8 | (8 * ng) << 16 | (8 * i) << 24, ea | eb << 16, slots, tab}};
            if (d) p.grp[g++] = G; else { p.grp[(nl >> 1) * 16 + 14 + (nl & 1)] = G; ++nl; }
        }
        ++qid;
    };
    for (u32 dd = 1; dd <= 8; ++dd) {
        const u32 d = dd & 7;
        u32 xs[4], ws[4]; int nx = 0, nwv = 0;
        for (u32 s = 0; s < 16; ++s) { if (x_dist(s) == d) xs[nx++] = s; if (w_dist(s) == d) ws[nwv++] = 16 + s; }
        if (d == 0) for (u32 h = 0; h < 2; ++h) add(knob("GL0", 2), 0, 0, 256, 0, (2 * h) << 4);
        else if (d == 2) add(2, d, 512, 512, xs[0] | xs[1] << 8 | xs[2] << 16 | xs[3] << 24, knob("D2F", D2F) << 8);
        else if (d & 1) add(2, d, n1, n1 + 256, xs[0] | xs[1] << 8 | ws[0] << 16 | ws[1] << 24, 1 | d << 4);
        else add(2, d, n0, n0 + 256, ws[0] | ws[1] << 8, 2 | d << 4);
    }
}

}

extern "C" void kernel_launch(int rank, int nranks, ncclComm_t comm,
                              hipStream_t stream,
                              void* const* d_in, const int* in_sizes, int n_in,
                              void* d_out, int out_size,
                              const int* in_dtypes, int out_dtype,
                              void* const* d_out_all,
                              void* const* d_ws_all, size_t ws_bytes) {
    if (nranks != NR || (unsigned)rank >= (unsigned)NR || ws_bytes < WS_NEED || n_in < 2 ||
        in_sizes[0] != 512 * 4096 || in_sizes[1] != 4096 * 512 || out_size != 4096 * 512) {
        fprintf(stderr, "unexpected problem shape\n");
        return;
    }
    Params p;
    p.x = (const char*)d_in[0];
    p.w = (const char*)d_in[1];
    p.out = (char*)d_out;
    for (int i = 0; i < NR; ++i) p.ws[i] = (char*)d_ws_all[i];
    p.rank = rank;
    make_schedule(p);
    hipLaunchKernelGGL(ag_gemm_kernel, dim3(256), dim3(768), 0, stream, p);
    HIP_CHECK(hipGetLastError());
}
```

```cpp
#include <hip/hip_runtime.h>
#include <rccl/rccl.h>
#include <cstdio>
#include <cstdlib>

#define HIP_CHECK(expr) do { hipError_t _e = (expr); if (_e != hipSuccess) { \
    fprintf(stderr, "HIP error %s at %s:%d\n", hipGetErrorString(_e), __FILE__, __LINE__); \
    return; } } while (0)

namespace {

typedef unsigned u32;
typedef unsigned long long u64;
typedef u32 u32x4 __attribute__((ext_vector_type(4)));
typedef u32 u32x2 __attribute__((ext_vector_type(2)));
typedef float f32x4 __attribute__((ext_vector_type(4)));
typedef float f32x2 __attribute__((ext_vector_type(2)));
typedef short s16x4 __attribute__((ext_vector_type(4)));
typedef __bf16 bf16x8 __attribute__((ext_vector_type(8)));
typedef __bf16 bf16x2 __attribute__((ext_vector_type(2)));

constexpr int NR = 8;
constexpr u32 PANEL = 512u << 10;
constexpr u32 OFF_S = 0;
constexpr u32 OFF_T = 32u << 20;
constexpr u32 OFF_P = 40u << 20;
constexpr u32 OFF_AF = 44u << 20;
constexpr u32 OFF_RF = OFF_AF + 16384;
constexpr u32 OFF_ST = OFF_RF + 4096;
constexpr u32 OFF_CTR = OFF_ST + 4096;
constexpr u32 OFF_TMO = OFF_CTR + 16384;
constexpr u32 OFF_Q = OFF_TMO + 1024;
constexpr u32 OFF_LT = OFF_Q + 4096;
constexpr u32 OFF_DBG = OFF_LT + 1024;
constexpr u32 WS_NEED = OFF_DBG + (1024u << 10);
constexpr u64 DEADLINE = 100000000ull;
constexpr int SYS = 17;
constexpr int AGT = 16;
constexpr u32 OOR = 0x80000000u;

#define NE 9
#define UP 23
#define LEAD 48
#define HW 192
#define RNUM 4
#define XL 16
#define WL 24
#define D2F 128
#define WS 3
#define WP 3
#define RING 2
#define RINGT 4
#define RINGB 2
#define GX 14
#define NPW 4

struct Group { u32 w[4]; };
struct Params {
    const char* x;
    const char* w;
    char* out;
    char* ws[NR];
    int rank; u32 pn;
    Group grp[16 * NPW];
    unsigned short tab[2][512];
};

__host__ __device__ __forceinline__ u32 x_dist(u32 p) { return (0x7520'3210u >> (4 * (p >> 1))) & 7u; }
__host__ __device__ __forceinline__ u32 x_panel(u32 p) { return ((p >> 3) << 1) | (p & 1u); }
__host__ __device__ __forceinline__ u32 w_dist(u32 q) { return (u32)(0x6530'7610'7430'5410ull >> (4 * q)) & 7u; }
__host__ __device__ __forceinline__ u32 w_panel(u32 q) { return q >> 2; }
__host__ __device__ __forceinline__ u32 x_pair(u32 i) { return (u32)(0xFDEC'BA98'7354'6210ull >> (4 * i)) & 15u; }

__device__ __forceinline__ __amdgpu_buffer_rsrc_t rsrc(const void* p, u32 bytes) {
    return __builtin_amdgcn_make_buffer_rsrc(const_cast<void*>(p), 0, bytes, 0x00020000);
}
__device__ __forceinline__ u32 pack_bf16(float a, float b) {
    return __builtin_bit_cast(u32, __builtin_convertvector((f32x2){a, b}, bf16x2));
}
__device__ __forceinline__ float gelu(float y) {
    const float t = y * (-2.302208198f - 0.1029432397f * y * y);
    return y * __builtin_amdgcn_rcpf(1.0f + __builtin_amdgcn_exp2f(t));
}
#define WAIT_VM(n) asm volatile("s_waitcnt vmcnt(%0)" :: "i"(n) : "memory")
#define CBAR() asm volatile("" ::: "memory")
#define RFL(x) ((u32)__builtin_amdgcn_readfirstlane(x))
#define INL __attribute__((always_inline))
#define LAS __attribute__((address_space(3)))
#define BARRIER() do { asm volatile("s_waitcnt lgkmcnt(0)" ::: "memory"); __builtin_amdgcn_s_barrier(); CBAR(); } while (0)
#define LDS_LD(p) RFL(*(volatile LAS u32*)(p))
#define LDS_ST(p, v) do { *(volatile LAS u32*)(p) = (v); } while (0)
#define DMA16(r, d, v, o) __builtin_amdgcn_raw_ptr_buffer_load_lds(r, d, 16, v, o, 0, SYS)

#define STAMP(i) do {} while (0)

constexpr u32 STAGE = 49152;
constexpr u32 LDS_POLL = 3 * STAGE;
constexpr u32 LDS_MISC = LDS_POLL + 1024;
constexpr u32 LDS_BYTES = LDS_MISC + 64;
enum { K_COPY, K_W, K_RED, K_SIG, K_END };

__global__ void __launch_bounds__(768, 3) ag_gemm_kernel(const Params P) {
    __shared__ __attribute__((aligned(16))) char smem[LDS_BYTES];
    LAS char* const lds = (LAS char*)smem;
    const u32 wid = RFL(threadIdx.x >> 6);
    const u32 lane = threadIdx.x & 63;
    const u32 rank = P.rank & 7;
    char* const myws = P.ws[rank];
    u32* const ctr = (u32*)(myws + OFF_CTR) + blockIdx.x * 16 + wid;
    const u32 ctrv = __hip_atomic_load(ctr, __ATOMIC_RELAXED, __HIP_MEMORY_SCOPE_AGENT);
    u32* const tmo = (u32*)(myws + OFF_TMO);
    const u32 l0 = lane ? OOR : 0u, l16 = lane * 16;
    const auto rm = rsrc(myws, WS_NEED);
    const u32 vx = (lane & 15) * 8192 + (lane >> 4) * 16, vw = (lane >> 3) * 1024 + (lane & 7) * 16;
    const u32 tro = ((lane >> 4) * 8 + ((lane >> 2) & 3)) * 128 + (lane & 3) * 32;

    if (blockIdx.x < 128) {
        const u32 g = blockIdx.x, multi = g < 112, pair = multi && wid < 8 ? 0u : wid & 3;
        const Group G = P.grp[pair * 16 + (g >> 3)];
        const u32 dist = G.w[0] & 7u, nw = (G.w[0] >> 16) & 255u, ea = G.w[1] & 0xffffu, eb = G.w[1] >> 16;
        const u32 peer = rank ^ dist, qo = OFF_Q + ((G.w[0] >> 8) & 15u) * 256;
        const auto rd = rsrc(P.ws[peer], WS_NEED);
        LAS char* const ring = lds + pair * 16384;
        LAS u32* const rdesc = (LAS u32*)(lds + 131072 + pair * 128);
        const u32 po = (lane & 1) * 1024 + ((lane >> 1) & 3) * 256 + (lane >> 3) * 16;
        bool dead = false;
        if (wid >= 8) {
            if (lane < 16) rdesc[16 + lane] = 0;
            const u32 up = P.pn & 255u, une = (P.pn >> 8) & 255u, df = G.w[3] >> 8;
            const u32 rsl = dist == 4 ? (17u | 19u << 8 | 21u << 16 | 23u << 24) : (25u | 27u << 8 | 29u << 16 | 30u << 24);
            const u32 rdl = dist == 4 ? (19u | 17u << 8 | 23u << 16 | 21u << 24) : (27u | 25u << 8 | 30u << 16 | 29u << 24);
            const u32* const tab = (const u32*)P.tab[G.w[3] & 1];
            const u32 urev = (peer & 4) ? 31u : 0u;
            const char* const xs = P.x + (peer & 1) * 4096;
            const char* const wsrc = P.w + (size_t)(peer & 1) * (2u << 20);
            const u32 xbytes = (4u << 20) - (peer & 1) * 4096;
            u32x4 dv = {0u, 0u, 0u, 0u};
            u32 cur = (G.w[0] >> 24) + (g & 7), nxt = cur + nw, c = 0, j = 0, pub = 0, used = 0, call = 0, dec = 0, sbase = 0, depth = RING;
            u64 t0 = 0;
            auto timed_out = [&]() INL { return __builtin_amdgcn_s_memrealtime() - t0 > DEADLINE; };
            auto publish = [&](u32 n) INL { if (n != pub) { pub = n; LDS_ST(rdesc + 16, n); } };
            auto slot_wait = [&]() INL {
                if (used + depth > j) return;
                used = LDS_LD(rdesc + 17);
                if (used + depth > j) return;
                WAIT_VM(0); publish(j);
                while (used + depth <= j) { __builtin_amdgcn_s_sleep(1); used = LDS_LD(rdesc + 17); }
            };
            auto word = [&](u32 w) INL -> u32 {
                const u32 l = w >> 2;
                const u32 v0 = __builtin_amdgcn_readlane(dv.x, l), v1 = __builtin_amdgcn_readlane(dv.y, l), v2 = __builtin_amdgcn_readlane(dv.z, l), v3 = __builtin_amdgcn_readlane(dv.w, l);
                return (w & 2) ? ((w & 1) ? v3 : v2) : ((w & 1) ? v1 : v0);
            };
            auto dep_wait = [&](u32 w, u32 sh, u32 want, u32 dvo) INL {
                if ((word(w) >> sh) >= want) return;
                WAIT_VM(0); publish(j);
                while (!dead) {
                    dv = __builtin_amdgcn_raw_buffer_load_b128(rm, dvo, 0, SYS);
                    u32 t = RFL(__hip_atomic_load(tmo, __ATOMIC_RELAXED, __HIP_MEMORY_SCOPE_AGENT));
                    asm volatile("" : "+s"(t));
                    if ((word(w) >> sh) >= want) break;
                    if (t == call || timed_out()) dead = true;
                    __builtin_amdgcn_s_sleep(1);
                }
                CBAR();
            };
            auto post = [&](u32 kind) INL { slot_wait(); LDS_ST(rdesc + (j & 3) * 4 + 3, kind); WAIT_VM(0); ++j; publish(j); };
            const u32 dvr = OFF_AF + ((rsl >> (8 * (lane >> 4))) & 31u) * 512 + (lane & 15) * 16;
#define PUSH_STEP(first) do { \
                const u32 e = (G.w[3] & 3) ? (tab[cur >> 1] >> ((cur & 1) * 16)) & 1023u \
                            : cur < 2 * df ? (cur >> 1) | (cur & 1) << 7 : cur >= 512 - 2 * df ? ((cur >> 1) - 128) | (2 + (cur & 1)) << 7 \
                            : (((cur - 2 * df) >> 2) + ((cur - 2 * df) & 2 ? 0u : df)) | ((cur - 2 * df) & 3) << 7; \
                const u32 i = e & 127u, k = (e >> 7) & 3u; \
                u32 kind = K_COPY, so, s1 = 1024, s2 = 2048, pc, d2 = 2048, vo = l16, src = 2; \
                if (e >> 9) { \
                    dep_wait(k * 64 + i, 0, call, dvr); \
                    so = sbase + ((rsl >> (8 * k)) & 31u) * PANEL + i * 4096; \
                    pc = ((rdl >> (8 * k)) & 31u) * 128 + i + 4 * up; \
                } else { \
                    const u32 v = i >> 2, slot = (G.w[2] >> (8 * k)) & 31u, u = (v < up ? v : v < up + une ? 31 + up - v : v - une) ^ urev, h = i & 3; \
                    pc = slot * 128 + i; \
                    if (slot >= 16) { kind = K_W; src = 1; vo = vw; s1 = 8192; s2 = 16384; so = (u * 64 + (h >> 1) * 32) * 1024 + w_panel(slot & 15) * 256 + (h & 1) * 128; } \
                    else { src = 0; vo = vx; s1 = 131072; s2 = 64; d2 = 8192; so = (x_panel(slot) * 128 + h * 32) * 8192 + u * 128; } \
                } \
                slot_wait(); \
                const auto rs = rsrc(src == 2 ? myws + 0 : src ? wsrc + 0 : xs + 0, src == 2 ? WS_NEED : src ? (2u << 20) : xbytes); \
                LAS char* const sl = ring + (j & 3) * 4096; \
                if (first && (!nw || !ea)) so = OOR; \
                DMA16(rs, sl, vo, so); DMA16(rs, sl + 1024, vo, so + s1); DMA16(rs, sl + 2048, vo, so + s2); DMA16(rs, sl + 3072, vo, so + s1 + s2); \
                *(volatile LAS u32x4*)(rdesc + (j & 3) * 4) = (u32x4){pc, d2, 0u, kind}; \
                if (first) { \
                    BARRIER(); \
                    asm volatile("s_waitcnt vmcnt(4)\n\tv_readfirstlane_b32 %0, %1" : "=s"(call) : "v"(ctrv) : "memory"); \
                    ++call; \
                    if (lane == 0) __hip_atomic_store(ctr, call, __ATOMIC_RELAXED, __HIP_MEMORY_SCOPE_AGENT); \
                    t0 = __builtin_amdgcn_s_memrealtime(); \
                    sbase = OFF_S + (call & 1u) * 32 * PANEL; \
                    dec = 3u * nw - (call - 1u) * eb; \
                    c = cur + 2 * nw - dec; \
                    if (!nw || !ea) break; \
                } else { \
                    WAIT_VM(4); \
                    publish(j); \
                } \
                cur = nxt; nxt = RFL(c) + dec; \
                c = __builtin_amdgcn_raw_ptr_buffer_atomic_add_i32(1, rm, l0, qo, AGT); \
                CBAR(); \
                ++j; \
             \
            } while (0)
            PUSH_STEP(true);
            if (!nw) return;
            while (cur < ea) PUSH_STEP(false);
            depth = 4;
            if (dist && eb > ea) post(K_SIG);
            const u32 jb = j;
            LDS_ST(rdesc + 18, j + 1);
            if (eb > ea) {
                const u32 lo = OFF_RF + ((G.w[3] >> 4) & 7u) * 512;
                u32 wv = __builtin_amdgcn_raw_buffer_load_b32(rm, 0, lo + (((cur - ea) >> 1) & 127u) * 4, SYS);
                asm volatile("" : "+v"(wv));
                while (dist && !dead) {
                    const u32 v = RFL(__builtin_amdgcn_raw_buffer_load_b32(rm, 0, qo + 128, SYS));
                    CBAR();
                    if (v + ((P.pn >> 16) & 255u) >= call * nw) break;
                    if (timed_out()) dead = true;
                    __builtin_amdgcn_s_sleep(1);
                }
                STAMP(pair * 10 + 7);
                while (cur < eb) {
                    const u32 r = cur - ea;
                    u32 en = RFL(wv);
                    if ((en >> 12) < call) {
                        WAIT_VM(0); publish(j);
                        while (!dead) {
                            const u32 w = __builtin_amdgcn_raw_buffer_load_b32(rm, lane & 1 ? OFF_TMO : lo + (r >> 1) * 4, 0, SYS);
                            en = RFL(w);
                            if ((en >> 12) >= call) break;
                            if ((u32)__builtin_amdgcn_readlane(w, 1) == call || timed_out()) dead = true;
                            __builtin_amdgcn_s_sleep(1);
                        }
                        CBAR();
                    }
                    const u32 id = en & 1023u, tp = x_pair(2 * (id >> 7) + ((id >> 2) & 1));
                    const u32 so = (en >> (10 + (r & 1))) & 1 ? OFF_T + id * 8192 + (r & 1) * 4096 : OOR;
                    const u32 unit = (((peer ^ x_dist(tp)) * 4 + x_panel(tp)) * 4 + w_panel((id >> 3) & 15u)) * 8 + (id & 3) * 2 + (r & 1);
                    if (j < jb + 4) slot_wait();
                    else for (;;) {
                        const u32x2 fr = *(volatile LAS u32x2*)(rdesc + 24 + (j & 3) * 2);
                        if (min(RFL(fr.x), RFL(fr.y)) + 3 >= j) break;
                        WAIT_VM(0); publish(j);
                        __builtin_amdgcn_s_sleep(1);
                    }
                    LAS char* const sl = ring + (j & 3) * 4096;
                    wv = __builtin_amdgcn_raw_buffer_load_b32(rm, 0, lo + (((nxt - ea) >> 1) & 127u) * 4, SYS);
                    const u32 tk = __builtin_amdgcn_raw_ptr_buffer_atomic_add_i32(dist ? 1 : 0, rm, dead ? OOR : l0, OFF_ST + unit * 4, AGT);
                    WAIT_VM(WP + 2); DMA16(rd, sl, l16, so);
                    WAIT_VM(WP + 2); DMA16(rd, sl + 1024, l16 + 1024, so);
                    WAIT_VM(WP + 2); DMA16(rd, sl + 2048, l16 + 2048, so);
                    WAIT_VM(WP + 2); DMA16(rd, sl + 3072, l16 + 3072, so);
                    WAIT_VM(4);
                    publish(j);
                    const u32 tv = RFL(tk) & 3u, t = (unit >> 6) & 1, cp = (unit >> 3) & 3, ob = peer ^ (t << 2 | ((cp ^ ((cp >> 1) & t)) & 1) << 1 | 1);
                    const u32 st = dead ? 2u : so == OOR ? 4u : dist && !tv ? 0u : ob == rank ? 1u : (dist ? tv == 3 : !tv) ? 3u : 5u;
                    u32 tp2 = t * 8 + ((unit >> 5) & 1), tq2 = cp * 4;
                    if (st == 1) {
                        while (x_dist(tp2) != (ob ^ (unit >> 7))) tp2 += 2;
                        while (w_dist(tq2) != (ob ^ rank)) ++tq2;
                    }
                    tp2 = x_pair(tp2);
                    *(volatile LAS u32x4*)(rdesc + (j & 3) * 4) = (u32x4){unit, st, OFF_T + ((tp2 >> 1) * 16 + tq2) * 65536 + (tp2 & 1) * 32768 + (so & 0x7000u), K_RED};
                    LDS_ST(rdesc + 19, j + 1);
                    cur = nxt; nxt = RFL(c) + dec;
                    c = __builtin_amdgcn_raw_ptr_buffer_atomic_add_i32(1, rm, l0, qo, AGT);
                    CBAR();
                    ++j;
                }
            }
            WAIT_VM(0); publish(j);
            LDS_ST(rdesc + 20, j + 1);
            if (dead && lane == 0) {
                __hip_atomic_store(tmo, call, __ATOMIC_RELAXED, __HIP_MEMORY_SCOPE_AGENT);
                *(volatile u32*)(P.out + (size_t)g * 32768 + pair * 4) = 0x7fc07fc0u;
            }
            return;
        }
        BARRIER();
        const u32 call = RFL(ctrv) + 1u;
        if (lane == 0) __hip_atomic_store(ctr, call, __ATOMIC_RELAXED, __HIP_MEMORY_SCOPE_AGENT);
        if (!nw) return;
        const u32 half = wid >> 2;
        u32 k = 0, full = 0;
        if (multi ? wid == 0 : !half) {
            STAMP(pair * 10);
            const u32 sbase = OFF_S + (call & 1u) * 32 * PANEL + l16;
            u32 pend = OOR;
#define FLAG() do { __builtin_amdgcn_raw_buffer_store_b32(call, rd, pend, 0, SYS); CBAR(); pend = OOR; } while (0)
            for (;;) {
                if (full <= k) {
                    full = LDS_LD(rdesc + 16);
                    if (full <= k) {
                        if (RFL(pend) != OOR) { WAIT_VM(0); FLAG(); }
                        while (full <= k && LDS_LD(rdesc + 18) != k + 1) { __builtin_amdgcn_s_sleep(1); full = LDS_LD(rdesc + 16); }
                    }
                    CBAR();
                }
                if (LDS_LD(rdesc + 18) == k + 1) { WAIT_VM(0); FLAG(); break; }
                const u32x4 dd = *(volatile LAS u32x4*)(rdesc + (k & 3) * 4);
                const u32 d0 = RFL(dd.x), d1 = RFL(dd.y), kind = RFL(dd.w);
                if (kind == K_SIG) {
                    (void)__builtin_amdgcn_raw_ptr_buffer_atomic_add_i32(1, rd, l0, qo + 128, AGT);
                    ++k; LDS_ST(rdesc + 17, k);
                    continue;
                }
                const LAS char* const sl = ring + (k & 3) * 4096;
                u32x4 f[4];
                if (kind == K_W) {
#pragma unroll
                    for (int n = 0; n < 4; ++n) {
                        const u32x2 lo = __builtin_bit_cast(u32x2, __builtin_amdgcn_ds_read_tr16_b64_v4i16((LAS s16x4*)(sl + tro + n * 8)));
                        const u32x2 hi = __builtin_bit_cast(u32x2, __builtin_amdgcn_ds_read_tr16_b64_v4i16((LAS s16x4*)(sl + tro + 512 + n * 8)));
                        f[n] = (u32x4){lo.x, lo.y, hi.x, hi.y};
                    }
                } else {
#pragma unroll
                    for (int n = 0; n < 4; ++n) f[n] = *(const LAS u32x4*)(sl + l16 + n * 1024);
                }
                asm volatile("s_waitcnt lgkmcnt(0)" : "+v"(f[0]), "+v"(f[1]), "+v"(f[2]), "+v"(f[3]) :: "memory");
                ++k; LDS_ST(rdesc + 17, k);
                const u32 vo0 = sbase + (d0 >> 2) * 16384 + (d0 & 3u) * (d1 == 8192 ? 2048u : 4096u), vo1 = vo0 + d1;
#define WSTEP(i) do { WAIT_VM(WS - 1); if (i == WS - 1) FLAG(); } while (0)
                WSTEP(0); __builtin_amdgcn_raw_buffer_store_b128(f[0], rd, vo0, 0, SYS);
                WSTEP(1); __builtin_amdgcn_raw_buffer_store_b128(f[1], rd, vo0 + 1024, 0, SYS);
                WSTEP(2); __builtin_amdgcn_raw_buffer_store_b128(f[2], rd, vo1, 0, SYS);
                WSTEP(3); __builtin_amdgcn_raw_buffer_store_b128(f[3], rd, vo1 + 1024, 0, SYS);
                if (WS > 4) { WAIT_VM(4); FLAG(); }
                pend = l0 | (OFF_AF + d0 * 4);
            }
            STAMP(pair * 10 + 1);
        }
        while (!(k = LDS_LD(rdesc + 18))) __builtin_amdgcn_s_sleep(8);
        --k;
        if (multi) k += (wid - k) & 3u;
        const auto ro = rsrc(P.out, 4u << 20);
        for (;; k += multi ? 4u : 1u) {
            bool more;
            while (!(more = LDS_LD(rdesc + 19) > k) && LDS_LD(rdesc + 20) - 1 > k) __builtin_amdgcn_s_sleep(1);
            if (!more && LDS_LD(rdesc + 19) <= k) break;
            CBAR();
            const u32x4 dd = *(volatile LAS u32x4*)(rdesc + (k & 3) * 4);
            const u32 d0 = RFL(dd.x), d1 = RFL(dd.y);
            u32x4 f[2], o[2];
            const u32 v2 = d1 == 1 ? po + half * 2048 : l16 + half * 2048, o2 = d1 == 1 ? RFL(dd.z) : d1 == 3 ? OFF_P + d0 * 4096 : OOR;
            o[0] = __builtin_amdgcn_raw_buffer_load_b128(rm, v2, o2, SYS);
            o[1] = __builtin_amdgcn_raw_buffer_load_b128(rm, v2 + (d1 == 1 ? 128u : 1024u), o2, SYS);
            while (full <= k) { full = LDS_LD(rdesc + 16); if (full <= k) __builtin_amdgcn_s_sleep(1); }
            CBAR();
            const LAS char* const sl = ring + (k & 3) * 4096 + half * 2048 + po;
            f[0] = *(const LAS u32x4*)sl; f[1] = *(const LAS u32x4*)(sl + 128);
            asm volatile("s_waitcnt lgkmcnt(0)" : "+v"(f[0]), "+v"(f[1]) :: "memory");
            LDS_ST(rdesc + 24 + (k & 3) * 2 + half, k + 1);
            if (d1 == 4) continue;
            if (d1 == 0) {
                const u32 pp = OFF_P + d0 * 4096 + half * 2048 + l16;
                __builtin_amdgcn_raw_buffer_store_b128(f[0], rm, pp, 0, SYS); __builtin_amdgcn_raw_buffer_store_b128(f[1], rm, pp + 1024, 0, SYS);
                WAIT_VM(0);
                (void)__builtin_amdgcn_raw_ptr_buffer_atomic_add_i32(1, rm, l0, OFF_ST + d0 * 4, AGT);
                CBAR();
                continue;
            }
            bool bad = d1 == 2;
            if (d1 == 5) {
                const u64 t0 = __builtin_amdgcn_s_memrealtime();
                while (RFL(__builtin_amdgcn_raw_buffer_load_b32(rm, 0, OFF_ST + d0 * 4, AGT)) & 3u) {
                    CBAR();
                    if (__builtin_amdgcn_s_memrealtime() - t0 > DEADLINE) { bad = true; break; }
                    __builtin_amdgcn_s_sleep(2);
                }
                CBAR();
                o[0] = __builtin_amdgcn_raw_buffer_load_b128(rm, v2, OFF_P + d0 * 4096, SYS);
                o[1] = __builtin_amdgcn_raw_buffer_load_b128(rm, v2 + 1024, OFF_P + d0 * 4096, SYS);
            }
            const float pz = bad ? __builtin_nanf("") : 0.f;
            const u32 oo = ((d0 >> 5) * 128 + ((d0 >> 2) & 1) * 64 + (d0 & 1) * 32 + half * 16 + (lane >> 3)) * 1024 + ((d0 >> 3) & 3) * 256 + ((d0 >> 1) & 1) * 128 + (lane & 7) * 16;
#pragma unroll
            for (int n = 0; n < 2; ++n) {
                u32x4 r;
#pragma unroll
                for (int q = 0; q < 4; ++q) {
                    const float lo = __builtin_bit_cast(float, f[n][q] << 16) + __builtin_bit_cast(float, o[n][q] << 16) + pz;
                    const float hi = __builtin_bit_cast(float, f[n][q] & 0xffff0000u) + __builtin_bit_cast(float, o[n][q] & 0xffff0000u) + pz;
                    r[q] = pack_bf16(gelu(lo), gelu(hi));
                }
                __builtin_amdgcn_raw_buffer_store_b128(r, ro, oo + n * 8192, 0, AGT);
            }
        }
        if (wid < 4) STAMP(wid * 10 + 4);
        return;
    }

    const u32 gg = blockIdx.x - 128;
    const u32 tj = ((gg & 7) >> 1) * 2 + (gg >> 6), tq = (gg & 1) * 8 + ((gg >> 3) & 7);
    const u32 list = w_dist(tq) ? w_dist(tq) : (tq >> 3) * 2;
    LAS u32* const misc = (LAS u32*)(lds + LDS_MISC);
    if (wid >= 8) {
        const u32 l = wid - 8;
        const u32 call = RFL(ctrv) + 1u;
        if (lane == 0) __hip_atomic_store(ctr, call, __ATOMIC_RELAXED, __HIP_MEMORY_SCOPE_AGENT);
        bool dead = false;
        const u64 t0 = __builtin_amdgcn_s_memrealtime();
        const u32 xb = OFF_S + ((call & 1u) * 32 + x_pair(2 * tj)) * PANEL, xd = (x_pair(2 * tj + 1) - x_pair(2 * tj)) * PANEL, wb = OFF_S + ((call & 1u) * 32 + 16 + tq) * PANEL;
        const u32 gx = l * 2048 + l16, gw = l * 4096 + l16;
        const u32 fsl = lane >> 4;
        const u32 fo = fsl == 3 ? OFF_TMO : OFF_AF + (fsl == 2 ? 16 + tq : x_pair(2 * tj + fsl)) * 512 + (lane & 15) * 16 + l * 4;
        const u32 fstep = fsl == 3 ? 0u : 16u;
        LAS u32* const pollbuf = (LAS u32*)(lds + LDS_POLL + l * 256);
        u32 avail = 0, pbase = 0;
#define POLL() do { pbase = avail; __builtin_amdgcn_raw_ptr_buffer_load_lds(rm, pollbuf, 4, fo + min(avail, 31u) * fstep, 0, 0, SYS); } while (0)
#define POLL_READ() do { const u32 _v = *(volatile LAS u32*)(pollbuf + lane); \
        const u64 _m = ~__ballot(_v >= call || pbase + (lane & 15) >= 32); \
        const u32 _a = pbase + __builtin_ctz((((u32)_m | (u32)(_m >> 16) | (u32)(_m >> 32)) & 0xffffu) | 0x10000u); \
        avail = max(avail, _a); if (__ballot(_v == call) >> 48) { dead = true; avail = 64; } } while (0)
#define SPIN(n) do { while (avail < (u32)(n)) { POLL(); WAIT_VM(0); POLL_READ(); \
        if (avail < (u32)(n)) { if (__builtin_amdgcn_s_memrealtime() - t0 > DEADLINE) { dead = true; avail = 64; } else __builtin_amdgcn_s_sleep(1); } } CBAR(); } while (0)
#define STAGE_IN(u, sb) do { LAS char* _x = lds + (sb) + l * 2048; LAS char* _w = lds + (sb) + 32768 + l * 4096; const u32 _x0 = xb + (u) * 16384, _w0 = wb + (u) * 16384; \
        DMA16(rm, _x, gx, _x0); DMA16(rm, _x + 1024, gx + 1024, _x0); DMA16(rm, _x + 16384, gx, _x0 + xd); DMA16(rm, _x + 17408, gx + 1024, _x0 + xd); \
        DMA16(rm, _w, gw, _w0); DMA16(rm, _w + 1024, gw + 1024, _w0); DMA16(rm, _w + 2048, gw + 2048, _w0); DMA16(rm, _w + 3072, gw + 3072, _w0); \
        DMA16(rm, _x + 8192, gx, _x0 + 8192); DMA16(rm, _x + 9216, gx + 1024, _x0 + 8192); DMA16(rm, _x + 24576, gx, _x0 + xd + 8192); DMA16(rm, _x + 25600, gx + 1024, _x0 + xd + 8192); CBAR(); } while (0)
        if (lane == 0) misc[l] = 0;
        SPIN(1); STAGE_IN(0, 0);
        SPIN(2); POLL(); STAGE_IN(1, STAGE);
        u32 sn = 2 * STAGE;
        for (u32 u = 0; u < 30; ++u) {
            WAIT_VM(12); BARRIER();
            POLL_READ();
            SPIN(u + 3); POLL(); STAGE_IN(u + 2, sn);
            sn = sn == 2 * STAGE ? 0 : sn + STAGE;
        }
        if (l == 0) STAMP(2);
        if (dead && lane == 0) { misc[l] = 1; __hip_atomic_store(tmo, call, __ATOMIC_RELAXED, __HIP_MEMORY_SCOPE_AGENT); }
        WAIT_VM(12); BARRIER();
        if (l == 0 && lane == 0)
            misc[4] = __hip_atomic_fetch_add((u32*)(myws + OFF_LT + list * 128), 1u, __ATOMIC_RELAXED, __HIP_MEMORY_SCOPE_AGENT);
        WAIT_VM(0); BARRIER();
        return;
    }
    u32 call;
    {
        const u32 h = wid & 3, isw = wid >> 2, v = gg & 31u, pn = gg >> 5, slot = isw ? 16 + pn * 4 : (pn >> 1) * 8 + (pn & 1);
        const u32 up = P.pn & 255u, une = (P.pn >> 8) & 255u, u = (v < up ? v : v < up + une ? 31 + up - v : v - une) ^ ((rank & 4) ? 31u : 0u);
        LAS char* const sc = lds + 2 * STAGE + wid * 4096;
        const auto rs = rsrc(isw ? P.w : P.x, 4u << 20);
        const u32 so = isw ? ((rank & 1) * 2048 + u * 64 + (h >> 1) * 32) * 1024 + pn * 256 + (h & 1) * 128 : (pn * 128 + h * 32) * 8192 + (rank & 1) * 4096 + u * 128;
        const u32 s1 = isw ? 8192u : 131072u, s2 = isw ? 16384u : 64u, vo = isw ? vw : vx;
        DMA16(rs, sc, vo, so); DMA16(rs, sc + 1024, vo, so + s1); DMA16(rs, sc + 2048, vo, so + s2); DMA16(rs, sc + 3072, vo, so + s1 + s2);
        CBAR();
        call = RFL(ctrv) + 1u;
        if (lane == 0) __hip_atomic_store(ctr, call, __ATOMIC_RELAXED, __HIP_MEMORY_SCOPE_AGENT);
        WAIT_VM(1);
        u32x4 f[4];
        if (isw) {
#pragma unroll
            for (int n = 0; n < 4; ++n) {
                const u32x2 lo = __builtin_bit_cast(u32x2, __builtin_amdgcn_ds_read_tr16_b64_v4i16((LAS s16x4*)(sc + tro + n * 8)));
                const u32x2 hi = __builtin_bit_cast(u32x2, __builtin_amdgcn_ds_read_tr16_b64_v4i16((LAS s16x4*)(sc + tro + 512 + n * 8)));
                f[n] = (u32x4){lo.x, lo.y, hi.x, hi.y};
            }
        } else {
#pragma unroll
            for (int n = 0; n < 4; ++n) f[n] = *(const LAS u32x4*)(sc + l16 + n * 1024);
        }
        const u32 vo0 = OFF_S + ((call & 1u) * 32 + slot) * PANEL + v * 16384 + h * (isw ? 4096u : 2048u) + l16, vo1 = vo0 + (isw ? 2048u : 8192u);
        __builtin_amdgcn_raw_buffer_store_b128(f[0], rm, vo0, 0, SYS); __builtin_amdgcn_raw_buffer_store_b128(f[1], rm, vo0 + 1024, 0, SYS);
        __builtin_amdgcn_raw_buffer_store_b128(f[2], rm, vo1, 0, SYS); __builtin_amdgcn_raw_buffer_store_b128(f[3], rm, vo1 + 1024, 0, SYS);
        WAIT_VM(0);
        __builtin_amdgcn_raw_buffer_store_b32(call, rm, l0, OFF_AF + (slot * 128 + v * 4 + h) * 4, SYS);
        if (wid == 0) STAMP(6);
    }
    const u32 xo = (wid >> 2) * 16384 + ((wid >> 1) & 1) * 4096 + l16, wo = 32768 + (wid & 1) * 4096 + l16;
    f32x4 acc[4][4];
#pragma unroll
    for (int m = 0; m < 4; ++m)
#pragma unroll
        for (int n = 0; n < 4; ++n) acc[m][n] = (f32x4){0.f, 0.f, 0.f, 0.f};
    u32 sb = 0;
    for (u32 u = 0; u < 32; ++u) {
        BARRIER();
        const LAS char* const xp = lds + sb + xo; const LAS char* const wp = lds + sb + wo;
        bf16x8 xf[2][4], wf[2][4];
#pragma unroll
        for (int k = 0; k < 2; ++k)
#pragma unroll
            for (int i = 0; i < 4; ++i) { wf[k][i] = *(const LAS bf16x8*)(wp + k * 8192 + i * 1024); xf[k][i] = *(const LAS bf16x8*)(xp + k * 8192 + i * 1024); }
        __builtin_amdgcn_sched_barrier(0);
#pragma unroll
        for (int k = 0; k < 2; ++k)
#pragma unroll
            for (int m = 0; m < 4; ++m)
#pragma unroll
                for (int n = 0; n < 4; ++n) acc[m][n] = __builtin_amdgcn_mfma_f32_16x16x32_bf16(wf[k][n], xf[k][m], acc[m][n], 0, 0, 0);
        __builtin_amdgcn_sched_barrier(0);
        sb = sb == 2 * STAGE ? 0 : sb + STAGE;
    }
    if (wid == 0) STAMP(3);
    const u32x4 dv = *(volatile LAS u32x4*)misc;
    const float pz = (dv.x | dv.y | dv.z | dv.w) ? __builtin_nanf("") : 0.f;
    const u32 id = (tj * 16 + tq) * 8 + wid;
    const auto rt = rsrc(myws + OFF_T + id * 8192, 8192);
#pragma unroll
    for (int m = 0; m < 4; ++m)
#pragma unroll
        for (int h = 0; h < 2; ++h) {
            u32x4 v;
            v.x = pack_bf16(acc[m][2 * h][0] + pz, acc[m][2 * h][1]);
            v.y = pack_bf16(acc[m][2 * h][2], acc[m][2 * h][3]);
            v.z = pack_bf16(acc[m][2 * h + 1][0], acc[m][2 * h + 1][1]);
            v.w = pack_bf16(acc[m][2 * h + 1][2], acc[m][2 * h + 1][3]);
            __builtin_amdgcn_raw_buffer_store_b128(v, rt, l16 + (m * 2 + h) * 1024, 0, SYS);
        }
    const u32 fo = OFF_RF + list * 512 + ((LDS_LD(misc + 4) & 15u) * 8 + wid) * 4;
    WAIT_VM(0);
    u32 todo = 3;
    if (w_dist(tq) == 0) {
        const u32 tp = x_pair(2 * tj + (wid >> 2)), so = OFF_ST + ((((rank ^ x_dist(tp)) * 4 + x_panel(tp)) * 4 + w_panel(tq)) * 8 + (wid & 3) * 2) * 4;
        const u32 old = __builtin_amdgcn_raw_ptr_buffer_atomic_add_i32(1, rm, lane < 2 ? lane * 4 : OOR, so, AGT);
        (void)__builtin_amdgcn_raw_ptr_buffer_atomic_add_i32(2, rm, lane < 2 && !(old & 3u) ? lane * 4 : OOR, so, AGT);
        todo = (u32)__ballot(old & 3u) & 3u;
    }
    __builtin_amdgcn_raw_buffer_store_b32(call << 12 | todo << 10 | id, rsrc(P.ws[rank ^ w_dist(tq)], WS_NEED), l0, fo, SYS);
    if (wid == 0) STAMP(4);
}

static int knob(const char*, int d) { return d; }
static void make_schedule(Params& p) {
    const int ne = knob("NE", NE), up = knob("UP", UP), lead = knob("LEAD", LEAD), hw = knob("HW", HW), rnum = knob("RNUM", RNUM), wl = knob("WL", WL);
    p.pn = up | ne << 8 | knob("GX", GX) << 16;
    int n1 = 0;
    const int xl = knob("XL", XL);
    for (int t = -lead; t < 128 + xl; ++t) {
        const int wi = t + lead;
        if (t >= 0 && t < 128) p.tab[1][n1++] = t;
        if (t >= xl) p.tab[1][n1++] = (t - xl) | 1 << 7;
        if (wi < 128 && (wi < 4 * up || wi >= 4 * (up + ne))) { p.tab[1][n1++] = wi | 2 << 7; p.tab[1][n1++] = wi | 3 << 7; }
    }
    int n0 = 0, nd = 0, r = 0;
    auto direct = [&](int e) { p.tab[0][n0++] = e; if (++nd >= hw) for (int k = 0; k < rnum && r < 16 * ne; ++k, ++r) p.tab[0][n0++] = (r >> 2) | (r & 3) << 7 | 1 << 9; };
    for (int t = -wl; t < 128; ++t) { if (t + wl < 128) direct(t + wl); if (t >= 0) direct(t | 1 << 7); }
    for (; r < 16 * ne; ++r) p.tab[0][n0++] = (r >> 2) | (r & 3) << 7 | 1 << 9;
    for (int i = n0; i < 512; ++i) p.tab[0][i] = 0;
    for (int i = n1; i < 512; ++i) p.tab[1][i] = 0;
    for (int i = 0; i < 16 * NPW; ++i) p.grp[i] = Group{};
    int g = 0, nl = 0, qid = 0;
    auto add = [&](int ng, u32 d, u32 ea, u32 eb, u32 slots, u32 tab) {
        for (int i = 0; i < ng; ++i) {
            const Group G = {{d | qid << 8 | (8 * ng) << 16 | (8 * i) << 24, ea | eb << 16, slots, tab}};
            if (d) p.grp[g++] = G; else { p.grp[(nl >> 1) * 16 + 14 + (nl & 1)] = G; ++nl; }
        }
        ++qid;
    };
    for (u32 dd = 1; dd <= 8; ++dd) {
        const u32 d = dd & 7;
        u32 xs[4], ws[4]; int nx = 0, nwv = 0;
        for (u32 s = 0; s < 16; ++s) { if (x_dist(s) == d) xs[nx++] = s; if (w_dist(s) == d) ws[nwv++] = 16 + s; }
        if (d == 0) for (u32 h = 0; h < 2; ++h) add(knob("GL0", 2), 0, 0, 256, 0, (2 * h) << 4);
        else if (d == 2) add(2, d, 512, 512, xs[0] | xs[1] << 8 | xs[2] << 16 | xs[3] << 24, knob("D2F", D2F) << 8);
        else if (d & 1) add(2, d, n1, n1 + 256, xs[0] | xs[1] << 8 | ws[0] << 16 | ws[1] << 24, 1 | d << 4);
        else add(2, d, n0, n0 + 256, ws[0] | ws[1] << 8, 2 | d << 4);
    }
}

}

extern "C" void kernel_launch(int rank, int nranks, ncclComm_t comm,
                              hipStream_t stream,
                              void* const* d_in, const int* in_sizes, int n_in,
                              void* d_out, int out_size,
                              const int* in_dtypes, int out_dtype,
                              void* const* d_out_all,
                              void* const* d_ws_all, size_t ws_bytes) {
    if (nranks != NR || (unsigned)rank >= (unsigned)NR || ws_bytes < WS_NEED || n_in < 2 ||
        in_sizes[0] != 512 * 4096 || in_sizes[1] != 4096 * 512 || out_size != 4096 * 512) {
        fprintf(stderr, "unexpected problem shape\n");
        return;
    }
    Params p;
    p.x = (const char*)d_in[0];
    p.w = (const char*)d_in[1];
    p.out = (char*)d_out;
    for (int i = 0; i < NR; ++i) p.ws[i] = (char*)d_ws_all[i];
    p.rank = rank;
    make_schedule(p);
    hipLaunchKernelGGL(ag_gemm_kernel, dim3(256), dim3(768), 0, stream, p);
    HIP_CHECK(hipGetLastError());
}
```

```cpp
#include <hip/hip_runtime.h>
#include <rccl/rccl.h>
#include <cstdio>

#define HIP_CHECK(expr) do { hipError_t _e = (expr); if (_e != hipSuccess) { \
    fprintf(stderr, "HIP error %s at %s:%d\n", hipGetErrorString(_e), __FILE__, __LINE__); \
    return; } } while (0)

namespace {

typedef unsigned u32;
typedef unsigned long long u64;
typedef u32 u32x4 __attribute__((ext_vector_type(4)));
typedef u32 u32x2 __attribute__((ext_vector_type(2)));
typedef float f32x4 __attribute__((ext_vector_type(4)));
typedef float f32x2 __attribute__((ext_vector_type(2)));
typedef short s16x4 __attribute__((ext_vector_type(4)));
typedef __bf16 bf16x8 __attribute__((ext_vector_type(8)));
typedef __bf16 bf16x2 __attribute__((ext_vector_type(2)));

constexpr int NR = 8;
constexpr u32 PANEL = 512u << 10;
constexpr u32 OFF_S = 0;
constexpr u32 OFF_T = 32u << 20;
constexpr u32 OFF_P = 40u << 20;
constexpr u32 OFF_AF = 44u << 20;
constexpr u32 OFF_RF = OFF_AF + 16384;
constexpr u32 OFF_ST = OFF_RF + 4096;
constexpr u32 OFF_CTR = OFF_ST + 4096;
constexpr u32 OFF_TMO = OFF_CTR + 16384;
constexpr u32 OFF_Q = OFF_TMO + 1024;
constexpr u32 OFF_LT = OFF_Q + 4096;
constexpr u32 WS_NEED = OFF_LT + 1024;
constexpr u64 DEADLINE = 100000000ull;
constexpr int SYS = 17;
constexpr int AGT = 16;
constexpr u32 OOR = 0x80000000u;

constexpr int NE = 9;
constexpr int UP = 23;
constexpr int LEAD = 48;
constexpr int HW = 192;
constexpr int RNUM = 4;
constexpr int XL = 16;
constexpr int WL = 24;
constexpr int D2F = 96;
constexpr int WP = 3;
constexpr int RING = 2;
constexpr int GX = 14;

struct alignas(64) Group { u32 w[4]; char* peer; u32 e0[8]; };
struct alignas(64) Params {
    const char* x;
    const char* w;
    char* out;
    char* own;
    int rank; u32 pn;
    alignas(64) char* ws[NR];
    Group grp[22];
    unsigned short tab[2][512];
};

__host__ __device__ __forceinline__ u32 x_dist(u32 p) { return (0x7520'3210u >> (4 * (p >> 1))) & 7u; }
__host__ __device__ __forceinline__ u32 x_panel(u32 p) { return ((p >> 3) << 1) | (p & 1u); }
__host__ __device__ __forceinline__ u32 w_dist(u32 q) { return (u32)(0x6530'7610'7430'5410ull >> (4 * q)) & 7u; }
__host__ __device__ __forceinline__ u32 w_panel(u32 q) { return q >> 2; }
__host__ __device__ __forceinline__ u32 x_pair(u32 i) { return (u32)(0xFDEC'BA98'7354'6210ull >> (4 * i)) & 15u; }

__device__ __forceinline__ __amdgpu_buffer_rsrc_t rsrc(const void* p, u32 bytes) {
    return __builtin_amdgcn_make_buffer_rsrc(const_cast<void*>(p), 0, bytes, 0x00020000);
}
__device__ __forceinline__ u32 pack_bf16(float a, float b) {
    return __builtin_bit_cast(u32, __builtin_convertvector((f32x2){a, b}, bf16x2));
}
__device__ __forceinline__ float gelu(float y) {
    const float t = y * (-2.302208198f - 0.1029432397f * y * y);
    return y * __builtin_amdgcn_rcpf(1.0f + __builtin_amdgcn_exp2f(t));
}
#define WAIT_VM(n) asm volatile("s_waitcnt vmcnt(%0)" :: "i"(n) : "memory")
#define CBAR() asm volatile("" ::: "memory")
#define RFL(x) ((u32)__builtin_amdgcn_readfirstlane(x))
#define INL __attribute__((always_inline))
#define LAS __attribute__((address_space(3)))
#define BARRIER() do { asm volatile("s_waitcnt lgkmcnt(0)" ::: "memory"); __builtin_amdgcn_s_barrier(); CBAR(); } while (0)
#define LDS_LD(p) RFL(*(volatile LAS u32*)(p))
#define LDS_ST(p, v) do { *(volatile LAS u32*)(p) = (v); } while (0)
#define DMA16(r, d, v, o) __builtin_amdgcn_raw_ptr_buffer_load_lds(r, d, 16, v, o, 0, SYS)


constexpr u32 STAGE = 49152;
constexpr u32 LDS_POLL = 3 * STAGE;
constexpr u32 LDS_MISC = LDS_POLL + 1024;
constexpr u32 LDS_BYTES = LDS_MISC + 64;
enum { K_COPY, K_W, K_RED, K_SIG };

__global__ void __launch_bounds__(768, 3) ag_gemm_kernel(const Params P) {
    __shared__ __attribute__((aligned(16))) char smem[LDS_BYTES];
    LAS char* const lds = (LAS char*)smem;
    const u32 wid = RFL(threadIdx.x >> 6);
    const u32 lane = threadIdx.x & 63;
    const u32 multi = blockIdx.x < 112, pair = multi && wid < 8 ? 0u : wid & 3;
    const Group G = P.grp[multi ? blockIdx.x >> 3 : pair * 2 + 14 + ((blockIdx.x >> 3) & 1)];
    const u32 e01 = P.grp[multi ? blockIdx.x >> 3 : 14].e0[blockIdx.x & 7];
    const char* const px = P.x; const char* const pw = P.w; char* const myws = P.own;
    const u32 rank = P.rank & 7, sch = P.pn;
    asm volatile("" :: "s"(px), "s"(pw), "s"(myws), "s"(rank), "s"(sch), "s"(G.w[0]), "s"(G.w[1]), "s"(G.w[2]), "s"(G.w[3]), "s"(G.peer), "s"(e01));
    u32* const ctr = (u32*)(myws + OFF_CTR) + blockIdx.x * 16 + wid;
    const u32 ctrv = __hip_atomic_load(ctr, __ATOMIC_RELAXED, __HIP_MEMORY_SCOPE_AGENT);
    u32* const tmo = (u32*)(myws + OFF_TMO);
    const u32 l0 = lane ? OOR : 0u, l16 = lane * 16;
    const auto rm = rsrc(myws, WS_NEED);
    const u32 vx = (lane & 15) * 8192 + (lane >> 4) * 16, vw = (lane >> 3) * 1024 + (lane & 7) * 16;
    const u32 tro = ((lane >> 4) * 8 + ((lane >> 2) & 3)) * 128 + (lane & 3) * 32;

    if (blockIdx.x < 128) {
        const u32 g = blockIdx.x;
        const u32 dist = G.w[0] & 7u, nw = multi && pair ? 0u : (G.w[0] >> 16) & 255u, ea = G.w[1] & 0xffffu, eb = G.w[1] >> 16;
        const u32 peer = rank ^ dist, qo = OFF_Q + ((G.w[0] >> 8) & 15u) * 256;
        const auto rd = rsrc(G.peer, WS_NEED);
        LAS char* const ring = lds + pair * 16384;
        LAS u32* const rdesc = (LAS u32*)(lds + 131072 + pair * 128);
        const u32 po = (lane & 1) * 1024 + ((lane >> 1) & 3) * 256 + (lane >> 3) * 16;
        bool dead = false;
        if (wid >= 8) {
            if (lane < 16) rdesc[16 + lane] = 0;
            const u32 up = sch & 255u, une = (sch >> 8) & 255u, df = G.w[3] >> 8;
            const u32 rsl = dist == 4 ? (17u | 19u << 8 | 21u << 16 | 23u << 24) : (25u | 27u << 8 | 29u << 16 | 30u << 24);
            const u32 rdl = dist == 4 ? (19u | 17u << 8 | 23u << 16 | 21u << 24) : (27u | 25u << 8 | 30u << 16 | 29u << 24);
            const u32* const tab = (const u32*)P.tab[G.w[3] & 1];
            const u32 urev = (peer & 4) ? 31u : 0u;
            const char* const xs = px + (peer & 1) * 4096;
            const char* const wsrc = pw + (size_t)(peer & 1) * (2u << 20);
            const u32 xbytes = (4u << 20) - (peer & 1) * 4096;
            u32x4 dv = {0u, 0u, 0u, 0u};
            u32 cur = (G.w[0] >> 24) + (g & 7), nxt = cur + nw, c = 0, j = 0, pub = 0, used = 0, call = 0, dec = 0, sbase = 0, depth = RING;
            u64 t0 = 0;
            const bool idle = !nw || !ea;
            auto timed_out = [&]() INL { return __builtin_amdgcn_s_memrealtime() - t0 > DEADLINE; };
            auto publish = [&](u32 n) INL { if (n != pub) { pub = n; LDS_ST(rdesc + 16, n); } };
            auto slot_wait = [&]() INL {
                if (used + depth > j) return;
                used = LDS_LD(rdesc + 17);
                if (used + depth > j) return;
                WAIT_VM(0); publish(j);
                while (used + depth <= j) { __builtin_amdgcn_s_sleep(1); used = LDS_LD(rdesc + 17); }
            };
            auto word = [&](u32 w) INL -> u32 {
                const u32 l = w >> 2;
                const u32 v0 = __builtin_amdgcn_readlane(dv.x, l), v1 = __builtin_amdgcn_readlane(dv.y, l), v2 = __builtin_amdgcn_readlane(dv.z, l), v3 = __builtin_amdgcn_readlane(dv.w, l);
                return (w & 2) ? ((w & 1) ? v3 : v2) : ((w & 1) ? v1 : v0);
            };
            auto dep_wait = [&](u32 w, u32 sh, u32 want, u32 dvo) INL {
                if ((word(w) >> sh) >= want) return;
                WAIT_VM(0); publish(j);
                while (!dead) {
                    dv = __builtin_amdgcn_raw_buffer_load_b128(rm, dvo, 0, SYS);
                    u32 t = RFL(__hip_atomic_load(tmo, __ATOMIC_RELAXED, __HIP_MEMORY_SCOPE_AGENT));
                    asm volatile("" : "+s"(t));
                    if ((word(w) >> sh) >= want) break;
                    if (t == call || timed_out()) dead = true;
                    __builtin_amdgcn_s_sleep(1);
                }
                CBAR();
            };
            auto post = [&](u32 kind) INL { slot_wait(); LDS_ST(rdesc + (j & 3) * 4 + 3, kind); WAIT_VM(0); ++j; publish(j); };
            const u32 dvr = OFF_AF + ((rsl >> (8 * (lane >> 4))) & 31u) * 512 + (lane & 15) * 16;
#define PUSH_STEP(mode) do { \
                const u32 e = mode ? (e01 >> (mode == 2 ? 16 : 0)) & 1023u : (G.w[3] & 3) ? (tab[cur >> 1] >> ((cur & 1) * 16)) & 1023u \
                            : cur < 2 * df ? (cur >> 1) | (cur & 1) << 7 : cur >= 512 - 2 * df ? ((cur >> 1) - 128) | (2 + (cur & 1)) << 7 \
                            : (((cur - 2 * df) >> 2) + ((cur - 2 * df) & 2 ? 0u : df)) | ((cur - 2 * df) & 3) << 7; \
                const u32 i = e & 127u, k = (e >> 7) & 3u; \
                u32 kind = K_COPY, so, s1 = 1024, s2 = 2048, pc, d2 = 2048, vo = l16, src = 2; \
                if (e >> 9) { \
                    dep_wait(k * 64 + i, 0, call, dvr); \
                    so = sbase + ((rsl >> (8 * k)) & 31u) * PANEL + i * 4096; \
                    pc = ((rdl >> (8 * k)) & 31u) * 128 + i + 4 * up; \
                } else { \
                    const u32 v = i >> 2, slot = (G.w[2] >> (8 * k)) & 31u, u = (v < up ? v : v < up + une ? 31 + up - v : v - une) ^ urev, h = i & 3; \
                    pc = slot * 128 + i; \
                    if (slot >= 16) { kind = K_W; src = 1; vo = vw; s1 = 8192; s2 = 16384; so = (u * 64 + (h >> 1) * 32) * 1024 + w_panel(slot & 15) * 256 + (h & 1) * 128; } \
                    else { src = 0; vo = vx; s1 = 131072; s2 = 64; d2 = 8192; so = (x_panel(slot) * 128 + h * 32) * 8192 + u * 128; } \
                } \
                slot_wait(); \
                const auto rs = rsrc(src == 2 ? myws + 0 : src ? wsrc + 0 : xs + 0, src == 2 ? WS_NEED : src ? (2u << 20) : xbytes); \
                LAS char* const sl = ring + (j & 3) * 4096; \
                if (mode && idle) so = OOR; \
                DMA16(rs, sl, vo, so); DMA16(rs, sl + 1024, vo, so + s1); DMA16(rs, sl + 2048, vo, so + s2); DMA16(rs, sl + 3072, vo, so + s1 + s2); \
                *(volatile LAS u32x4*)(rdesc + (j & 3) * 4) = (u32x4){pc, d2, 0u, kind}; \
                if (mode == 1) { \
                    c = __builtin_amdgcn_raw_ptr_buffer_atomic_add_i32(1, rm, idle ? OOR : l0, qo, AGT); \
                    CBAR(); \
                    if (!idle) { cur = nxt; nxt += nw; j = 1; } \
                    break; \
                } \
                if (mode == 2) { \
                    BARRIER(); \
                    asm volatile("s_waitcnt vmcnt(5)\n\tv_readfirstlane_b32 %0, %1" : "=s"(call) : "v"(ctrv) : "memory"); \
                    publish(j); \
                    ++call; \
                    t0 = __builtin_amdgcn_s_memrealtime(); \
                    sbase = OFF_S + (call & 1u) * 32 * PANEL; \
                    dec = 3u * nw - (call - 1u) * eb; \
                    if (lane == 0) __hip_atomic_store(ctr, call, __ATOMIC_RELAXED, __HIP_MEMORY_SCOPE_AGENT); \
                    if (idle) { c = cur + 2 * nw - dec; break; } \
                } else { \
                    WAIT_VM(4); \
                    publish(j); \
                } \
                cur = nxt; nxt = RFL(c) + dec; \
                c = __builtin_amdgcn_raw_ptr_buffer_atomic_add_i32(1, rm, l0, qo, AGT); \
                CBAR(); \
                ++j; \
            } while (0)
            PUSH_STEP(1); PUSH_STEP(2);

            if (!nw) return;
            while (cur < ea) PUSH_STEP(0);
            depth = 4;
            if (dist && eb > ea) post(K_SIG);
            const u32 jb = j;
            LDS_ST(rdesc + 18, j + 1);
            if (eb > ea) {
                const u32 lo = OFF_RF + ((G.w[3] >> 4) & 7u) * 512;
                u32 wv = __builtin_amdgcn_raw_buffer_load_b32(rm, 0, lo + (((cur - ea) >> 1) & 127u) * 4, SYS);
                asm volatile("" : "+v"(wv));
                while (dist && !dead) {
                    const u32 v = RFL(__builtin_amdgcn_raw_buffer_load_b32(rm, 0, qo + 128, SYS));
                    CBAR();
                    if (v + ((sch >> 16) & 255u) >= call * nw) break;
                    if (timed_out()) dead = true;
                    __builtin_amdgcn_s_sleep(1);
                }
                while (cur < eb) {
                    const u32 r = cur - ea;
                    u32 en = RFL(wv);
                    if ((en >> 12) < call) {
                        WAIT_VM(0); publish(j);
                        while (!dead) {
                            const u32 w = __builtin_amdgcn_raw_buffer_load_b32(rm, lane & 1 ? OFF_TMO : lo + (r >> 1) * 4, 0, SYS);
                            en = RFL(w);
                            if ((en >> 12) >= call) break;
                            if ((u32)__builtin_amdgcn_readlane(w, 1) == call || timed_out()) dead = true;
                            __builtin_amdgcn_s_sleep(1);
                        }
                        CBAR();
                    }
                    const u32 id = en & 1023u, tp = x_pair(2 * (id >> 7) + ((id >> 2) & 1));
                    const u32 so = (en >> (10 + (r & 1))) & 1 ? OFF_T + id * 8192 + (r & 1) * 4096 : OOR;
                    const u32 unit = (((peer ^ x_dist(tp)) * 4 + x_panel(tp)) * 4 + w_panel((id >> 3) & 15u)) * 8 + (id & 3) * 2 + (r & 1);
                    if (j < jb + 4) slot_wait();
                    else for (;;) {
                        const u32x2 fr = *(volatile LAS u32x2*)(rdesc + 24 + (j & 3) * 2);
                        if (min(RFL(fr.x), RFL(fr.y)) + 3 >= j) break;
                        WAIT_VM(0); publish(j);
                        __builtin_amdgcn_s_sleep(1);
                    }
                    LAS char* const sl = ring + (j & 3) * 4096;
                    wv = __builtin_amdgcn_raw_buffer_load_b32(rm, 0, lo + (((nxt - ea) >> 1) & 127u) * 4, SYS);
                    const u32 tk = __builtin_amdgcn_raw_ptr_buffer_atomic_add_i32(dist ? 1 : 0, rm, dead ? OOR : l0, OFF_ST + unit * 4, AGT);
                    WAIT_VM(WP + 2); DMA16(rd, sl, l16, so);
                    WAIT_VM(WP + 2); DMA16(rd, sl + 1024, l16 + 1024, so);
                    WAIT_VM(WP + 2); DMA16(rd, sl + 2048, l16 + 2048, so);
                    WAIT_VM(WP + 2); DMA16(rd, sl + 3072, l16 + 3072, so);
                    WAIT_VM(4);
                    publish(j);
                    const u32 tv = RFL(tk) & 3u, t = (unit >> 6) & 1, cp = (unit >> 3) & 3, ob = peer ^ (t << 2 | ((cp ^ ((cp >> 1) & t)) & 1) << 1 | 1);
                    const u32 st = dead ? 2u : so == OOR ? 4u : dist && !tv ? 0u : ob == rank ? 1u : (dist ? tv == 3 : !tv) ? 3u : 5u;
                    u32 tp2 = t * 8 + ((unit >> 5) & 1), tq2 = cp * 4;
                    if (st == 1) {
                        while (x_dist(tp2) != (ob ^ (unit >> 7))) tp2 += 2;
                        while (w_dist(tq2) != (ob ^ rank)) ++tq2;
                    }
                    tp2 = x_pair(tp2);
                    *(volatile LAS u32x4*)(rdesc + (j & 3) * 4) = (u32x4){unit, st, OFF_T + ((tp2 >> 1) * 16 + tq2) * 65536 + (tp2 & 1) * 32768 + (so & 0x7000u), K_RED};
                    LDS_ST(rdesc + 19, j + 1);
                    cur = nxt; nxt = RFL(c) + dec;
                    c = __builtin_amdgcn_raw_ptr_buffer_atomic_add_i32(1, rm, l0, qo, AGT);
                    CBAR();
                    ++j;
                }
            }
            WAIT_VM(0); publish(j);
            LDS_ST(rdesc + 20, j + 1);
            if (dead && lane == 0) {
                __hip_atomic_store(tmo, call, __ATOMIC_RELAXED, __HIP_MEMORY_SCOPE_AGENT);
                *(volatile u32*)(P.out + (size_t)g * 32768 + pair * 4) = 0x7fc07fc0u;
            }
            return;
        }
        BARRIER();
        const u32 call = RFL(ctrv) + 1u;
        if (lane == 0) __hip_atomic_store(ctr, call, __ATOMIC_RELAXED, __HIP_MEMORY_SCOPE_AGENT);
        if (!nw) return;
        const u32 half = wid >> 2;
        u32 k = 0, full = 0;
        if (multi ? wid == 0 : !half) {
            const u32 sbase = OFF_S + (call & 1u) * 32 * PANEL + l16;
            u32 pend = OOR;
#define FLAG() do { __builtin_amdgcn_raw_buffer_store_b32(call, rd, pend, 0, SYS); CBAR(); pend = OOR; } while (0)
            for (;;) {
                if (full <= k) {
                    full = LDS_LD(rdesc + 16);
                    if (full <= k) {
                        if (RFL(pend) != OOR) { WAIT_VM(0); FLAG(); }
                        while (full <= k && LDS_LD(rdesc + 18) != k + 1) { __builtin_amdgcn_s_sleep(1); full = LDS_LD(rdesc + 16); }
                    }
                    CBAR();
                }
                if (LDS_LD(rdesc + 18) == k + 1) { WAIT_VM(0); FLAG(); break; }
                const u32x4 dd = *(volatile LAS u32x4*)(rdesc + (k & 3) * 4);
                const u32 d0 = RFL(dd.x), d1 = RFL(dd.y), kind = RFL(dd.w);
                if (kind == K_SIG) {
                    (void)__builtin_amdgcn_raw_ptr_buffer_atomic_add_i32(1, rd, l0, qo + 128, AGT);
                    ++k; LDS_ST(rdesc + 17, k);
                    continue;
                }
                const LAS char* const sl = ring + (k & 3) * 4096;
                u32x4 f[4];
                if (kind == K_W) {
#pragma unroll
                    for (int n = 0; n < 4; ++n) {
                        const u32x2 lo = __builtin_bit_cast(u32x2, __builtin_amdgcn_ds_read_tr16_b64_v4i16((LAS s16x4*)(sl + tro + n * 8)));
                        const u32x2 hi = __builtin_bit_cast(u32x2, __builtin_amdgcn_ds_read_tr16_b64_v4i16((LAS s16x4*)(sl + tro + 512 + n * 8)));
                        f[n] = (u32x4){lo.x, lo.y, hi.x, hi.y};
                    }
                } else {
#pragma unroll
                    for (int n = 0; n < 4; ++n) f[n] = *(const LAS u32x4*)(sl + l16 + n * 1024);
                }
                asm volatile("s_waitcnt lgkmcnt(0)" : "+v"(f[0]), "+v"(f[1]), "+v"(f[2]), "+v"(f[3]) :: "memory");
                ++k; LDS_ST(rdesc + 17, k);
                const u32 vo0 = sbase + (d0 >> 2) * 16384 + (d0 & 3u) * (d1 == 8192 ? 2048u : 4096u), vo1 = vo0 + d1;
                WAIT_VM(3); __builtin_amdgcn_raw_buffer_store_b128(f[0], rd, vo0, 0, SYS);
                WAIT_VM(2); __builtin_amdgcn_raw_buffer_store_b128(f[1], rd, vo0 + 1024, 0, SYS);
                WAIT_VM(2); FLAG(); __builtin_amdgcn_raw_buffer_store_b128(f[2], rd, vo1, 0, SYS);
                WAIT_VM(3); __builtin_amdgcn_raw_buffer_store_b128(f[3], rd, vo1 + 1024, 0, SYS);
                pend = l0 | (OFF_AF + d0 * 4);
            }
        }
        while (!(k = LDS_LD(rdesc + 18))) __builtin_amdgcn_s_sleep(8);
        --k;
        if (multi) k += (wid - k) & 3u;
        const auto ro = rsrc(P.out, 4u << 20);
        for (;; k += multi ? 4u : 1u) {
            bool more;
            while (!(more = LDS_LD(rdesc + 19) > k) && LDS_LD(rdesc + 20) - 1 > k) __builtin_amdgcn_s_sleep(1);
            if (!more && LDS_LD(rdesc + 19) <= k) break;
            CBAR();
            const u32x4 dd = *(volatile LAS u32x4*)(rdesc + (k & 3) * 4);
            const u32 d0 = RFL(dd.x), d1 = RFL(dd.y);
            u32x4 f[2], o[2];
            const u32 v2 = d1 == 1 ? po + half * 2048 : l16 + half * 2048, o2 = d1 == 1 ? RFL(dd.z) : d1 == 3 ? OFF_P + d0 * 4096 : OOR;
            o[0] = __builtin_amdgcn_raw_buffer_load_b128(rm, v2, o2, SYS);
            o[1] = __builtin_amdgcn_raw_buffer_load_b128(rm, v2 + (d1 == 1 ? 128u : 1024u), o2, SYS);
            while (full <= k) { full = LDS_LD(rdesc + 16); if (full <= k) __builtin_amdgcn_s_sleep(1); }
            CBAR();
            const LAS char* const sl = ring + (k & 3) * 4096 + half * 2048 + po;
            f[0] = *(const LAS u32x4*)sl; f[1] = *(const LAS u32x4*)(sl + 128);
            asm volatile("s_waitcnt lgkmcnt(0)" : "+v"(f[0]), "+v"(f[1]) :: "memory");
            LDS_ST(rdesc + 24 + (k & 3) * 2 + half, k + 1);
            if (d1 == 4) continue;
            if (d1 == 0) {
                const u32 pp = OFF_P + d0 * 4096 + half * 2048 + l16;
                __builtin_amdgcn_raw_buffer_store_b128(f[0], rm, pp, 0, SYS); __builtin_amdgcn_raw_buffer_store_b128(f[1], rm, pp + 1024, 0, SYS);
                WAIT_VM(0);
                (void)__builtin_amdgcn_raw_ptr_buffer_atomic_add_i32(1, rm, l0, OFF_ST + d0 * 4, AGT);
                CBAR();
                continue;
            }
            bool bad = d1 == 2;
            if (d1 == 5) {
                const u64 t0 = __builtin_amdgcn_s_memrealtime();
                while (RFL(__builtin_amdgcn_raw_buffer_load_b32(rm, 0, OFF_ST + d0 * 4, AGT)) & 3u) {
                    CBAR();
                    if (__builtin_amdgcn_s_memrealtime() - t0 > DEADLINE) { bad = true; break; }
                    __builtin_amdgcn_s_sleep(2);
                }
                CBAR();
                o[0] = __builtin_amdgcn_raw_buffer_load_b128(rm, v2, OFF_P + d0 * 4096, SYS);
                o[1] = __builtin_amdgcn_raw_buffer_load_b128(rm, v2 + 1024, OFF_P + d0 * 4096, SYS);
            }
            const float pz = bad ? __builtin_nanf("") : 0.f;
            const u32 oo = ((d0 >> 5) * 128 + ((d0 >> 2) & 1) * 64 + (d0 & 1) * 32 + half * 16 + (lane >> 3)) * 1024 + ((d0 >> 3) & 3) * 256 + ((d0 >> 1) & 1) * 128 + (lane & 7) * 16;
#pragma unroll
            for (int n = 0; n < 2; ++n) {
                u32x4 r;
#pragma unroll
                for (int q = 0; q < 4; ++q) {
                    const float lo = __builtin_bit_cast(float, f[n][q] << 16) + __builtin_bit_cast(float, o[n][q] << 16) + pz;
                    const float hi = __builtin_bit_cast(float, f[n][q] & 0xffff0000u) + __builtin_bit_cast(float, o[n][q] & 0xffff0000u) + pz;
                    r[q] = pack_bf16(gelu(lo), gelu(hi));
                }
                __builtin_amdgcn_raw_buffer_store_b128(r, ro, oo + n * 8192, 0, AGT);
            }
        }
        return;
    }

    const u32 gg = blockIdx.x - 128;
    const u32 tj = ((gg & 7) >> 1) * 2 + (gg >> 6), tq = (gg & 1) * 8 + ((gg >> 3) & 7);
    const u32 list = w_dist(tq) ? w_dist(tq) : (tq >> 3) * 2;
    LAS u32* const misc = (LAS u32*)(lds + LDS_MISC);
    if (wid >= 8) {
        const u32 l = wid - 8;
        const u32 call = RFL(ctrv) + 1u;
        if (lane == 0) __hip_atomic_store(ctr, call, __ATOMIC_RELAXED, __HIP_MEMORY_SCOPE_AGENT);
        bool dead = false;
        const u64 t0 = __builtin_amdgcn_s_memrealtime();
        const u32 xb = OFF_S + ((call & 1u) * 32 + x_pair(2 * tj)) * PANEL, xd = (x_pair(2 * tj + 1) - x_pair(2 * tj)) * PANEL, wb = OFF_S + ((call & 1u) * 32 + 16 + tq) * PANEL;
        const u32 gx = l * 2048 + l16, gw = l * 4096 + l16;
        const u32 fsl = lane >> 4;
        const u32 fo = fsl == 3 ? OFF_TMO : OFF_AF + (fsl == 2 ? 16 + tq : x_pair(2 * tj + fsl)) * 512 + (lane & 15) * 16 + l * 4;
        const u32 fstep = fsl == 3 ? 0u : 16u;
        LAS u32* const pollbuf = (LAS u32*)(lds + LDS_POLL + l * 256);
        u32 avail = 0, pbase = 0;
#define POLL() do { pbase = avail; __builtin_amdgcn_raw_ptr_buffer_load_lds(rm, pollbuf, 4, fo + min(avail, 31u) * fstep, 0, 0, SYS); } while (0)
#define POLL_READ() do { const u32 _v = *(volatile LAS u32*)(pollbuf + lane); \
        const u64 _m = ~__ballot(_v >= call || pbase + (lane & 15) >= 32); \
        const u32 _a = pbase + __builtin_ctz((((u32)_m | (u32)(_m >> 16) | (u32)(_m >> 32)) & 0xffffu) | 0x10000u); \
        avail = max(avail, _a); if (__ballot(_v == call) >> 48) { dead = true; avail = 64; } } while (0)
#define SPIN(n) do { while (avail < (u32)(n)) { POLL(); WAIT_VM(0); POLL_READ(); \
        if (avail < (u32)(n)) { if (__builtin_amdgcn_s_memrealtime() - t0 > DEADLINE) { dead = true; avail = 64; } else __builtin_amdgcn_s_sleep(1); } } CBAR(); } while (0)
#define STAGE_IN(u, sb) do { LAS char* _x = lds + (sb) + l * 2048; LAS char* _w = lds + (sb) + 32768 + l * 4096; const u32 _x0 = xb + (u) * 16384, _w0 = wb + (u) * 16384; \
        DMA16(rm, _x, gx, _x0); DMA16(rm, _x + 1024, gx + 1024, _x0); DMA16(rm, _x + 16384, gx, _x0 + xd); DMA16(rm, _x + 17408, gx + 1024, _x0 + xd); \
        DMA16(rm, _w, gw, _w0); DMA16(rm, _w + 1024, gw + 1024, _w0); DMA16(rm, _w + 2048, gw + 2048, _w0); DMA16(rm, _w + 3072, gw + 3072, _w0); \
        DMA16(rm, _x + 8192, gx, _x0 + 8192); DMA16(rm, _x + 9216, gx + 1024, _x0 + 8192); DMA16(rm, _x + 24576, gx, _x0 + xd + 8192); DMA16(rm, _x + 25600, gx + 1024, _x0 + xd + 8192); CBAR(); } while (0)
        if (lane == 0) misc[l] = 0;
        SPIN(1); STAGE_IN(0, 0);
        SPIN(2); POLL(); STAGE_IN(1, STAGE);
        u32 sn = 2 * STAGE;
        for (u32 u = 0; u < 30; ++u) {
            WAIT_VM(12); BARRIER();
            POLL_READ();
            SPIN(u + 3); POLL(); STAGE_IN(u + 2, sn);
            sn = sn == 2 * STAGE ? 0 : sn + STAGE;
        }
        if (dead && lane == 0) { misc[l] = 1; __hip_atomic_store(tmo, call, __ATOMIC_RELAXED, __HIP_MEMORY_SCOPE_AGENT); }
        WAIT_VM(12); BARRIER();
        if (l == 0 && lane == 0)
            misc[4] = __hip_atomic_fetch_add((u32*)(myws + OFF_LT + list * 128), 1u, __ATOMIC_RELAXED, __HIP_MEMORY_SCOPE_AGENT);
        WAIT_VM(0); BARRIER();
        return;
    }
    u32 call;
    {
        const u32 h = wid & 3, isw = wid >> 2, v = gg & 31u, pn = gg >> 5, slot = isw ? 16 + pn * 4 : (pn >> 1) * 8 + (pn & 1);
        const u32 up = sch & 255u, une = (sch >> 8) & 255u, u = (v < up ? v : v < up + une ? 31 + up - v : v - une) ^ ((rank & 4) ? 31u : 0u);
        LAS char* const sc = lds + 2 * STAGE + wid * 4096;
        const auto rs = rsrc(isw ? pw + 0 : px + 0, 4u << 20);
        const u32 so = isw ? ((rank & 1) * 2048 + u * 64 + (h >> 1) * 32) * 1024 + pn * 256 + (h & 1) * 128 : (pn * 128 + h * 32) * 8192 + (rank & 1) * 4096 + u * 128;
        const u32 s1 = isw ? 8192u : 131072u, s2 = isw ? 16384u : 64u, vo = isw ? vw : vx;
        DMA16(rs, sc, vo, so); DMA16(rs, sc + 1024, vo, so + s1); DMA16(rs, sc + 2048, vo, so + s2); DMA16(rs, sc + 3072, vo, so + s1 + s2);
        CBAR();
        call = RFL(ctrv) + 1u;
        if (lane == 0) __hip_atomic_store(ctr, call, __ATOMIC_RELAXED, __HIP_MEMORY_SCOPE_AGENT);
        WAIT_VM(1);
        u32x4 f[4];
        if (isw) {
#pragma unroll
            for (int n = 0; n < 4; ++n) {
                const u32x2 lo = __builtin_bit_cast(u32x2, __builtin_amdgcn_ds_read_tr16_b64_v4i16((LAS s16x4*)(sc + tro + n * 8)));
                const u32x2 hi = __builtin_bit_cast(u32x2, __builtin_amdgcn_ds_read_tr16_b64_v4i16((LAS s16x4*)(sc + tro + 512 + n * 8)));
                f[n] = (u32x4){lo.x, lo.y, hi.x, hi.y};
            }
        } else {
#pragma unroll
            for (int n = 0; n < 4; ++n) f[n] = *(const LAS u32x4*)(sc + l16 + n * 1024);
        }
        const u32 vo0 = OFF_S + ((call & 1u) * 32 + slot) * PANEL + v * 16384 + h * (isw ? 4096u : 2048u) + l16, vo1 = vo0 + (isw ? 2048u : 8192u);
        __builtin_amdgcn_raw_buffer_store_b128(f[0], rm, vo0, 0, SYS); __builtin_amdgcn_raw_buffer_store_b128(f[1], rm, vo0 + 1024, 0, SYS);
        __builtin_amdgcn_raw_buffer_store_b128(f[2], rm, vo1, 0, SYS); __builtin_amdgcn_raw_buffer_store_b128(f[3], rm, vo1 + 1024, 0, SYS);
        WAIT_VM(0);
        __builtin_amdgcn_raw_buffer_store_b32(call, rm, l0, OFF_AF + (slot * 128 + v * 4 + h) * 4, SYS);
    }
    const u32 xo = (wid >> 2) * 16384 + ((wid >> 1) & 1) * 4096 + l16, wo = 32768 + (wid & 1) * 4096 + l16;
    f32x4 acc[4][4];
#pragma unroll
    for (int m = 0; m < 4; ++m)
#pragma unroll
        for (int n = 0; n < 4; ++n) acc[m][n] = (f32x4){0.f, 0.f, 0.f, 0.f};
    u32 sb = 0;
    for (u32 u = 0; u < 32; ++u) {
        BARRIER();
        const LAS char* const xp = lds + sb + xo; const LAS char* const wp = lds + sb + wo;
        bf16x8 xf[2][4], wf[2][4];
#pragma unroll
        for (int k = 0; k < 2; ++k)
#pragma unroll
            for (int i = 0; i < 4; ++i) { wf[k][i] = *(const LAS bf16x8*)(wp + k * 8192 + i * 1024); xf[k][i] = *(const LAS bf16x8*)(xp + k * 8192 + i * 1024); }
        __builtin_amdgcn_sched_barrier(0);
#pragma unroll
        for (int k = 0; k < 2; ++k)
#pragma unroll
            for (int m = 0; m < 4; ++m)
#pragma unroll
                for (int n = 0; n < 4; ++n) acc[m][n] = __builtin_amdgcn_mfma_f32_16x16x32_bf16(wf[k][n], xf[k][m], acc[m][n], 0, 0, 0);
        __builtin_amdgcn_sched_barrier(0);
        sb = sb == 2 * STAGE ? 0 : sb + STAGE;
    }
    const u32x4 dv = *(volatile LAS u32x4*)misc;
    const float pz = (dv.x | dv.y | dv.z | dv.w) ? __builtin_nanf("") : 0.f;
    const u32 id = (tj * 16 + tq) * 8 + wid;
    const auto rt = rsrc(myws + OFF_T + id * 8192, 8192);
#pragma unroll
    for (int m = 0; m < 4; ++m)
#pragma unroll
        for (int h = 0; h < 2; ++h) {
            u32x4 v;
            v.x = pack_bf16(acc[m][2 * h][0] + pz, acc[m][2 * h][1]);
            v.y = pack_bf16(acc[m][2 * h][2], acc[m][2 * h][3]);
            v.z = pack_bf16(acc[m][2 * h + 1][0], acc[m][2 * h + 1][1]);
            v.w = pack_bf16(acc[m][2 * h + 1][2], acc[m][2 * h + 1][3]);
            __builtin_amdgcn_raw_buffer_store_b128(v, rt, l16 + (m * 2 + h) * 1024, 0, SYS);
        }
    const u32 fo = OFF_RF + list * 512 + ((LDS_LD(misc + 4) & 15u) * 8 + wid) * 4;
    WAIT_VM(0);
    u32 todo = 3;
    if (w_dist(tq) == 0) {
        const u32 tp = x_pair(2 * tj + (wid >> 2)), so = OFF_ST + ((((rank ^ x_dist(tp)) * 4 + x_panel(tp)) * 4 + w_panel(tq)) * 8 + (wid & 3) * 2) * 4;
        const u32 old = __builtin_amdgcn_raw_ptr_buffer_atomic_add_i32(1, rm, lane < 2 ? lane * 4 : OOR, so, AGT);
        (void)__builtin_amdgcn_raw_ptr_buffer_atomic_add_i32(2, rm, lane < 2 && !(old & 3u) ? lane * 4 : OOR, so, AGT);
        todo = (u32)__ballot(old & 3u) & 3u;
    }
    __builtin_amdgcn_raw_buffer_store_b32(call << 12 | todo << 10 | id, rsrc(P.ws[rank ^ w_dist(tq)], WS_NEED), l0, fo, SYS);
}

static void make_schedule(Params& p) {
    const int ne = NE, up = UP, lead = LEAD, hw = HW, rnum = RNUM, wl = WL;
    p.pn = up | ne << 8 | GX << 16;
    int n1 = 0;
    const int xl = XL;
    for (int t = -lead; t < 128 + xl; ++t) {
        const int wi = t + lead;
        if (t >= 0 && t < 128) p.tab[1][n1++] = t;
        if (t >= xl) p.tab[1][n1++] = (t - xl) | 1 << 7;
        if (wi < 128 && (wi < 4 * up || wi >= 4 * (up + ne))) { p.tab[1][n1++] = wi | 2 << 7; p.tab[1][n1++] = wi | 3 << 7; }
    }
    int n0 = 0, nd = 0, r = 0;
    auto direct = [&](int e) { p.tab[0][n0++] = e; if (++nd >= hw) for (int k = 0; k < rnum && r < 16 * ne; ++k, ++r) p.tab[0][n0++] = (r >> 2) | (r & 3) << 7 | 1 << 9; };
    for (int t = -wl; t < 128; ++t) { if (t + wl < 128) direct(t + wl); if (t >= 0) direct(t | 1 << 7); }
    for (; r < 16 * ne; ++r) p.tab[0][n0++] = (r >> 2) | (r & 3) << 7 | 1 << 9;
    for (int i = n0; i < 512; ++i) p.tab[0][i] = 0;
    for (int i = n1; i < 512; ++i) p.tab[1][i] = 0;
    int g = 0, nl = 0, qid = 0;
    auto add = [&](int ng, u32 d, u32 ea, u32 eb, u32 slots, u32 tab) {
        for (int i = 0; i < ng; ++i) {
            Group G = {{d | qid << 8 | (8 * ng) << 16 | (8 * i) << 24, ea | eb << 16, slots, tab}, p.ws[p.rank ^ d], {}};
            for (u32 q = 0; q < 16; ++q) { const u32 pos = 8 * i + (q & 7) + (q >> 3) * 8 * ng; G.e0[q & 7] |= ((tab & 3) ? p.tab[tab & 1][pos] : (pos >> 1) | (pos & 1) << 7) << ((q >> 3) * 16); }
            p.grp[d ? g++ : 14 + nl++] = G;
        }
        ++qid;
    };
    for (u32 dd = 1; dd <= 8; ++dd) {
        const u32 d = dd & 7;
        u32 xs[4], ws[4]; int nx = 0, nwv = 0;
        for (u32 s = 0; s < 16; ++s) { if (x_dist(s) == d) xs[nx++] = s; if (w_dist(s) == d) ws[nwv++] = 16 + s; }
        if (d == 0) for (u32 h = 0; h < 2; ++h) add(4, 0, 0, 256, 0, (2 * h) << 4);
        else if (d == 2) add(2, d, 512, 512, xs[0] | xs[1] << 8 | xs[2] << 16 | xs[3] << 24, D2F << 8);
        else if (d & 1) add(2, d, n1, n1 + 256, xs[0] | xs[1] << 8 | ws[0] << 16 | ws[1] << 24, 1 | d << 4);
        else add(2, d, n0, n0 + 256, ws[0] | ws[1] << 8, 2 | d << 4);
    }
}

}

extern "C" void kernel_launch(int rank, int nranks, ncclComm_t comm,
                              hipStream_t stream,
                              void* const* d_in, const int* in_sizes, int n_in,
                              void* d_out, int out_size,
                              const int* in_dtypes, int out_dtype,
                              void* const* d_out_all,
                              void* const* d_ws_all, size_t ws_bytes) {
    if (nranks != NR || (unsigned)rank >= (unsigned)NR || ws_bytes < WS_NEED || n_in < 2 ||
        in_sizes[0] != 512 * 4096 || in_sizes[1] != 4096 * 512 || out_size != 4096 * 512) {
        fprintf(stderr, "unexpected problem shape\n");
        return;
    }
    Params p;
    p.x = (const char*)d_in[0];
    p.w = (const char*)d_in[1];
    p.out = (char*)d_out;
    for (int i = 0; i < NR; ++i) p.ws[i] = (char*)d_ws_all[i];
    p.rank = rank;
    p.own = p.ws[rank];
    make_schedule(p);
    hipLaunchKernelGGL(ag_gemm_kernel, dim3(256), dim3(768), 0, stream, p);
    HIP_CHECK(hipGetLastError());
}
```
